```python
import numpy as np

kernel_name = "gpu_gemm_rs_m4096_k4096_n4096_e4m3e5m2_dqrelu"


ABSMAX_RTOL = 0.05

COMPUTE_DTYPE = "float32"
EPILOGUE = "dequant_relu"
IN_QUANT = None
OUT_QUANT = None
IN_FP8 = "e4m3e5m2"
IN_DTYPES = {"x": "e4m3", "w_mat": "e5m2",
             "scale_x": "float32", "scale_w": "float32"}
WORLD_SIZE = 8

def _snap_fp8(x, mant_bits, max_abs, min_normal):
    a = np.abs(x.astype(np.float64))
    _m, _e = np.frexp(a)
    q = np.where(a >= min_normal, np.ldexp(1.0, _e - 1 - mant_bits),
                 min_normal * 2.0 ** -mant_bits)
    snapped = np.minimum(np.round(a / q) * q, max_abs)
    return (np.sign(x) * snapped).astype(np.float32)

def setup_inputs():

    x = _snap_fp8(np.random.randn(4096, 4096).astype(np.float32), 3, 448.0, 2.0 ** -6)
    w_mat = _snap_fp8(np.random.randn(4096, 4096).astype(np.float32), 2, 57344.0, 2.0 ** -14)
    scale_x = (0.02 * (1.0 + np.random.rand(1))).astype(np.float32)
    scale_w = (0.02 * (1.0 + np.random.rand(1))).astype(np.float32)
    return {"x": x, "w_mat": w_mat, "scale_x": scale_x, "scale_w": scale_w}

def reference(x, w_mat, scale_x, scale_w):
    acc = (x @ w_mat).astype(np.float32)
    y = (acc * (scale_x * scale_w)).astype(np.float32)
    return np.maximum(y, 0.0).astype(np.float32)
```

```cpp
#include <hip/hip_runtime.h>
#include <rccl/rccl.h>
#include <cstdio>

#define HIP_CHECK(expr) do { hipError_t _e = (expr); if (_e != hipSuccess) { \
    fprintf(stderr, "HIP error %s at %s:%d\n", hipGetErrorString(_e), __FILE__, __LINE__); \
    return; } } while (0)

typedef unsigned u32;
typedef unsigned long long u64;
typedef unsigned u32x4 __attribute__((ext_vector_type(4)));
typedef unsigned u32x2 __attribute__((ext_vector_type(2)));
typedef int i32x8 __attribute__((ext_vector_type(8)));
typedef float f32x4 __attribute__((ext_vector_type(4)));
#define GAS __attribute__((address_space(1)))
#define LAS __attribute__((address_space(3)))

#define WIN 14
#define XTH 24
#define PUSH_LAG 100000
#define PUSH_WIN 1
#define PUSH_XCDS 8
#define FENCE_MODE 1
#define NSW 11
#define NSX 6

constexpr int NR = 8;
constexpr int NWG = 256, NTHR = 448;
constexpr int LDX = 512, LDW = 4096, LDO = 4096;
constexpr int MAIN_B = 6144;
constexpr int BLK_B = 8320;
constexpr int XT_B = 8192;
constexpr int NSTEP = 64;

constexpr size_t WST_OFF = 0, WST_PAR = 4u << 20;
constexpr size_t XRV_OFF = 8u << 20, XRV_PAR = 2u << 20;
constexpr size_t WFL_OFF = 12u << 20, WFL_PAR = 16u << 10;
constexpr size_t XFL_OFF = WFL_OFF + (32u << 10), XFL_PAR = 4u << 10;
constexpr size_t CNT_OFF = WFL_OFF + (64u << 10);
constexpr size_t TMO_OFF = WFL_OFF + (128u << 10);
constexpr size_t RDV_OFF = TMO_OFF + 1024;
constexpr size_t DBG_OFF = WFL_OFF + (256u << 10);
constexpr size_t WS_NEED = 13u << 20;
static_assert((size_t)256 * BLK_B <= WST_PAR, "w staging");

constexpr int L_W = 0;
constexpr int L_X = L_W + NSW * BLK_B;
constexpr int L_MASK = L_X + NSX * XT_B;
constexpr int L_EXC = L_MASK + 8192;
constexpr int L_CTL = L_EXC + 2176;
constexpr int LDS_BYTES = L_CTL + 128;
static_assert(LDS_BYTES <= 160 * 1024, "LDS");
constexpr int C_FULLW = 0, C_FULLX = 1, C_FAIL = 2, C_FREE = 4  , C_ALLOC = 8, C_ARR1 = 9, C_ARR2 = 10, C_TPUB = 11;

constexpr u64 DEADLINE = 30000000ull;

struct Params {
    const unsigned char* x;
    const unsigned char* w;
    const float* sx;
    const float* sw;
    float* out;
    void* const* ws_all;
    char* ws;
    int rank;
    int pad;
};

__device__ __forceinline__ u64 rt() { return __builtin_amdgcn_s_memrealtime(); }
__device__ __forceinline__ u32 rfl(u32 v) { return (u32)__builtin_amdgcn_readfirstlane(v); }
__device__ __forceinline__ u32 rdl(u32 v, int l) { return (u32)__builtin_amdgcn_readlane(v, l); }
#define WAIT_VM(n) asm volatile("s_waitcnt vmcnt(%0)" :: "i"(n) : "memory")
#define WAIT_LGKM0() asm volatile("s_waitcnt lgkmcnt(0)" ::: "memory")

template <int OFF, bool SYS>
__device__ __forceinline__ void dma16(u64 base, u32 voff, u32 ldsaddr) {
    u32 keep;
    if constexpr (SYS)
        asm volatile("s_mov_b32 %0, m0\n\ts_mov_b32 m0, %3\n\ts_nop 3\n\tglobal_load_lds_dwordx4 %1, %2 offset:%4 sc0 sc1\n\ts_mov_b32 m0, %0"
                     : "=&s"(keep) : "v"(voff), "s"(base), "s"(ldsaddr), "i"(OFF) : "memory");
    else
        asm volatile("s_mov_b32 %0, m0\n\ts_mov_b32 m0, %3\n\ts_nop 3\n\tglobal_load_lds_dwordx4 %1, %2 offset:%4\n\ts_mov_b32 m0, %0"
                     : "=&s"(keep) : "v"(voff), "s"(base), "s"(ldsaddr), "i"(OFF) : "memory");
}
__device__ __forceinline__ void st16_sys(void* p, u32x4 v) {
    asm volatile("global_store_dwordx4 %0, %1, off sc0 sc1\n\ts_nop 1" :: "v"(p), "v"(v) : "memory");
}
__device__ __forceinline__ void st8_sys(void* p, u32x2 v) {
    asm volatile("global_store_dwordx2 %0, %1, off sc0 sc1\n\ts_nop 1" :: "v"(p), "v"(v) : "memory");
}
__device__ __forceinline__ u32 wave_scan_incl(u32 v) {
    v += __builtin_amdgcn_update_dpp(0, v, 0x111, 0xf, 0xf, true);
    v += __builtin_amdgcn_update_dpp(0, v, 0x112, 0xf, 0xf, true);
    v += __builtin_amdgcn_update_dpp(0, v, 0x114, 0xf, 0xf, true);
    v += __builtin_amdgcn_update_dpp(0, v, 0x118, 0xf, 0xf, true);
    v += __builtin_amdgcn_update_dpp(0, v, 0x142, 0xa, 0xf, false);
    v += __builtin_amdgcn_update_dpp(0, v, 0x143, 0xc, 0xf, false);
    return v;
}

__device__ __forceinline__ void step_map(int s, int rank, int grp, int& tile, int& src, int& ks) {
    tile = s >> 5; ks = (s >> 3) & 3;
    const int i = s & 7, op = grp % 7;
    src = i == op ? rank : (rank + 1 + (grp + i - (i > op)) % 7) & 7;
}

#define STAMP(i) do {} while (0)

__global__ void __launch_bounds__(NTHR, 2) gemm_xchg(Params p) {
    extern __shared__ __attribute__((aligned(16))) unsigned char lds_raw[];
    LAS unsigned char* L = (LAS unsigned char*)lds_raw;
    const u32 lbase = (u32)(size_t)L;
    volatile LAS u32* ctl = (volatile LAS u32*)(L + L_CTL);
    const int tid = threadIdx.x, lane = tid & 63;
    const int wave = rfl(tid >> 6);
    const int bid = blockIdx.x, xc = bid & 7, jj = bid >> 3;
    const int rank = p.rank;
    const int rb = jj & 7, grp = xc * 4 + (jj >> 3);
    GAS u32* tmo = (GAS u32*)(p.ws + TMO_OFF);
    STAMP(0);

    GAS u32* cntp = (GAS u32*)(p.ws + CNT_OFF) + bid * 8 + wave;
    u32 call = __hip_atomic_load(cntp, __ATOMIC_RELAXED, __HIP_MEMORY_SCOPE_AGENT);

    if (wave < 4) {
        const int cbe = (jj & 1) + 2 * (xc + 8 * ((jj >> 1) & 3)), kse = jj >> 3;
        const u32 mbuf = lbase + L_MASK + wave * 2048;
        {
            const u64 src = (u64)p.w + (u64)(kse * 128) * LDW + cbe * 64 + wave * 16;
            dma16<0, false>(src, (u32)lane * LDW, mbuf);
            dma16<0, false>(src, (u32)(lane + 64) * LDW, mbuf + 1024);
        }
        if (wave == 0 && lane < 16) ctl[lane] = 0;
        float scale = p.sx[0] * p.sw[0];
        char* fpeer = (char*)p.ws_all[lane & 7];
        WAIT_LGKM0();
        __builtin_amdgcn_s_barrier();
        call = rfl(call) + 1;
        const u32 par = call & 1;
        WAIT_VM(0);
        u32 x[8];
        {
            u32x2 t0, t1, t2, t3;
            const u32 a = mbuf + (u32)(((lane >> 4) * 16 + ((lane & 15) >> 1)) * 16 + (lane & 1) * 8);
            asm volatile("ds_read_b64_tr_b8 %0, %4\n\tds_read_b64_tr_b8 %1, %4 offset:128\n\t"
                         "ds_read_b64_tr_b8 %2, %4 offset:1024\n\tds_read_b64_tr_b8 %3, %4 offset:1152\n\ts_waitcnt lgkmcnt(0)"
                         : "=&v"(t0), "=&v"(t1), "=&v"(t2), "=&v"(t3) : "v"(a) : "memory");
            x[0] = t0.x; x[1] = t0.y; x[2] = t1.x; x[3] = t1.y; x[4] = t2.x; x[5] = t2.y; x[6] = t3.x; x[7] = t3.y;
        }
        asm volatile("" : "+v"(scale), "+v"(fpeer));
        STAMP(1);
        *(LAS u32x4*)(L + L_MASK + wave * 2048 + lane * 16) = u32x4{0, 0, 0, 0};
        *(LAS u32x4*)(L + L_MASK + wave * 2048 + 1024 + lane * 16) = u32x4{0, 0, 0, 0};
        u32 sum = 0;
#pragma unroll
        for (int i = 0; i < 8; ++i) sum = __builtin_amdgcn_sad_u8(x[i] & 0x7F7F7F7Fu, 0u, sum);
        sum = rdl(wave_scan_incl(sum), 63);
        int lo = ((int)sum - 41165) >> 11;
        lo = lo < 0 ? 0 : (lo > 96 ? 96 : lo);
        const u32 LOe = (u32)lo * 0x01010101u, CLe = (u32)(128 - lo) * 0x01010101u;
        u32 code[8], mk[8], cnt = 0;
#pragma unroll
        for (int i = 0; i < 8; ++i) {
            const u32 d = ((x[i] & 0x7F7F7F7Fu) + CLe) & 0x1F1F1F1Fu;
            code[i] = (x[i] & 0x80808080u) | d;
            mk[i] = (x[i] ^ (d + LOe)) & 0x60606060u;
            cnt += __builtin_popcount((mk[i] | (mk[i] << 1)) & 0x40404040u);
        }
        u32x4 P0; u32x2 P1;
        P0.x = code[0] | ((code[6] & 0x03030303u) << 5);
        P0.y = code[1] | ((code[6] & 0x0C0C0C0Cu) << 3);
        P0.z = code[2] | ((code[6] & 0x10101010u) << 1) | ((code[6] & 0x80808080u) >> 1);
        P0.w = code[3] | ((code[7] & 0x03030303u) << 5);
        P1.x = code[4] | ((code[7] & 0x0C0C0C0Cu) << 3);
        P1.y = code[5] | ((code[7] & 0x10101010u) << 1) | ((code[7] & 0x80808080u) >> 1);
        char* blk = p.ws + WST_OFF + par * WST_PAR + (size_t)(cbe * 4 + kse) * BLK_B;
        st16_sys(blk + wave * 1536 + lane * 16, P0);
        st8_sys(blk + wave * 1536 + 1024 + lane * 8, P1);
        const u32 incl = wave_scan_incl(cnt);
        const u32 total = rdl(incl, 63);
        const bool dense = total > 255;
        const u32 bytes = dense ? 512u : ((total * 2 + 7) & ~7u);
        u32 start = 0;
        if (lane == 0) start = __hip_atomic_fetch_add((LAS u32*)(L + L_CTL) + C_ALLOC, bytes, __ATOMIC_RELAXED, __HIP_MEMORY_SCOPE_WORKGROUP);
        start = rfl(start) + 16;
        if (dense) {
            u32x2 D;
            D.x = (mk[0] >> 5) | (mk[1] >> 3) | (mk[2] >> 1) | (mk[3] << 1);
            D.y = (mk[4] >> 5) | (mk[5] >> 3) | (mk[6] >> 1) | (mk[7] << 1);
            *(LAS u32x2*)(L + L_EXC + start + lane * 8) = D;
        } else if (cnt) {
            u32 o = L_EXC + start + (incl - cnt) * 2;
#pragma unroll
            for (int i = 0; i < 8; ++i) {
                u32 nz = (mk[i] | (mk[i] << 1)) & 0x40404040u;
                while (nz) {
                    const u32 b = __builtin_ctz(nz) >> 3;
                    const u32 e = (u32)((i >> 2) * 1024 + (i & 3) * 4) + lane * 16 + b + (((mk[i] >> (8 * b + 5)) & 3u) << 11);
                    *(LAS unsigned short*)(L + o) = (unsigned short)e;
                    o += 2; nz &= nz - 1;
                }
            }
        }
        if (lane == 0) *(LAS u32*)(L + L_EXC + wave * 4) = (u32)lo | (dense ? 128u : 0u) | (start << 8) | ((dense ? 0u : total) << 20);
        WAIT_LGKM0();
        u32 a1 = 0;
        if (lane == 0) a1 = __hip_atomic_fetch_add((LAS u32*)(L + L_CTL) + C_ARR1, 1u, __ATOMIC_RELAXED, __HIP_MEMORY_SCOPE_WORKGROUP);
        if (rfl(a1) == 3) {
            const u32 n16 = (ctl[C_ALLOC] + 31) >> 4;
            for (u32 i = lane; i < n16; i += 64) st16_sys(blk + MAIN_B + i * 16, *(volatile LAS u32x4*)(L + L_EXC + i * 16));
        }
        WAIT_VM(0);
        u32 a2 = 0;
        if (lane == 0) a2 = __hip_atomic_fetch_add((LAS u32*)(L + L_CTL) + C_ARR2, 1u, __ATOMIC_RELAXED, __HIP_MEMORY_SCOPE_WORKGROUP);
        if (rfl(a2) == 3) {
            const u32 n16 = (ctl[C_ALLOC] + 31) >> 4;
            if (lane < NR) {
                GAS u64* f = (GAS u64*)(fpeer + WFL_OFF + par * WFL_PAR) + ((cbe * 8 + rank) * 4 + kse);
                __hip_atomic_store(f, ((u64)call << 32) | n16, __ATOMIC_RELAXED, __HIP_MEMORY_SCOPE_SYSTEM);
            }
            ctl[C_TPUB] = (u32)rt();
            ctl[C_ARR2] = 5;
        }
        STAMP(2);

        f32x4 accA[4], accB[4];
#pragma unroll
        for (int f = 0; f < 4; ++f) { accA[f] = f32x4{0, 0, 0, 0}; accB[f] = f32x4{0, 0, 0, 0}; }
        u32 seenW = 0, seenX = 0;
        int sw = 0, sx = 0;
        auto step = [&](int s, f32x4 (&acc)[4]) {
            if (seenW <= (u32)s || seenX <= (u32)s) {
                u64 t0 = 0;
                for (u32 it = 1;; ++it) {
                    const u32x2 fl = *(volatile LAS u32x2*)(L + L_CTL);
                    seenW = rfl(fl.x); seenX = rfl(fl.y);
                    if (seenW > (u32)s && seenX > (u32)s) break;
                    if ((it & 1023) == 0) {
                        const u64 t = rt();
                        if (t0 == 0) t0 = t;
                        if (t - t0 > 2 * DEADLINE) { ctl[C_FAIL] = 1; seenW = seenX = NSTEP; break; }
                    }
                    __builtin_amdgcn_s_sleep(1);
                }
                asm volatile("" ::: "memory");
            }
            LAS unsigned char* W = L + L_W + sw * BLK_B;
            LAS unsigned char* X = L + L_X + sx * XT_B;
            const u32 hdr = rfl(*(volatile LAS u32*)(W + MAIN_B + wave * 4));
            const u32 st = (hdr >> 8) & 0xFFEu, n = (hdr >> 20) & 0xFFu;
            LAS unsigned char* MB = L + L_MASK + wave * 2048;
            const u32 e0 = *(volatile LAS unsigned short*)(W + MAIN_B + st + lane * 2);
            const u32x4 q0 = *(volatile LAS u32x4*)(W + wave * 1536 + lane * 16);
            const u32x2 q1 = *(volatile LAS u32x2*)(W + wave * 1536 + 1024 + lane * 8);
            u32x4 xa[4][2];
#pragma unroll
            for (int f = 0; f < 2; ++f) {
                xa[f][0] = *(volatile LAS u32x4*)(X + f * 2048 + lane * 16);
                xa[f][1] = *(volatile LAS u32x4*)(X + f * 2048 + 1024 + lane * 16);
            }
            LAS unsigned char* ea = (u32)lane < n ? MB + (e0 & 2047u) : L + L_CTL + 64 + lane;
            *(volatile LAS unsigned char*)ea = (unsigned char)((e0 >> 6) & 0x60u);
            if (n > 64)
                for (u32 i = lane + 64; i < n; i += 64) {
                    const u32 e = *(volatile LAS unsigned short*)(W + MAIN_B + st + i * 2);
                    *(volatile LAS unsigned char*)(MB + (e & 2047u)) = (unsigned char)((e >> 6) & 0x60u);
                }
            u32x4 m0 = *(volatile LAS u32x4*)(MB + lane * 16);
            u32x4 m1 = *(volatile LAS u32x4*)(MB + 1024 + lane * 16);
            *(volatile LAS unsigned char*)ea = 0;
#pragma unroll
            for (int f = 2; f < 4; ++f) {
                xa[f][0] = *(volatile LAS u32x4*)(X + f * 2048 + lane * 16);
                xa[f][1] = *(volatile LAS u32x4*)(X + f * 2048 + 1024 + lane * 16);
            }
            if (n > 64)
                for (u32 i = lane + 64; i < n; i += 64) {
                    const u32 e = *(volatile LAS unsigned short*)(W + MAIN_B + st + i * 2);
                    *(volatile LAS unsigned char*)(MB + (e & 2047u)) = 0;
                }
            if (hdr & 128u) {
                const u32x2 D = *(volatile LAS u32x2*)(W + MAIN_B + (st & 0xFF8u) + lane * 8);
                m0.x = (D.x << 5) & 0x60606060u; m0.y = (D.x << 3) & 0x60606060u; m0.z = (D.x << 1) & 0x60606060u; m0.w = (D.x >> 1) & 0x60606060u;
                m1.x = (D.y << 5) & 0x60606060u; m1.y = (D.y << 3) & 0x60606060u; m1.z = (D.y << 1) & 0x60606060u; m1.w = (D.y >> 1) & 0x60606060u;
            }
            const u32 LO = (hdr & 0x7Fu) * 0x01010101u;
            const u32 e6 = ((q0.x >> 5) & 0x03030303u) | ((q0.y >> 3) & 0x0C0C0C0Cu) | ((q0.z >> 1) & 0x10101010u) | ((q0.z << 1) & 0x80808080u);
            const u32 e7 = ((q0.w >> 5) & 0x03030303u) | ((q1.x >> 3) & 0x0C0C0C0Cu) | ((q1.y >> 1) & 0x10101010u) | ((q1.y << 1) & 0x80808080u);
            i32x8 wf;
            wf[0] = ((q0.x & 0x9F9F9F9Fu) + LO) ^ m0.x;
            wf[1] = ((q0.y & 0x9F9F9F9Fu) + LO) ^ m0.y;
            wf[2] = ((q0.z & 0x9F9F9F9Fu) + LO) ^ m0.z;
            wf[3] = ((q0.w & 0x9F9F9F9Fu) + LO) ^ m0.w;
            wf[4] = ((q1.x & 0x9F9F9F9Fu) + LO) ^ m1.x;
            wf[5] = ((q1.y & 0x9F9F9F9Fu) + LO) ^ m1.y;
            wf[6] = (e6 + LO) ^ m1.z;
            wf[7] = (e7 + LO) ^ m1.w;
            asm volatile("s_waitcnt lgkmcnt(0)" : "+v"(xa[0][0]), "+v"(xa[0][1]), "+v"(xa[1][0]), "+v"(xa[1][1]),
                         "+v"(xa[2][0]), "+v"(xa[2][1]), "+v"(xa[3][0]), "+v"(xa[3][1]), "+v"(wf) :: "memory");
            ctl[C_FREE + wave] = s + 1;
#pragma unroll
            for (int f = 0; f < 4; ++f) {
                i32x8 xf;
                xf[0] = xa[f][0].x; xf[1] = xa[f][0].y; xf[2] = xa[f][0].z; xf[3] = xa[f][0].w;
                xf[4] = xa[f][1].x; xf[5] = xa[f][1].y; xf[6] = xa[f][1].z; xf[7] = xa[f][1].w;
                acc[f] = __builtin_amdgcn_mfma_scale_f32_16x16x128_f8f6f4(wf, xf, acc[f], 1  , 0  , 0, 0, 0, 0);
            }
            sw = sw + 1 == NSW ? 0 : sw + 1;
            sx = sx + 1 == NSX ? 0 : sx + 1;
        };
        auto epilogue = [&](f32x4 (&acc)[4], int cb) {
            const bool fail = rfl(ctl[C_FAIL]) != 0;
            float* o = p.out + (size_t)(rb * 64 + (lane & 15)) * LDO + cb * 64 + wave * 16 + (lane >> 4) * 4;
#pragma unroll
            for (int f = 0; f < 4; ++f) {
                f32x4 y = acc[f] * scale;
#pragma unroll
                for (int j = 0; j < 4; ++j) { y[j] = y[j] < 0.f ? 0.f : y[j]; if (fail) y[j] = __builtin_nanf(""); }
                *(f32x4*)(o + (size_t)f * 16 * LDO) = y;
            }
        };
#pragma nounroll
        for (int s = 0; s < 32; ++s) { step(s, accA); if (s == 0) STAMP(3); }
        STAMP(4);
        epilogue(accA, grp);
#pragma nounroll
        for (int s = 32; s < 64; ++s) step(s, accB);
        STAMP(5);
        epilogue(accB, grp + 32);
        STAMP(6);
    } else if (wave == 4) {
        int tile, src, ks;
        step_map(lane, rank, grp, tile, src, ks);
        const int cb = grp + tile * 32;
        const u64 peer = (u64)p.ws_all[src];
        __builtin_amdgcn_s_barrier();
        call = rfl(call) + 1;
        const u32 par = call & 1;
        const u64 baddr = peer + WST_OFF + par * WST_PAR + (u64)(cb * 4 + ks) * BLK_B;
        const u32 blo = (u32)baddr, bhi = (u32)(baddr >> 32);
        GAS u64* fl = (GAS u64*)(p.ws + WFL_OFF + par * WFL_PAR) + ((cb * 8 + src) * 4 + ks);
        const u32 voff = lane * 16;
        u32 n16v = 1;
        bool dead = false;
        auto wait_flags = [&](u64 need) {
            const u64 t0 = rt();
            for (u32 it = 0;; ++it) {
                const u64 v = __hip_atomic_load(fl, __ATOMIC_RELAXED, __HIP_MEMORY_SCOPE_SYSTEM);
                const u64 ok = __ballot((u32)(v >> 32) >= call);
                n16v = (u32)v;
                if ((ok & need) == need) break;
                if ((it & 15) == 15 && (rfl(__hip_atomic_load(tmo, __ATOMIC_RELAXED, __HIP_MEMORY_SCOPE_AGENT)) == call || rt() - t0 > DEADLINE)) {
                    __hip_atomic_store(tmo, call, __ATOMIC_RELAXED, __HIP_MEMORY_SCOPE_AGENT);
                    dead = true; break;
                }
                __builtin_amdgcn_s_sleep(2);
            }
            n16v = n16v < 1 ? 1 : (n16v > 129 ? 129 : n16v);
        };
        int issued = 0, pub = 0, endpub = 0, freed = 0, slot = 0;
        auto cnt_of = [&](int s) { return 6 + (int)((rdl(n16v, s) + 63) >> 6); };
        auto tick = [&]() {
            ++issued;
            WAIT_VM(WIN);
            if (issued - WIN >= endpub) { ++pub; ctl[C_FULLW] = pub; endpub += cnt_of(pub & 63); }
        };
        auto drain = [&](int s) {
            WAIT_VM(0);
            if (pub < s) { pub = s; ctl[C_FULLW] = pub; }
            issued = 0; endpub = s < NSTEP ? cnt_of(s) : 0;
        };
        auto load_step = [&](int s, auto sys) {
            constexpr bool SYS = decltype(sys)::value;
            if (s - freed >= NSW) {
                const u64 t0 = rt();
                bool drained = false;
                for (;;) {
                    const u32x4 fr = *(volatile LAS u32x4*)(L + L_CTL + C_FREE * 4);
                    freed = rfl(min(min(fr.x, fr.y), min(fr.z, fr.w)));
                    if (s - freed < NSW) break;
                    if (!drained) { drain(s); drained = true; continue; }
                    if (rt() - t0 > 2 * DEADLINE) { dead = true; break; }
                    __builtin_amdgcn_s_sleep(1);
                }
                if (dead) return;
            }
            const u64 b = ((u64)rdl(bhi, s) << 32) | rdl(blo, s);
            const u32 n16 = rdl(n16v, s);
            const u32 la = lbase + L_W + slot * BLK_B;
            dma16<0, SYS>(b, voff, la); tick();
            dma16<1024, SYS>(b, voff, la); tick();
            dma16<2048, SYS>(b, voff, la); tick();
            dma16<3072, SYS>(b, voff, la); tick();
            dma16<0, SYS>(b + 4096, voff, la + 4096); tick();
            dma16<1024, SYS>(b + 4096, voff, la + 4096); tick();
            for (u32 i = 0; i < n16; i += 64) {
                if (lane + i < n16) dma16<0, SYS>(b + MAIN_B + i * 16, voff, la + MAIN_B + i * 16);
                tick();
            }
            slot = slot + 1 == NSW ? 0 : slot + 1;
        };
        if (FENCE_MODE == 2) { __builtin_amdgcn_fence(__ATOMIC_ACQUIRE, ""); WAIT_VM(0); STAMP(3); }
        wait_flags(~0ull);
        STAMP(1);
        if (FENCE_MODE != 2) { __builtin_amdgcn_fence(__ATOMIC_ACQUIRE, ""); WAIT_VM(0); }
        STAMP(2);
        if (!dead) {
            GAS u64* rv = (GAS u64*)(p.ws + RDV_OFF + xc * 128);
            if (FENCE_MODE == 1 && lane == 0) __hip_atomic_fetch_add(rv, 1ull << 27, __ATOMIC_RELAXED, __HIP_MEMORY_SCOPE_AGENT);
            GAS u64* xf = lane < 32 ? (GAS u64*)(p.ws + XFL_OFF + par * XFL_PAR) + (rb * 32 + lane) : rv;
            const u64 t0 = rt();
            for (;;) {
                const u64 v = __hip_atomic_load(xf, __ATOMIC_RELAXED, __HIP_MEMORY_SCOPE_SYSTEM);
                const u64 ok = __ballot((u32)(v >> 32) >= call);
                if ((__builtin_popcount((u32)ok) >= XTH && (FENCE_MODE != 1 || (ok >> 32))) || rt() - t0 > 2000) break;
                __builtin_amdgcn_s_sleep(2);
            }
            if (rfl(__hip_atomic_load(tmo, __ATOMIC_RELAXED, __HIP_MEMORY_SCOPE_AGENT)) == call) dead = true;
            endpub = cnt_of(0);
        }
        STAMP(4);
        for (int s = 0; s < NSTEP && !dead; ++s) load_step(s, std::false_type{});
        STAMP(5);
        if (dead) { ctl[C_FAIL] = 1; WAIT_VM(0); ctl[C_FULLW] = NSTEP; }
        else {
            WAIT_VM(7);
            if (issued - 7 >= endpub) { ++pub; ctl[C_FULLW] = pub; }
            drain(NSTEP);
        }
        STAMP(6);
    } else if (wave == 5) {
        int tile, src, ks;
        step_map(lane, rank, grp, tile, src, ks);
        __builtin_amdgcn_s_barrier();
        call = rfl(call) + 1;
        const u32 par = call & 1;
        const u32 toff = (u32)((src * 8 + rb) * 4 + ks) * XT_B;
        const u64 xbase = (u64)p.ws + XRV_OFF + par * XRV_PAR;
        GAS u64* fl = (GAS u64*)(p.ws + XFL_OFF + par * XFL_PAR) + ((rb * 8 + src) * 4 + ks);
        const u32 voff = lane * 16;
        u64 ready = 0;
        int pub = 0, freed = 0, slot = 0;
        bool dead = false;
        for (int s = 0; s < NSTEP; ++s) {
            if (!((ready >> s) & 1) || s - freed >= NSX) {
                const u32x4 fr0 = *(volatile LAS u32x4*)(L + L_CTL + C_FREE * 4);
                freed = rfl(min(min(fr0.x, fr0.y), min(fr0.z, fr0.w)));
                if (!((ready >> s) & 1) || s - freed >= NSX) {
                    WAIT_VM(0);
                    if (pub < s) { pub = s; ctl[C_FULLX] = pub; }
                    const u64 t0 = rt();
                    for (u32 it = 0; !((ready >> s) & 1); ++it) {
                        const u64 v = __hip_atomic_load(fl, __ATOMIC_RELAXED, __HIP_MEMORY_SCOPE_SYSTEM);
                        ready = __ballot((u32)(v >> 32) >= call);
                        if (ready == ~0ull && rfl(__hip_atomic_load(tmo, __ATOMIC_RELAXED, __HIP_MEMORY_SCOPE_AGENT)) == call) { dead = true; break; }
                        if ((ready >> s) & 1) break;
                        if ((it & 15) == 15 && (rfl(__hip_atomic_load(tmo, __ATOMIC_RELAXED, __HIP_MEMORY_SCOPE_AGENT)) == call || rt() - t0 > DEADLINE)) {
                            __hip_atomic_store(tmo, call, __ATOMIC_RELAXED, __HIP_MEMORY_SCOPE_AGENT);
                            dead = true; break;
                        }
                        __builtin_amdgcn_s_sleep(2);
                    }
                    while (!dead && s - freed >= NSX) {
                        const u32x4 fr = *(volatile LAS u32x4*)(L + L_CTL + C_FREE * 4);
                        freed = rfl(min(min(fr.x, fr.y), min(fr.z, fr.w)));
                        if (s - freed < NSX) break;
                        if (rt() - t0 > 2 * DEADLINE) { dead = true; break; }
                        __builtin_amdgcn_s_sleep(1);
                    }
                    if (dead) break;
                }
            }
            const u64 b = xbase + rdl(toff, s);
            const u32 la = lbase + L_X + slot * XT_B;
            dma16<0, true>(b, voff, la); dma16<1024, true>(b, voff, la); dma16<2048, true>(b, voff, la); dma16<3072, true>(b, voff, la);
            dma16<0, true>(b + 4096, voff, la + 4096); dma16<1024, true>(b + 4096, voff, la + 4096);
            dma16<2048, true>(b + 4096, voff, la + 4096); dma16<3072, true>(b + 4096, voff, la + 4096);
            slot = slot + 1 == NSX ? 0 : slot + 1;
            WAIT_VM(16);
            if (pub < s - 1) { pub = s - 1; ctl[C_FULLX] = pub; }
        }
        if (dead) ctl[C_FAIL] = 1;
        WAIT_VM(0);
        ctl[C_FULLX] = NSTEP;
        STAMP(6);
    } else {
        constexpr int PT = 8 / PUSH_XCDS;
        const int prb = jj >> 2, pks = jj & 3;
        u32x4 v[PT][8];
        char* peer[PT];
        if (xc < PUSH_XCDS) {
#pragma unroll
            for (int t = 0; t < PT; ++t) {
                const int d = (xc + t * PUSH_XCDS + jj) & 7;
                const unsigned char* g = p.x + (size_t)(d * 512 + prb * 64 + (lane & 15)) * LDX + pks * 128 + (lane >> 4) * 16;
#pragma unroll
                for (int i = 0; i < 8; ++i) v[t][i] = *(const GAS u32x4*)(g + (size_t)(i >> 1) * 16 * LDX + (i & 1) * 64);
                peer[t] = (char*)p.ws_all[d];
            }
        }
        __builtin_amdgcn_s_barrier();
        call = rfl(call) + 1;
        const u32 par = call & 1;
        if (xc < PUSH_XCDS) {
#pragma unroll
            for (int t = 0; t < PT; ++t)
                asm volatile("" : "+v"(v[t][0]), "+v"(v[t][1]), "+v"(v[t][2]), "+v"(v[t][3]), "+v"(v[t][4]), "+v"(v[t][5]), "+v"(v[t][6]), "+v"(v[t][7]), "+v"(peer[t]));
            const size_t off = XRV_OFF + par * XRV_PAR + (size_t)((rank * 8 + prb) * 4 + pks) * XT_B + lane * 16;
            bool open = false;
#pragma unroll
            for (int i = 0; i < 8; ++i)
#pragma unroll
                for (int t = 0; t < PT; ++t) {
                    st16_sys(peer[t] + off + i * 1024, v[t][i]);
                    if (!open) {
                        WAIT_VM(PUSH_WIN - 1);
                        if (rfl(ctl[C_ARR2]) == 5) {
                            const u64 now = rt();
                            const u32 tp = rfl(ctl[C_TPUB]);
                            open = (u32)now - tp >= PUSH_LAG;
                        }
                    }
                }
            STAMP(1);
            WAIT_VM(0);
            if (lane == 0) {
#pragma unroll
                for (int t = 0; t < PT; ++t) {
                    GAS u64* f = (GAS u64*)(peer[t] + XFL_OFF + par * XFL_PAR) + ((prb * 8 + rank) * 4 + pks);
                    __hip_atomic_store(f, (u64)call << 32, __ATOMIC_RELAXED, __HIP_MEMORY_SCOPE_SYSTEM);
                }
            }
            STAMP(2);
        }
    }
    if (lane == 0) __hip_atomic_store(cntp, call, __ATOMIC_RELAXED, __HIP_MEMORY_SCOPE_AGENT);
}

extern "C" void kernel_launch(int rank, int nranks, ncclComm_t comm,
                              hipStream_t stream,
                              void* const* d_in, const int* in_sizes, int n_in,
                              void* d_out, int out_size,
                              const int* in_dtypes, int out_dtype,
                              void* const* d_out_all,
                              void* const* d_ws_all, size_t ws_bytes) {
    (void)comm; (void)in_dtypes; (void)out_dtype; (void)d_out_all;
    if (nranks != NR || (unsigned)rank >= (unsigned)NR || n_in != 4 || ws_bytes < WS_NEED ||
        in_sizes[0] != 4096 * 512 || in_sizes[1] != 512 * 4096 || out_size != 512 * 4096) {
        fprintf(stderr, "unexpected problem shape\n");
        return;
    }
    HIP_CHECK(hipFuncSetAttribute((const void*)gemm_xchg, hipFuncAttributeMaxDynamicSharedMemorySize, LDS_BYTES));
    Params p;
    p.x = (const unsigned char*)d_in[0];
    p.w = (const unsigned char*)d_in[1];
    p.sx = (const float*)d_in[2];
    p.sw = (const float*)d_in[3];
    p.out = (float*)d_out;
    p.ws_all = d_ws_all;
    p.ws = (char*)d_ws_all[rank];
    p.rank = rank;
    p.pad = 0;
    hipLaunchKernelGGL(gemm_xchg, dim3(NWG), dim3(NTHR), LDS_BYTES, stream, p);
    HIP_CHECK(hipGetLastError());
}
```

```cpp
#include <hip/hip_runtime.h>
#include <rccl/rccl.h>
#include <cstdio>

#define HIP_CHECK(expr) do { hipError_t _e = (expr); if (_e != hipSuccess) { \
    fprintf(stderr, "HIP error %s at %s:%d\n", hipGetErrorString(_e), __FILE__, __LINE__); \
    return; } } while (0)

typedef unsigned u32;
typedef unsigned long long u64;
typedef unsigned u32x4 __attribute__((ext_vector_type(4)));
typedef unsigned u32x2 __attribute__((ext_vector_type(2)));
typedef int i32x8 __attribute__((ext_vector_type(8)));
typedef float f32x4 __attribute__((ext_vector_type(4)));
#define GAS __attribute__((address_space(1)))
#define LAS __attribute__((address_space(3)))

#define WIN 6
#define WIN2 6
#define NW2 2
#define LAZY 0
#define EPI 0
#define PULL_W 1
#define CH 3
#define NPUSH 32
#define XTH 12
#define PUSH_W0 2
#define PUSH_W1 2
#define PUSH_LAG 60
#define FENCE_MODE 3
#define XD 3
#define NSW 10
#define NSX 4

constexpr int NR = 8;
constexpr int NWG = 256, NTHR = 512;
constexpr int LDX = 512, LDW = 4096, LDO = 4096;
constexpr int MAIN_B = 6144, OVF_B = 6400;
constexpr int MAX16 = (256 + 16 + 4 * 512) / 16;
constexpr int BLK_B = 8576;
constexpr int BLK_G = 8704;
constexpr int XT_B = 8192;
constexpr int XS_B = 2 * XT_B;
constexpr int NSTEP = 32, NREM = 28;
static_assert(MAIN_B + MAX16 * 16 <= BLK_B && BLK_B <= BLK_G, "block");

constexpr size_t WST_OFF = 0, WST_PAR = 4u << 20;
constexpr size_t XRV_OFF = 8u << 20, XRV_PAR = 2u << 20;
constexpr size_t XST_OFF = 12u << 20, XST_PAR = 2u << 20;
constexpr size_t WFL_OFF = 16u << 20, WFL_PAR = 16u << 10;
constexpr size_t XFL_OFF = WFL_OFF + (32u << 10), XFL_PAR = 4u << 10;
constexpr size_t XSF_OFF = WFL_OFF + (40u << 10), XSF_PAR = 4u << 10;
constexpr size_t CNT_OFF = WFL_OFF + (64u << 10);
constexpr size_t TMO_OFF = WFL_OFF + (128u << 10);
constexpr size_t INV_OFF = TMO_OFF + 1024;
constexpr size_t DBG_OFF = WFL_OFF + (256u << 10);
constexpr size_t WS_NEED = 17u << 20;
static_assert((size_t)256 * BLK_G <= WST_PAR, "w staging");

constexpr int L_W = 0;
constexpr int L_X = L_W + NSW * BLK_B;
constexpr int L_MASK = L_X + NSX * XS_B;
constexpr int L_CTL = L_MASK + 8192;
constexpr int LDS_BYTES = L_CTL + 128;
static_assert(LDS_BYTES <= 160 * 1024 && NSW >= 8 && XD <= NSX && XD <= 4, "LDS");
constexpr int C_FULLW = 0, C_FULLO = 1, C_FULLX = 2, C_FAIL = 3, C_FREE = 4  , C_ALLOC = 8, C_ARR = 9, C_TPUB = 10, C_GO = 11;

constexpr u64 DEADLINE = 30000000ull;

struct Params {
    const unsigned char* x;
    const unsigned char* w;
    const float* sx;
    const float* sw;
    float* out;
    void* const* ws_all;
    char* ws;
    int rank;
    int pad;
};

__device__ __forceinline__ u64 rt() { return __builtin_amdgcn_s_memrealtime(); }
__device__ __forceinline__ u32 rfl(u32 v) { return (u32)__builtin_amdgcn_readfirstlane(v); }
__device__ __forceinline__ u32 rdl(u32 v, int l) { return (u32)__builtin_amdgcn_readlane(v, l); }
#define WAIT_VM(n) asm volatile("s_waitcnt vmcnt(%0)" :: "i"(n) : "memory")
#define WAIT_LGKM0() asm volatile("s_waitcnt lgkmcnt(0)" ::: "memory")

template <int OFF, int POL, int NOP = 0>
__device__ __forceinline__ void dma16(u64 base, u32 voff, u32 ldsaddr) {
    if constexpr (POL == 1)
        asm volatile("s_mov_b32 m0, %2\n\ts_nop %4\n\tglobal_load_lds_dwordx4 %0, %1 offset:%3 sc0 sc1"
                     :: "v"(voff), "s"(base), "s"(ldsaddr), "i"(OFF), "i"(NOP) : "memory");
    else if constexpr (POL == 2)
        asm volatile("s_mov_b32 m0, %2\n\ts_nop %4\n\tglobal_load_lds_dwordx4 %0, %1 offset:%3 nt"
                     :: "v"(voff), "s"(base), "s"(ldsaddr), "i"(OFF), "i"(NOP) : "memory");
    else
        asm volatile("s_mov_b32 m0, %2\n\ts_nop %4\n\tglobal_load_lds_dwordx4 %0, %1 offset:%3"
                     :: "v"(voff), "s"(base), "s"(ldsaddr), "i"(OFF), "i"(NOP) : "memory");
}
__device__ __forceinline__ void st4_sys(void* p, u32 v) {
    asm volatile("global_store_dword %0, %1, off sc0 sc1\n\ts_nop 1" :: "v"(p), "v"(v) : "memory");
}
__device__ __forceinline__ void st2_sys(void* p, u32 v) {
    asm volatile("global_store_short %0, %1, off sc0 sc1\n\ts_nop 1" :: "v"(p), "v"(v) : "memory");
}
__device__ __forceinline__ void st16_sys(void* p, u32x4 v) {
    asm volatile("global_store_dwordx4 %0, %1, off sc0 sc1\n\ts_nop 1" :: "v"(p), "v"(v) : "memory");
}
__device__ __forceinline__ void st8_sys(void* p, u32x2 v) {
    asm volatile("global_store_dwordx2 %0, %1, off sc0 sc1\n\ts_nop 1" :: "v"(p), "v"(v) : "memory");
}
__device__ __forceinline__ u32 wave_scan_incl(u32 v) {
    v += __builtin_amdgcn_update_dpp(0, v, 0x111, 0xf, 0xf, true);
    v += __builtin_amdgcn_update_dpp(0, v, 0x112, 0xf, 0xf, true);
    v += __builtin_amdgcn_update_dpp(0, v, 0x114, 0xf, 0xf, true);
    v += __builtin_amdgcn_update_dpp(0, v, 0x118, 0xf, 0xf, true);
    v += __builtin_amdgcn_update_dpp(0, v, 0x142, 0xa, 0xf, false);
    v += __builtin_amdgcn_update_dpp(0, v, 0x143, 0xc, 0xf, false);
    return v;
}

__device__ __forceinline__ int src_of_rem(int R, int rank, int cb) { return (rank + 1 + (cb + R) % 7) & 7; }

constexpr int LAG = CH == 2 ? 3 : 2, LAST = LAG * (CH - 1), PC = 6 / CH;
static_assert(WIN >= 1 && WIN <= WIN2 && WIN2 <= 21 && NSW > LAST + 2, "window");
static_assert(!LAZY || FENCE_MODE == 3, "one acquire behind all flags");
constexpr int w_cum(int d) { int c = 0; for (int t = 0; t <= d; ++t) c += (CH == 3 && t < LAG ? 2 : 0) + PC + 1; return c; }
constexpr int w_behind(int b, int q) { return b >= NREM - LAST ? q - w_cum(b - (NREM - LAST)) : q + 7 * (NREM - LAST - 1 - b); }
constexpr int w_landed(int q, int win) { int u = 0; while (u < NREM && w_behind(u, q) >= win) ++u; return u; }
template <int N, int I = 0, class F> __device__ __forceinline__ void static_for(F&& f) {
    if constexpr (I < N) { f(std::integral_constant<int, I>{}); static_for<N, I + 1>(f); }
}

#define STAMP(i) do {} while (0)
#define LDS_AT(T, a) (*(volatile LAS T*)(size_t)(a))

__global__ void __launch_bounds__(NTHR, 2) gemm_xchg(Params p) {
    extern __shared__ __attribute__((aligned(16))) unsigned char lds_raw[];
    LAS unsigned char* L = (LAS unsigned char*)lds_raw;
    const u32 lbase = (u32)(size_t)L;
    volatile LAS u32* ctl = (volatile LAS u32*)(L + L_CTL);
    const int tid = threadIdx.x, lane = tid & 63;
    const int wave = rfl(tid >> 6);
    const int bid = blockIdx.x, xc = bid & 7, jj = bid >> 3;
    const int rank = p.rank;
    const int rb = (jj & 3) * 2, cb = xc * 8 + (jj >> 2);
    GAS u32* tmo = (GAS u32*)(p.ws + TMO_OFF) + 1;
    STAMP(0);

    GAS u32* cntp = (GAS u32*)(p.ws + CNT_OFF) + bid * 8 + wave;
    u32 call = __hip_atomic_load(cntp, __ATOMIC_RELAXED, __HIP_MEMORY_SCOPE_AGENT);
    char* fpeer = (char*)p.ws_all[lane & 7];
    u32 par;
    auto expired = [&](u64 t0) {
        if (rfl(__hip_atomic_load(tmo, __ATOMIC_RELAXED, __HIP_MEMORY_SCOPE_AGENT)) != call && rt() - t0 <= DEADLINE) return false;
        __hip_atomic_store(tmo, call, __ATOMIC_RELAXED, __HIP_MEMORY_SCOPE_AGENT);
        return true;
    };
    auto min_free = [&]() { const u32x4 fr = LDS_AT(u32x4, lbase + L_CTL + C_FREE * 4); return (int)rfl(min(min(fr.x, fr.y), min(fr.z, fr.w))); };

    if (wave < 4) {
        const int cbe = (jj & 1) + 2 * (xc + 8 * ((jj >> 1) & 3)), kse = jj >> 3;
        const u32 amb = lbase + L_MASK + wave * 2048;
        {
            const u64 src = (u64)p.w + (u64)(kse * 128) * LDW + cbe * 64 + wave * 16;
            dma16<0, 0, 4>(src, (u32)lane * LDW, amb);
            dma16<0, 0, 4>(src, (u32)(lane + 64) * LDW, amb + 1024);
        }
        if (wave == 0 && lane < 16) ctl[lane] = 0;
        float scale = p.sx[0] * p.sw[0];
        asm volatile("" : "+v"(scale), "+v"(fpeer), "+v"(call));
        call = rfl(call) + 1;
        par = call & 1;
        WAIT_VM(0);
        u32 x[8];
        {
            u32x2 t0, t1, t2, t3;
            const u32 a = amb + (u32)(((lane >> 4) * 16 + ((lane & 15) >> 1)) * 16 + (lane & 1) * 8);
            asm volatile("ds_read_b64_tr_b8 %0, %4\n\tds_read_b64_tr_b8 %1, %4 offset:128\n\t"
                         "ds_read_b64_tr_b8 %2, %4 offset:1024\n\tds_read_b64_tr_b8 %3, %4 offset:1152\n\ts_waitcnt lgkmcnt(0)"
                         : "=&v"(t0), "=&v"(t1), "=&v"(t2), "=&v"(t3) : "v"(a) : "memory");
            x[0] = t0.x; x[1] = t0.y; x[2] = t1.x; x[3] = t1.y; x[4] = t2.x; x[5] = t2.y; x[6] = t3.x; x[7] = t3.y;
        }
        STAMP(1);
        u32 sum = 0;
#pragma unroll
        for (int i = 0; i < 8; ++i) sum = __builtin_amdgcn_sad_u8(x[i] & 0x7F7F7F7Fu, 0u, sum);
        sum = rdl(wave_scan_incl(sum), 63);
        int lo = ((int)sum - 41165) >> 11;
        lo = lo < 0 ? 0 : (lo > 96 ? 96 : lo);
        const u32 LOe = (u32)lo * 0x01010101u, CLe = (u32)(128 - lo) * 0x01010101u;
        u32 code[8], mk[8];
#pragma unroll
        for (int i = 0; i < 8; ++i) {
            const u32 d = ((x[i] & 0x7F7F7F7Fu) + CLe) & 0x1F1F1F1Fu;
            code[i] = (x[i] & 0x80808080u) | d;
            mk[i] = x[i] ^ (d + LOe);
        }
        u32x4 P0; u32x2 P1;
        P0.x = code[0] | ((code[6] & 0x03030303u) << 5);
        P0.y = code[1] | ((code[6] & 0x0C0C0C0Cu) << 3);
        P0.z = code[2] | ((code[6] & 0x10101010u) << 1) | ((code[6] & 0x80808080u) >> 1);
        P0.w = code[3] | ((code[7] & 0x03030303u) << 5);
        P1.x = code[4] | ((code[7] & 0x0C0C0C0Cu) << 3);
        P1.y = code[5] | ((code[7] & 0x10101010u) << 1) | ((code[7] & 0x80808080u) >> 1);
        char* blk = p.ws + WST_OFF + par * WST_PAR + (size_t)(cbe * 4 + kse) * BLK_G;
        st16_sys(blk + wave * 1536 + lane * 16, P0);
        st8_sys(blk + wave * 1536 + 1024 + lane * 8, P1);
        u32 bm5 = 0, bm6 = 0;
#pragma unroll
        for (int i = 0; i < 8; ++i) {
            bm5 |= (__builtin_amdgcn_udot4(mk[i] & 0x20202020u, 0x08040201u, 0u, false) >> 5) << (4 * i);
            bm6 |= (__builtin_amdgcn_udot4(mk[i] & 0x40404040u, 0x08040201u, 0u, false) >> 6) << (4 * i);
        }
        const u32 cnt = __builtin_popcount(bm5 | bm6);
        const u32 incl = wave_scan_incl(cnt);
        const u32 total = rdl(incl, 63);
        const bool dense = total > 31 + 256;
        auto entry = [&](u32 pos) { return (pos >> 4) * 1024 + lane * 16 + (pos & 15) + (((bm5 >> pos) & 1u) << 11) + (((bm6 >> pos) & 1u) << 12); };
        u32 ev = 0;
        if (!dense) {
            u32 bm = bm5 | bm6, idx = incl - cnt + 1;
            while (__any(bm != 0 && idx < 32)) {
                const u32 pos = __builtin_ctz(bm | 0x80000000u);
                ev |= (u32)__builtin_amdgcn_ds_permute((int)((bm != 0 && idx < 32 ? idx : 63u) * 4), (int)entry(pos));
                ++idx; bm &= bm - 1;
            }
        }
        if (lane == 0) ev = (u32)lo | (total > 31 ? 128u : 0u);
        if (lane < 32) st2_sys(blk + MAIN_B + wave * 64 + lane * 2, ev);
        __builtin_amdgcn_s_barrier();
        if (__builtin_expect(total > 31, 0)) {
            const u32 bytes = dense ? 512u : (((total - 31) * 2 + 7) & ~7u);
            u32 start = 0;
            if (lane == 0) start = __hip_atomic_fetch_add((LAS u32*)(L + L_CTL) + C_ALLOC, bytes, __ATOMIC_RELAXED, __HIP_MEMORY_SCOPE_WORKGROUP);
            start = rfl(start) + 16;
            if (lane == 0) st4_sys(blk + OVF_B + wave * 4, start | (dense ? 1u << 12 : (total - 31) << 16));
            if (dense) {
                u32x2 D;
                D.x = ((mk[0] & 0x60606060u) >> 5) | ((mk[1] & 0x60606060u) >> 3) | ((mk[2] & 0x60606060u) >> 1) | ((mk[3] & 0x60606060u) << 1);
                D.y = ((mk[4] & 0x60606060u) >> 5) | ((mk[5] & 0x60606060u) >> 3) | ((mk[6] & 0x60606060u) >> 1) | ((mk[7] & 0x60606060u) << 1);
                st8_sys(blk + OVF_B + start + lane * 8, D);
            } else {
                u32 bm = bm5 | bm6, idx = incl - cnt;
                while (bm) {
                    if (idx >= 31) st2_sys(blk + OVF_B + start + (idx - 31) * 2, entry(__builtin_ctz(bm)));
                    ++idx; bm &= bm - 1;
                }
            }
        }
        LDS_AT(u32x4, amb + lane * 16) = u32x4{0, 0, 0, 0};
        LDS_AT(u32x4, amb + 1024 + lane * 16) = u32x4{0, 0, 0, 0};
        WAIT_VM(0);
        u32 a2 = 0;
        if (lane == 0) a2 = __hip_atomic_fetch_add((LAS u32*)(L + L_CTL) + C_ARR, 1u, __ATOMIC_RELAXED, __HIP_MEMORY_SCOPE_WORKGROUP);
        if (rfl(a2) == 3) {
            const u32 al = ctl[C_ALLOC];
            const u32 n16 = 16 + (al ? (al + 31) >> 4 : 0);
            if (lane < NR) {
                GAS u64* f = (GAS u64*)(fpeer + WFL_OFF + par * WFL_PAR) + ((cbe * 8 + rank) * 4 + kse);
                __hip_atomic_store(f, ((u64)call << 32) | n16, __ATOMIC_RELAXED, __HIP_MEMORY_SCOPE_SYSTEM);
            }
            ctl[C_TPUB] = (u32)rt() | 1u;
        }
        STAMP(2);

        f32x4 acc[8];
#pragma unroll
        for (int f = 0; f < 8; ++f) acc[f] = f32x4{0, 0, 0, 0};
        int avail = 0;
        u32 xo = 0, wo = 0;
        const u32 aq0 = lbase + L_W + wave * 1536 + lane * 16, aq1 = lbase + L_W + wave * 1536 + 1024 + lane * 8;
        const u32 aen = lbase + L_W + MAIN_B + wave * 64 + (lane & 31) * 2, axx = lbase + L_X + lane * 16;
        const u32 scr = lbase + L_CTL + 64 + lane;
#pragma nounroll
        for (int i = 0; i < NSTEP; ++i) {
            if (i >= avail) {
                u64 t0 = 0;
                for (u32 it = 1;; ++it) {
                    const u32x4 fl = LDS_AT(u32x4, lbase + L_CTL);
                    avail = (int)rfl(min(fl.y < 4 ? fl.y : fl.x + 4, fl.z));
                    if (i < avail) break;
                    if (__builtin_expect((it & 1023) == 0, 0)) {
                        const u64 t = rt();
                        if (t0 == 0) t0 = t;
                        if (t - t0 > 2 * DEADLINE) { ctl[C_FAIL] = 1; avail = NSTEP; break; }
                    }
                    __builtin_amdgcn_s_sleep(1);
                }
                asm volatile("" ::: "memory");
            }
            const u32 ax = axx + xo;
            const u32 e0 = LDS_AT(unsigned short, aen + wo);
            const u32x4 q0 = LDS_AT(u32x4, aq0 + wo);
            const u32x2 q1 = LDS_AT(u32x2, aq1 + wo);
            u32x4 xa[16];
#pragma unroll
            for (int f = 0; f < 6; ++f) xa[f] = LDS_AT(u32x4, ax + f * 1024);
            __builtin_amdgcn_sched_barrier(0);
            const u32 hdr = rfl(e0);
            const u32 ea = e0 >> 11 ? amb + (e0 & 2047u) : scr;
            LDS_AT(unsigned char, ea) = (unsigned char)(e0 >> 11 << 5);
            if (__builtin_expect(hdr & 128u, 0)) {
                const u32 ov = lbase + L_W + OVF_B + wo;
                const u32 h2 = rfl(LDS_AT(u32, ov + wave * 4)), st = h2 & 0xFF8u, n = (h2 >> 16) & 0x1FFu;
                if (h2 & (1u << 12)) {
                    const u32x2 D = LDS_AT(u32x2, ov + st + lane * 8);
                    LDS_AT(u32x4, amb + lane * 16) = u32x4{(D.x << 5) & 0x60606060u, (D.x << 3) & 0x60606060u, (D.x << 1) & 0x60606060u, (D.x >> 1) & 0x60606060u};
                    LDS_AT(u32x4, amb + 1024 + lane * 16) = u32x4{(D.y << 5) & 0x60606060u, (D.y << 3) & 0x60606060u, (D.y << 1) & 0x60606060u, (D.y >> 1) & 0x60606060u};
                } else {
                    for (u32 k = 0; k < n; k += 64) {
                        const u32 e = LDS_AT(unsigned short, ov + st + (k + lane) * 2);
                        if (k + lane < n) LDS_AT(unsigned char, amb + (e & 2047u)) = (unsigned char)(e >> 11 << 5);
                    }
                }
                __builtin_amdgcn_s_waitcnt(0xC07F);
            }
            const u32x4 m0 = LDS_AT(u32x4, amb + lane * 16);
            const u32x4 m1 = LDS_AT(u32x4, amb + 1024 + lane * 16);
            LDS_AT(unsigned char, ea) = 0;
            if (__builtin_expect(hdr & 128u, 0)) { LDS_AT(u32x4, amb + lane * 16) = u32x4{0, 0, 0, 0}; LDS_AT(u32x4, amb + 1024 + lane * 16) = u32x4{0, 0, 0, 0}; }
#pragma unroll
            for (int f = 6; f < 16; ++f) xa[f] = LDS_AT(u32x4, ax + f * 1024);
            __builtin_amdgcn_sched_barrier(0);
            const u32 LO = (hdr & 0x7Fu) * 0x01010101u;
            const u32 e6 = ((q0.x >> 5) & 0x03030303u) | ((q0.y >> 3) & 0x0C0C0C0Cu) | ((q0.z >> 1) & 0x10101010u) | ((q0.z << 1) & 0x80808080u);
            const u32 e7 = ((q0.w >> 5) & 0x03030303u) | ((q1.x >> 3) & 0x0C0C0C0Cu) | ((q1.y >> 1) & 0x10101010u) | ((q1.y << 1) & 0x80808080u);
            i32x8 wf;
            wf[0] = (q0.x & 0x9F9F9F9Fu) + LO; wf[1] = (q0.y & 0x9F9F9F9Fu) + LO; wf[2] = (q0.z & 0x9F9F9F9Fu) + LO; wf[3] = (q0.w & 0x9F9F9F9Fu) + LO;
            wf[4] = (q1.x & 0x9F9F9F9Fu) + LO; wf[5] = (q1.y & 0x9F9F9F9Fu) + LO; wf[6] = e6 + LO; wf[7] = e7 + LO;
            asm volatile("" : "+v"(wf));
            wf[0] ^= m0.x; wf[1] ^= m0.y; wf[2] ^= m0.z; wf[3] ^= m0.w; wf[4] ^= m1.x; wf[5] ^= m1.y; wf[6] ^= m1.z; wf[7] ^= m1.w;
            __builtin_amdgcn_sched_barrier(0);
#pragma unroll
            for (int f = 0; f < 8; ++f) {
                const u32x4 a = xa[2 * f], b = xa[2 * f + 1];
                i32x8 xf;
                xf[0] = a.x; xf[1] = a.y; xf[2] = a.z; xf[3] = a.w; xf[4] = b.x; xf[5] = b.y; xf[6] = b.z; xf[7] = b.w;
                if (f == 3) {
                    asm volatile("" :: "v"(xa[15]) : "memory");
                    ctl[C_FREE + wave] = i + 1;
                    __builtin_amdgcn_sched_barrier(0);
                }
                acc[f] = __builtin_amdgcn_mfma_scale_f32_16x16x128_f8f6f4(wf, xf, acc[f], 1  , 0  , 0, 0, 0, 0);
            }
            xo = xo + XS_B == NSX * XS_B ? 0 : xo + XS_B;
            wo = wo + BLK_B == NSW * BLK_B ? 0 : wo + BLK_B;
        }
        STAMP(5);
        float* o = p.out + (size_t)(rb * 64 + (lane & 15)) * LDO + cb * 64 + wave * 16 + (lane >> 4) * 4;
        const float sc = ctl[C_FAIL] != 0 ? __builtin_nanf("") : scale;
#pragma unroll
        for (int f = 0; f < 8; ++f)
        {
            const f32x4 y = __builtin_elementwise_maximum(acc[f] * sc, f32x4{0, 0, 0, 0});
            if (EPI == 1) st16_sys(o + (size_t)f * 16 * LDO, __builtin_bit_cast(u32x4, y));
            else if (EPI == 2) __builtin_nontemporal_store(y, (f32x4*)(o + (size_t)f * 16 * LDO));
            else *(f32x4*)(o + (size_t)f * 16 * LDO) = y;
        }
        STAMP(6);
    } else if (wave == 4) {
        __builtin_amdgcn_s_barrier();
        call = rfl(call) + 1;
        par = call & 1;
        const int R = lane < NREM ? lane : 0, src = src_of_rem(R, rank, cb), ks = R / 7;
        const u64 baddr = (((u64)(u32)__shfl((int)((u64)fpeer >> 32), src) << 32) | (u32)__shfl((int)(u32)(u64)fpeer, src)) + WST_OFF + par * WST_PAR + (u64)(cb * 4 + ks) * BLK_G;
        const u32 blo = (u32)baddr, bhi = (u32)(baddr >> 32);
        GAS u64* fl = (GAS u64*)(p.ws + WFL_OFF + par * WFL_PAR) + ((cb * 8 + src) * 4 + ks);
        const u32 voff = lane * 16;
        u32 n16v = 16;
        bool dead = false;
        if (FENCE_MODE == 3) {
            const u32 xcc = __builtin_amdgcn_s_getreg((3 << 11) | 20) & 15;
            GAS u64* inv = (GAS u64*)(p.ws + INV_OFF + xcc * 256);
            u32 prev = 0;
            if (lane == 0) prev = __hip_atomic_fetch_max((GAS u32*)(inv + 16), call, __ATOMIC_RELAXED, __HIP_MEMORY_SCOPE_AGENT);
            if (rfl(prev) < call) {
                __builtin_amdgcn_fence(__ATOMIC_ACQUIRE, "");
                WAIT_VM(0);
                if (lane == 0) __hip_atomic_store(inv, (u64)call << 32, __ATOMIC_RELAXED, __HIP_MEMORY_SCOPE_AGENT);
            }
            if (lane == 63) fl = inv;
            STAMP(3);
        }
        if (lane >= 28 && lane < 56) {
            const int q = lane - 28, src2 = (rank + 1 + q % 7) & 7, j = q / 7;
            fl = (GAS u64*)(p.ws + XFL_OFF + par * XFL_PAR) + (NPUSH <= 16 ? ((rb + (j >> 1)) * 8 + src2) * 4 + (j & 1) : (rb * 8 + src2) * 4 + j);
        }
        if (lane == 62) fl = (GAS u64*)(p.ws + TMO_OFF);
        u64 ready = 0;
        auto wait_flags = [&](u64 need, int xth) {
            const u64 t0 = rt();
            for (u32 it = 1;; ++it) {
                const u64 v = __hip_atomic_load(fl, __ATOMIC_RELAXED, __HIP_MEMORY_SCOPE_SYSTEM);
                n16v = (u32)v;
                ready = __ballot((u32)(v >> 32) >= call);
                if ((ready >> 62) & 1) { dead = rdl((u32)(v >> 32), 62) == call; if (dead) break; }
                if ((ready & need) == need && (__builtin_popcount((u32)(ready >> 28) & 0xFFFFFFFu) >= xth || rt() - t0 > 2000)) break;
                if ((it & 15) == 0 && expired(t0)) { dead = true; break; }
                __builtin_amdgcn_s_sleep(1);
            }
            n16v = n16v < 16 ? 16 : (n16v > MAX16 ? MAX16 : n16v);
        };
        wait_flags((FENCE_MODE == 3 ? 1ull << 63 : 0) | (LAZY ? 1 : 0xFFFFFFFull), FENCE_MODE == 0 ? 0 : XTH);
        STAMP(1);
        if (FENCE_MODE == 0) { __builtin_amdgcn_fence(__ATOMIC_ACQUIRE, ""); WAIT_VM(0); wait_flags(0, XTH); }
        ctl[C_GO] = 1;
        STAMP(4);
        constexpr int RP = FENCE_MODE == 3 ? 2 : 0;
        int pub = 0, freed = 0;
        u32 lo = lbase + L_W + 4 * BLK_B;
        auto publish = [&](int upto) { if (upto > pub) { pub = upto; ctl[C_FULLW] = upto; } };
        u64 g[LAST + 1]; u32 l[LAST + 1];
#pragma unroll
        for (int j = 0; j <= LAST; ++j) { g[j] = 0; l[j] = 0; }
#define W_LOAD(OFF, J, K, NOP) do { dma16<OFF, RP, NOP>(g[J], voff, l[J]); WAIT_VM(W); after(std::integral_constant<int, K>{}); } while (0)
        auto rest_of_slot = [&](int n, bool mid, auto win, auto&& after) {
            constexpr int W = decltype(win)::value;
            if constexpr (CH == 3) { if (mid) { W_LOAD(2048, LAG, 3, 0); W_LOAD(3072, LAG, 4, 0); } }
            if (n >= LAST) {
                g[LAST] += PC * (CH - 1) * 1024; l[LAST] += PC * (CH - 1) * 1024;
                if constexpr (CH == 2) { W_LOAD(0, LAST, 4, 0); W_LOAD(1024, LAST, 5, 0); W_LOAD(2048, LAST, 6, 0); }
                else { W_LOAD(0, LAST, 5, 0); W_LOAD(1024, LAST, 6, 0); }
                const u32 n16 = rdl(n16v, n - LAST);
                if (__builtin_expect(n16 > 64, 0))
                    for (u32 i = 64; i < n16; i += 64) { if (lane + i < n16) dma16<PC * 1024, RP>(g[LAST] + i * 16, voff, l[LAST] + i * 16); WAIT_VM(W); }
                if ((u32)lane < n16) dma16<PC * 1024, RP>(g[LAST], voff, l[LAST]);
                WAIT_VM(W); after(std::integral_constant<int, 7>{});
            }
#pragma unroll
            for (int j = LAST; j > 0; --j) { g[j] = g[j - 1]; l[j] = l[j - 1]; }
        };
        auto slot = [&](int n, auto win) {
            constexpr int W = decltype(win)::value;
            constexpr int PK = (W - 1) % 7 + 1, PL = LAST + 1 + (W - PK) / 7;
            auto after = [&](auto k) { if constexpr (decltype(k)::value == PK) publish(n - PL + 1); };
            if (LAZY && !((ready >> n) & 1)) { WAIT_VM(0); publish(n - LAST); wait_flags(1ull << n, 0); if (dead) return; }
            if (freed <= n + 4 - NSW) {
                const int need = n + 4 - NSW;
                const u64 t0 = rt();
                bool drained = false;
                for (;;) {
                    freed = min_free();
                    if (freed > need) break;
                    if (!drained) { WAIT_VM(0); publish(n - LAST); drained = true; continue; }
                    if (rt() - t0 > 2 * DEADLINE) { dead = true; break; }
                    __builtin_amdgcn_s_sleep(1);
                }
                if (dead) return;
            }
            g[0] = ((u64)rdl(bhi, n) << 32) | rdl(blo, n);
            l[0] = lo;
            lo = lo + BLK_B == lbase + L_W + NSW * BLK_B ? lbase + L_W : lo + BLK_B;
            W_LOAD(0, 0, 1, 4); W_LOAD(1024, 0, 2, 0);
            if constexpr (CH == 2) W_LOAD(2048, 0, 3, 0);
            rest_of_slot(n, n >= LAG, win, after);
        };
        for (int n = 0; n < NREM - NW2 && !dead; ++n) slot(n, std::integral_constant<int, WIN>{});
        for (int n = NREM - NW2; n < NREM && !dead; ++n) slot(n, std::integral_constant<int, WIN2>{});
        if (dead) { ctl[C_FAIL] = 1; WAIT_VM(0); ctl[C_FULLW] = NREM; }
        else {
            static_for<LAST>([&](auto dd) {
                constexpr int d = decltype(dd)::value;
                rest_of_slot(NREM + d, d < LAG, std::integral_constant<int, WIN2>{},
                             [&](auto k) { publish(std::integral_constant<int, w_landed(w_cum(d) - (7 - decltype(k)::value), WIN2)>::value); });
            });
            STAMP(5);
            static_for<LAST + 3>([&](auto dd) {
                constexpr int upto = NREM - LAST - 2 + decltype(dd)::value, behind = w_behind(upto - 1, w_cum(LAST - 1));
                if constexpr (behind < WIN2) { WAIT_VM(behind); publish(upto); }
            });
        }
        STAMP(6);
    } else if (wave == 5) {
        __builtin_amdgcn_s_barrier();
        call = rfl(call) + 1;
        par = call & 1;
        const int R = max((lane & 31) - 4, 0), src = src_of_rem(R, rank, cb), ks = R / 7;
        const u32 toff = (u32)((src * 4 + ks) * 8 + rb) * XT_B;
        const u64 xbase = (u64)p.ws + XRV_OFF + par * XRV_PAR;
        GAS u64* fl = (GAS u64*)(p.ws + XFL_OFF + par * XFL_PAR) + (((rb + (lane >> 5)) * 8 + src) * 4 + ks);
        constexpr u32 OWN = 0xFu;
        const u32 voff = lane * 16;
        const u32 xoff = (u32)(lane & 15) * LDX + (lane >> 4) * 16;
        u32 ready = OWN;
        u32 it = 0, la = lbase + L_X;
        bool dead = false;
        const u64 t0 = rt();
        int pub = 0, freed = 0;
        for (int s = 0; s < NSTEP && !dead; ++s) {
            if (!((ready >> s) & 1)) {
                WAIT_VM(0);
                if (pub < s) { pub = s; ctl[C_FULLX] = s; }
                for (;;) {
                    const u64 v = __hip_atomic_load(fl, __ATOMIC_RELAXED, __HIP_MEMORY_SCOPE_SYSTEM);
                    const u64 ok = __ballot((u32)(v >> 32) >= call);
                    ready = ((u32)ok & (u32)(ok >> 32)) | OWN;
                    if (ready == ~0u) { dead = rfl(__hip_atomic_load(tmo, __ATOMIC_RELAXED, __HIP_MEMORY_SCOPE_AGENT)) == call; break; }
                    if ((ready >> s) & 1) break;
                    if ((++it & 15) == 0 && expired(t0)) { dead = true; break; }
                    __builtin_amdgcn_s_sleep(2);
                }
            }
            while (!dead && freed <= s - NSX) {
                freed = min_free();
                if (freed > s - NSX) break;
                if ((++it & 1023) == 0 && rt() - t0 > 2 * DEADLINE) dead = true;
                __builtin_amdgcn_s_sleep(1);
            }
            if (dead) break;
            if (s < 4) {
                const u64 b = (u64)p.x + (u64)(rank * 512 + rb * 64) * LDX + s * 128;
#pragma unroll
                for (int i = 0; i < 16; ++i) dma16<0, 0>(b + (i >> 1) * 16 * LDX + (i & 1) * 64, xoff, la + i * 1024);
            } else {
                const u64 b = xbase + rdl(toff, s);
#pragma unroll
                for (int i = 0; i < 4; ++i) {
                    dma16<0, 1, 4>(b + i * 4096, voff, la + i * 4096); dma16<1024, 1>(b + i * 4096, voff, la + i * 4096);
                    dma16<2048, 1>(b + i * 4096, voff, la + i * 4096); dma16<3072, 1>(b + i * 4096, voff, la + i * 4096);
                }
            }
            la = la + XS_B == lbase + L_X + NSX * XS_B ? lbase + L_X : la + XS_B;
            WAIT_VM(16 * (XD - 1));
            if (pub < s - XD + 2) { pub = s - XD + 2; ctl[C_FULLX] = pub; }
        }
        if (dead) { ctl[C_FAIL] = 1; WAIT_VM(0); }
        else {
            WAIT_VM(32); ctl[C_FULLX] = max(pub, NSTEP - 2);
            WAIT_VM(16); ctl[C_FULLX] = max(pub, NSTEP - 1);
            WAIT_VM(0);
        }
        ctl[C_FULLX] = NSTEP;
        STAMP(6);
    } else if (wave == 7) {
        __builtin_amdgcn_s_barrier();
        call = rfl(call) + 1;
        par = call & 1;
        GAS u64* fl = (GAS u64*)(p.ws + WFL_OFF + par * WFL_PAR) + ((cb * 8 + rank) * 4 + (lane & 3));
        const u64 t0 = rt();
        bool dead = false;
        u32 n16v;
        for (u32 it = 1;; ++it) {
            const u64 v = __hip_atomic_load(fl, __ATOMIC_RELAXED, __HIP_MEMORY_SCOPE_SYSTEM);
            n16v = (u32)v;
            if (__ballot((u32)(v >> 32) >= call) == ~0ull) break;
            if ((it & 15) == 0 && expired(t0)) { dead = true; break; }
            __builtin_amdgcn_s_sleep(4);
        }
        if (rfl(__hip_atomic_load(tmo, __ATOMIC_RELAXED, __HIP_MEMORY_SCOPE_AGENT)) == call) dead = true;
        n16v = MAIN_B / 16 + (n16v < 16 ? 16 : (n16v > MAX16 ? MAX16 : n16v));
        for (int k = 0; k < 4 && !dead; ++k) {
            const u64 wb = (u64)p.ws + WST_OFF + par * WST_PAR + (u64)(cb * 4 + k) * BLK_G;
            const u32 n16 = rdl(n16v, k);
            for (u32 i = 0; i < n16; i += 64)
                if (lane + i < n16) dma16<0, 1, 4>(wb + i * 16, lane * 16, lbase + L_W + k * BLK_B + i * 16);
        }
        if (dead) ctl[C_FAIL] = 1;
        WAIT_VM(21); ctl[C_FULLO] = 1;
        WAIT_VM(14); ctl[C_FULLO] = 2;
        WAIT_VM(7); ctl[C_FULLO] = 3;
        WAIT_VM(0); ctl[C_FULLO] = 4;
    } else {
        const int id = bid / 7, d = (rank + 1 + bid % 7) & 7, s = (rank + 7 - bid % 7) & 7;
        const bool push = id < NPUSH;
        u32x4 v[8];
        if (id < 32) {
            const unsigned char* g = p.x + (size_t)(d * 512 + (id & 7) * 64 + (lane & 15)) * LDX + (id >> 3) * 128 + (lane >> 4) * 16;
#pragma unroll
            for (int i = 0; i < 8; ++i) v[i] = *(const GAS u32x4*)(g + (size_t)(i >> 1) * 16 * LDX + (i & 1) * 64);
        }
        call = rfl(call) + 1;
        par = call & 1;
        if (id < 32) {
            asm volatile("" : "+v"(v[0]), "+v"(v[1]), "+v"(v[2]), "+v"(v[3]), "+v"(v[4]), "+v"(v[5]), "+v"(v[6]), "+v"(v[7]), "+v"(fpeer));
            char* peer = (char*)(((u64)rdl((u32)((u64)fpeer >> 32), d) << 32) | rdl((u32)(u64)fpeer, d));
            char* dst = push ? peer + XRV_OFF + par * XRV_PAR + (size_t)(rank * 32 + id) * XT_B + lane * 16
                             : p.ws + XST_OFF + par * XST_PAR + (size_t)(d * 32 + id) * XT_B + lane * 16;
            bool open = false;
#pragma unroll
            for (int i = 0; i < 8; ++i) {
                st16_sys(dst + i * 1024, v[i]);
                if (i == 0) __builtin_amdgcn_s_barrier();
                if (push) {
                    if (!open) { const u32 tp = rfl(ctl[C_TPUB]); open = tp != 0 && (u32)rt() - tp >= PUSH_LAG; }
                    if (open) WAIT_VM(PUSH_W1 - 1); else WAIT_VM(PUSH_W0 - 1);
                }
            }
            STAMP(1);
            WAIT_VM(0);
            if (lane == 0) {
                GAS u64* f = push ? (GAS u64*)(peer + XFL_OFF + par * XFL_PAR) + (((id & 7) * 8 + rank) * 4 + (id >> 3))
                                  : (GAS u64*)(peer + XSF_OFF + par * XSF_PAR) + (rank * 32 + id);
                __hip_atomic_store(f, (u64)call << 32, __ATOMIC_RELAXED, __HIP_MEMORY_SCOPE_SYSTEM);
            }
            STAMP(2);
            if (!push) {
                GAS u64* f = (GAS u64*)(p.ws + XSF_OFF + par * XSF_PAR) + (s * 32 + id);
                const u64 t0 = rt();
                bool dead = false;
                for (u32 it = 1; (u32)(__hip_atomic_load(f, __ATOMIC_RELAXED, __HIP_MEMORY_SCOPE_SYSTEM) >> 32) < call; ++it) {
                    if ((it & 15) == 0 && expired(t0)) { dead = true; break; }
                    __builtin_amdgcn_s_sleep(4);
                }
                while (!dead && rfl(ctl[C_GO]) == 0 && rt() - t0 < 4000) __builtin_amdgcn_s_sleep(4);
                if (!dead) {
                    const char* from = (char*)(((u64)rdl((u32)((u64)fpeer >> 32), s) << 32) | rdl((u32)(u64)fpeer, s))
                                       + XST_OFF + par * XST_PAR + (size_t)(rank * 32 + id) * XT_B + lane * 16;
#pragma unroll
                    for (int i = 0; i < 8; i += PULL_W) {
                        if constexpr (PULL_W == 1)
                            asm volatile("global_load_dwordx4 %0, %1, off sc0 sc1\n\ts_waitcnt vmcnt(0)" : "=&v"(v[i]) : "v"(from + i * 1024) : "memory");
                        else
                            asm volatile("global_load_dwordx4 %0, %2, off sc0 sc1\n\tglobal_load_dwordx4 %1, %2, off offset:1024 sc0 sc1\n\ts_waitcnt vmcnt(0)"
                                         : "=&v"(v[i]), "=&v"(v[i + 1]) : "v"(from + i * 1024) : "memory");
                    }
                    char* to = p.ws + XRV_OFF + par * XRV_PAR + (size_t)(s * 32 + id) * XT_B + lane * 16;
#pragma unroll
                    for (int i = 0; i < 8; ++i) st16_sys(to + i * 1024, v[i]);
                    WAIT_VM(0);
                    if (lane == 0)
                        __hip_atomic_store((GAS u64*)(p.ws + XFL_OFF + par * XFL_PAR) + (((id & 7) * 8 + s) * 4 + (id >> 3)), (u64)call << 32,
                                           __ATOMIC_RELAXED, __HIP_MEMORY_SCOPE_SYSTEM);
                }
                STAMP(3);
            }
        } else {
            __builtin_amdgcn_s_barrier();
        }
    }
    if (lane == 0) __hip_atomic_store(cntp, call, __ATOMIC_RELAXED, __HIP_MEMORY_SCOPE_AGENT);
}

extern "C" void kernel_launch(int rank, int nranks, ncclComm_t comm,
                              hipStream_t stream,
                              void* const* d_in, const int* in_sizes, int n_in,
                              void* d_out, int out_size,
                              const int* in_dtypes, int out_dtype,
                              void* const* d_out_all,
                              void* const* d_ws_all, size_t ws_bytes) {
    (void)comm; (void)in_dtypes; (void)out_dtype; (void)d_out_all;
    if (nranks != NR || (unsigned)rank >= (unsigned)NR || n_in != 4 || ws_bytes < WS_NEED ||
        in_sizes[0] != 4096 * 512 || in_sizes[1] != 512 * 4096 || out_size != 512 * 4096) {
        fprintf(stderr, "unexpected problem shape\n");
        return;
    }
    HIP_CHECK(hipFuncSetAttribute((const void*)gemm_xchg, hipFuncAttributeMaxDynamicSharedMemorySize, LDS_BYTES));
    Params p;
    p.x = (const unsigned char*)d_in[0];
    p.w = (const unsigned char*)d_in[1];
    p.sx = (const float*)d_in[2];
    p.sw = (const float*)d_in[3];
    p.out = (float*)d_out;
    p.ws_all = d_ws_all;
    p.ws = (char*)d_ws_all[rank];
    p.rank = rank;
    p.pad = 0;
    hipLaunchKernelGGL(gemm_xchg, dim3(NWG), dim3(NTHR), LDS_BYTES, stream, p);
    HIP_CHECK(hipGetLastError());
}
```

```cpp
#include <hip/hip_runtime.h>
#include <rccl/rccl.h>
#include <cstdio>

#define HIP_CHECK(expr) do { hipError_t _e = (expr); if (_e != hipSuccess) { \
    fprintf(stderr, "HIP error %s at %s:%d\n", hipGetErrorString(_e), __FILE__, __LINE__); \
    return; } } while (0)

typedef unsigned u32;
typedef unsigned long long u64;
typedef unsigned u32x4 __attribute__((ext_vector_type(4)));
typedef unsigned u32x2 __attribute__((ext_vector_type(2)));
typedef int i32x8 __attribute__((ext_vector_type(8)));
typedef float f32x4 __attribute__((ext_vector_type(4)));
#define GAS __attribute__((address_space(1)))
#define LAS __attribute__((address_space(3)))

#define WIN 5
#define WIN2 5
#define NW2 2
#define LAZY 0
#define PRIO_W 3
#define PULL_W 1
#define NPUSH 32
#define XTH 10
#define PUSH_W0 2
#define PUSH_W1 2
#define PUSH_LAG 60
#define FENCE_MODE 3
#define XD 2
#define NSW 9
#define NSX 4

constexpr int NR = 8;
constexpr int NWG = 256, NTHR = 704;
constexpr int LDX = 512, LDW = 4096, LDO = 4096;
constexpr int MAIN_B = 5120, FR_B = 1280, ENT_B = 24, POOL_B = MAIN_B + 4 * ENT_B;
constexpr int MIN16 = 4 * ENT_B / 16, MAX16 = (4 * ENT_B + 4 * 768) / 16;
constexpr int BLK_B = 8320;
constexpr int BLK_G = 8448;
constexpr int XT_B = 8192;
constexpr int XS_B = 2 * XT_B;
constexpr int NSTEP = 32, NREM = 28;
static_assert(MAIN_B + MAX16 * 16 + 16 <= BLK_B && BLK_B <= BLK_G, "block");

constexpr size_t WST_OFF = 0, WST_PAR = 4u << 20;
constexpr size_t XRV_OFF = 8u << 20, XRV_PAR = 2u << 20;
constexpr size_t XST_OFF = 12u << 20, XST_PAR = 2u << 20;
constexpr size_t WFL_OFF = 16u << 20, WFL_PAR = 16u << 10;
constexpr size_t XFL_OFF = WFL_OFF + (32u << 10), XFL_PAR = 4u << 10;
constexpr size_t XSF_OFF = WFL_OFF + (40u << 10), XSF_PAR = 4u << 10;
constexpr size_t CNT_OFF = WFL_OFF + (64u << 10);
constexpr size_t TMO_OFF = WFL_OFF + (128u << 10);
constexpr size_t INV_OFF = TMO_OFF + 1024;
constexpr size_t DBG_OFF = WFL_OFF + (256u << 10);
constexpr size_t WS_NEED = 17u << 20;
static_assert((size_t)256 * BLK_G <= WST_PAR, "w staging");

constexpr int L_W = 0;
constexpr int L_X = L_W + NSW * BLK_B;
constexpr int L_MASK = L_X + NSX * XS_B;
constexpr int L_CTL = L_MASK + 16384;
constexpr int LDS_BYTES = L_CTL + 128;
static_assert(NSW == 9 && NSX == 4, "the output staging areas are chosen for these rings");
static_assert(LDS_BYTES <= 160 * 1024 && NSW >= 8 && XD <= NSX && XD <= 4, "LDS");
constexpr int C_FULLW = 0, C_FULLX = 1, C_FAIL = 2, C_ALLOC = 3, C_ARR = 4, C_TPUB = 5, C_GO = 6, C_FREE = 8  ;

constexpr u64 DEADLINE = 30000000ull;

struct Params {
    const unsigned char* x;
    const unsigned char* w;
    const float* sx;
    const float* sw;
    float* out;
    void* const* ws_all;
    char* ws;
    int rank;
    int pad;
};

__device__ __forceinline__ u64 rt() { return __builtin_amdgcn_s_memrealtime(); }
__device__ __forceinline__ u32 rfl(u32 v) { return (u32)__builtin_amdgcn_readfirstlane(v); }
__device__ __forceinline__ u32 rdl(u32 v, int l) { return (u32)__builtin_amdgcn_readlane(v, l); }
#define WAIT_VM(n) asm volatile("s_waitcnt vmcnt(%0)" :: "i"(n) : "memory")
#define WG_BARRIER(call) do { u32 c_; asm volatile("s_barrier" ::: "memory"); \
    asm volatile("v_readfirstlane_b32 %0, %1\n\ts_nop 4" : "=s"(c_) : "v"(call) : "memory"); call = c_; } while (0)
#define WAIT_LGKM0() asm volatile("s_waitcnt lgkmcnt(0)" ::: "memory")

template <int OFF, int POL, int NOP = 0>
__device__ __forceinline__ void dma16(u64 base, u32 voff, u32 ldsaddr) {
    if constexpr (POL == 1)
        asm volatile("s_mov_b32 m0, %2\n\ts_nop %4\n\tglobal_load_lds_dwordx4 %0, %1 offset:%3 sc0 sc1"
                     :: "v"(voff), "s"(base), "s"(ldsaddr), "i"(OFF), "i"(NOP) : "memory");
    else if constexpr (POL == 2)
        asm volatile("s_mov_b32 m0, %2\n\ts_nop %4\n\tglobal_load_lds_dwordx4 %0, %1 offset:%3 nt"
                     :: "v"(voff), "s"(base), "s"(ldsaddr), "i"(OFF), "i"(NOP) : "memory");
    else
        asm volatile("s_mov_b32 m0, %2\n\ts_nop %4\n\tglobal_load_lds_dwordx4 %0, %1 offset:%3"
                     :: "v"(voff), "s"(base), "s"(ldsaddr), "i"(OFF), "i"(NOP) : "memory");
}
__device__ __forceinline__ void st4_sys(void* p, u32 v) {
    asm volatile("global_store_dword %0, %1, off sc0 sc1\n\ts_nop 1" :: "v"(p), "v"(v) : "memory");
}
__device__ __forceinline__ void st2_sys(void* p, u32 v) {
    asm volatile("global_store_short %0, %1, off sc0 sc1\n\ts_nop 1" :: "v"(p), "v"(v) : "memory");
}
__device__ __forceinline__ void st16_sys(void* p, u32x4 v) {
    asm volatile("global_store_dwordx4 %0, %1, off sc0 sc1\n\ts_nop 1" :: "v"(p), "v"(v) : "memory");
}
__device__ __forceinline__ void st8_sys(void* p, u32x2 v) {
    asm volatile("global_store_dwordx2 %0, %1, off sc0 sc1\n\ts_nop 1" :: "v"(p), "v"(v) : "memory");
}
__device__ __forceinline__ u32 wave_scan_incl(u32 v) {
    v += __builtin_amdgcn_update_dpp(0, v, 0x111, 0xf, 0xf, true);
    v += __builtin_amdgcn_update_dpp(0, v, 0x112, 0xf, 0xf, true);
    v += __builtin_amdgcn_update_dpp(0, v, 0x114, 0xf, 0xf, true);
    v += __builtin_amdgcn_update_dpp(0, v, 0x118, 0xf, 0xf, true);
    v += __builtin_amdgcn_update_dpp(0, v, 0x142, 0xa, 0xf, false);
    v += __builtin_amdgcn_update_dpp(0, v, 0x143, 0xc, 0xf, false);
    return v;
}

__device__ __forceinline__ int src_of_rem(int R, int rank, int cb) { return (rank + 1 + (cb + R) % 7) & 7; }

constexpr int LAG = 2, LAST = 2 * LAG, SL = 6, NB = NREM - 1;
static_assert(WIN >= 1 && WIN <= WIN2 && WIN2 <= 18 && NSW > LAST + 2, "window");
static_assert(!LAZY || FENCE_MODE == 3, "one acquire behind all flags");
constexpr int w_cum(int d) { int c = 0; for (int t = 0; t <= d; ++t) c += t == LAST ? SL : (t < LAG ? 4 : 2); return c; }
constexpr int w_behind(int b, int q) { return b >= NB - LAST ? q - w_cum(b - (NB - LAST)) : q + SL * (NB - LAST - 1 - b); }
constexpr int w_landed(int q, int win) { int u = 0; while (u < NREM && w_behind(u, q) >= win) ++u; return u; }
template <int N, int I = 0, class F> __device__ __forceinline__ void static_for(F&& f) {
    if constexpr (I < N) { f(std::integral_constant<int, I>{}); static_for<N, I + 1>(f); }
}

#define STAMP(i) do {} while (0)
#define LDS_AT(T, a) (*(volatile LAS T*)(size_t)(a))

__global__ void __launch_bounds__(NTHR, 3) gemm_xchg(Params p) {
    extern __shared__ __attribute__((aligned(16))) unsigned char lds_raw[];
    LAS unsigned char* L = (LAS unsigned char*)lds_raw;
    const u32 lbase = (u32)(size_t)L;
    volatile LAS u32* ctl = (volatile LAS u32*)(L + L_CTL);
    const int tid = threadIdx.x, lane = tid & 63;
    const int wave = rfl(tid >> 6);
    const int bid = blockIdx.x, xc = bid & 7, jj = bid >> 3;
    const int rank = p.rank;
    const int rb = (jj & 3) * 2, cb = xc * 8 + (jj >> 2);
    GAS u32* tmo = (GAS u32*)(p.ws + TMO_OFF) + 1;
    STAMP(0);

    GAS u32* cntp = (GAS u32*)(p.ws + CNT_OFF) + bid * 16 + wave;
    u32 call = __hip_atomic_load(cntp, __ATOMIC_RELAXED, __HIP_MEMORY_SCOPE_AGENT);
    char* fpeer = (char*)p.ws_all[lane & 7];
    u32 par;
    auto expired = [&](u64 t0) {
        if (rfl(__hip_atomic_load(tmo, __ATOMIC_RELAXED, __HIP_MEMORY_SCOPE_AGENT)) != call && rt() - t0 <= DEADLINE) return false;
        __hip_atomic_store(tmo, call, __ATOMIC_RELAXED, __HIP_MEMORY_SCOPE_AGENT);
        return true;
    };
    auto min_free = [&]() {
        const u32x4 a = LDS_AT(u32x4, lbase + L_CTL + C_FREE * 4), b = LDS_AT(u32x4, lbase + L_CTL + C_FREE * 4 + 16);
        return (int)rfl(min(min(min(a.x, a.y), min(a.z, a.w)), min(min(b.x, b.y), min(b.z, b.w))));
    };

    if (wave < 8) {
        const u32 amb = lbase + L_MASK + wave * 2048;
        float scale;
        if (wave < 4) {
        const int cbe = (jj & 1) + 2 * (xc + 8 * ((jj >> 1) & 3)), kse = jj >> 3;
        {
            const u64 src = (u64)p.w + (u64)(kse * 128) * LDW + cbe * 64 + wave * 16;
            dma16<0, 0, 4>(src, (u32)lane * LDW, amb);
            dma16<0, 0, 4>(src, (u32)(lane + 64) * LDW, amb + 1024);
        }
        if (wave == 0 && lane < 16) ctl[lane] = lane >= C_FREE && lane < C_FREE + 4 ? 4 : 0;
        WAIT_LGKM0();
        WG_BARRIER(call);
        scale = p.sx[0] * p.sw[0];
        asm volatile("" : "+v"(scale), "+v"(fpeer));
        call = rfl(call) + 1;
        par = call & 1;
        WAIT_VM(0);
        u32 x[8];
        {
            u32x2 t0, t1, t2, t3;
            const u32 a = amb + (u32)(((lane >> 4) * 16 + ((lane & 15) >> 1)) * 16 + (lane & 1) * 8);
            asm volatile("ds_read_b64_tr_b8 %0, %4\n\tds_read_b64_tr_b8 %1, %4 offset:128\n\t"
                         "ds_read_b64_tr_b8 %2, %4 offset:1024\n\tds_read_b64_tr_b8 %3, %4 offset:1152\n\ts_waitcnt lgkmcnt(0)"
                         : "=&v"(t0), "=&v"(t1), "=&v"(t2), "=&v"(t3) : "v"(a) : "memory");
            x[0] = t0.x; x[1] = t0.y; x[2] = t1.x; x[3] = t1.y; x[4] = t2.x; x[5] = t2.y; x[6] = t3.x; x[7] = t3.y;
        }
        STAMP(1);
        const u32 sbuf = lbase + L_W + (NSW - 1) * BLK_B + wave * 1024;
        LDS_AT(u32x4, sbuf + lane * 16) = u32x4{0, 0, 0, 0};
        u32 sum = 0;
#pragma unroll
        for (int i = 0; i < 8; ++i) sum = __builtin_amdgcn_sad_u8(x[i] & 0x7F7F7F7Fu, 0u, sum);
        sum = rdl(wave_scan_incl(sum), 63);
        int T0 = ((int)sum - 60211) >> 13;
        T0 = T0 < 0 ? 0 : (T0 > 21 ? 21 : T0);
        const u32 Q = (u32)(121 - T0) * 0x01010101u, T4 = (u32)T0 * 0x04040404u;
        const u32 LC = (T4 + 0x24201C00u) | 0xFFu, LKlo = T4 + 0x08040028u, LKhi = T4 + 0x1814100Cu;
        u32 mk[8], Cw[2], esc[2], ne[2], bm = 0;
#pragma unroll
        for (int h = 0; h < 2; ++h) {
            u32 Cb = 0, sh = 0;
            Cw[h] = 0; esc[h] = 0;
#pragma unroll
            for (int i = 0; i < 4; ++i) {
                const int j = 4 * h + i;
                const u32 q = ((x[j] >> 2) & 0x1F1F1F1Fu) + Q;
                const u32 g7 = q & 0x80808080u, gm = (g7 >> 4) - (g7 >> 7);
                u32 c = (q + 0x01010101u) & gm & 0x03030303u, k = q & ~gm & 0x07070707u;
                const u32 e = ~(c | (c >> 1)) & 0x01010101u;
                const u32 t = e + (e << 8), v = t + (t << 16), u = v + Cb;
                if (i >= 2) { const u32 ov = e & ((u + 0x77777777u) >> 7); c |= ov; k &= ~((ov << 3) - ov); }
                const u32 four = (k >> 15) & 0xE00u & (u32)__builtin_amdgcn_sbfe((int)v, 26, 1);
                esc[h] |= __builtin_amdgcn_udot4(k, __builtin_amdgcn_perm(0u, 0x40080100u, v), four, false) << sh;
                sh += 3 * (v >> 24);
                Cb = __builtin_amdgcn_perm(u, u, 0x03030303u);
                Cw[h] |= c << (2 * i);
                const u32 pc = __builtin_amdgcn_perm(0u, LC, c);
                const u32 pk = __builtin_amdgcn_perm(LKhi, LKlo, k | (~pc & 0x80808080u));
                mk[j] = (x[j] ^ (pk & pc)) & 0x7C7C7C7Cu;
                const u32 nz = __builtin_amdgcn_udot4((mk[j] + 0x7C7C7C7Cu) & 0x80808080u, 0x08040201u, 0u, false);
                bm |= j < 2 ? nz >> (7 - 4 * j) : nz << (4 * j - 7);
            }
            ne[h] = min(sh, 24u);
        }
        const u32 cnt = __builtin_popcount(bm), nb = ne[0] + ne[1];
        const u32 both = wave_scan_incl(nb | (cnt << 16)), tot = rdl(both, 63);
        const u32 ndw = ((tot & 0xFFFFu) + 31) >> 5, nex = tot >> 16, nov = nex > 10 ? nex - 10 : 0;
        const u32 need = ndw + (nov ? (nov + 2) >> 1 : 0);
        const bool raw = need > 192;
        u32 start = 0;
        if (lane == 0) start = __hip_atomic_fetch_add((LAS u32*)(L + L_CTL) + C_ALLOC, raw ? 192u : need, __ATOMIC_RELAXED, __HIP_MEMORY_SCOPE_WORKGROUP);
        char* blk = p.ws + WST_OFF + par * WST_PAR + (size_t)(cbe * 4 + kse) * BLK_G;
        u32x4 P0; u32 P1;
        P0.x = (x[0] & 0x83838383u) | (__builtin_amdgcn_alignbit(x[1], x[1], 29) & 0x1C1C1C1Cu) | ((x[6] << 5) & 0x60606060u);
        P0.y = (x[2] & 0x83838383u) | (__builtin_amdgcn_alignbit(x[3], x[3], 29) & 0x1C1C1C1Cu) | ((x[6] >> 1) & 0x40404040u) | ((x[7] >> 2) & 0x20202020u);
        P0.z = (x[4] & 0x83838383u) | (__builtin_amdgcn_alignbit(x[5], x[5], 29) & 0x1C1C1C1Cu) | ((x[7] << 5) & 0x60606060u);
        P0.w = Cw[0]; P1 = Cw[1];
        if (__builtin_expect(raw, 0)) { P0 = u32x4{x[0], x[1], x[2], x[3]}; P1 = x[4]; }
        st16_sys(blk + wave * FR_B + lane * 16, P0);
        st4_sys(blk + wave * FR_B + 1024 + lane * 4, P1);
        start = rfl(start);
        char* pool = blk + POOL_B + start * 4;
        u32 ev = 0;
        if (__builtin_expect(raw, 0)) {
            st4_sys(pool + lane * 4, x[5]); st4_sys(pool + 256 + lane * 4, x[6]); st4_sys(pool + 512 + lane * 4, x[7]);
        } else {
            const u32 bp = (both & 0xFFFFu) - nb;
            const u64 V = (u64)esc[0] | ((u64)esc[1] << ne[0]), U = V << (bp & 31);
            LAS u32* sp = (LAS u32*)(size_t)(sbuf + (bp >> 5) * 4);
            __hip_atomic_fetch_or(sp, (u32)U, __ATOMIC_RELAXED, __HIP_MEMORY_SCOPE_WORKGROUP);
            __hip_atomic_fetch_or(sp + 1, (u32)(U >> 32), __ATOMIC_RELAXED, __HIP_MEMORY_SCOPE_WORKGROUP);
            __hip_atomic_fetch_or(sp + 2, ((u32)(V >> 32) >> 1) >> (31 - (bp & 31)), __ATOMIC_RELAXED, __HIP_MEMORY_SCOPE_WORKGROUP);
            for (u32 i = lane; i < ndw; i += 64) st4_sys(pool + i * 4, LDS_AT(u32, sbuf + i * 4));
            u32 idx = (both >> 16) - cnt;
            while (__any(bm != 0)) {
                const u32 pos = __builtin_ctz(bm | 0x80000000u);
                const bool b0 = pos & 4, b1 = pos & 8, b2 = pos & 16;
                const u32 m01 = b0 ? mk[1] : mk[0], m23 = b0 ? mk[3] : mk[2], m45 = b0 ? mk[5] : mk[4], m67 = b0 ? mk[7] : mk[6];
                const u32 m03 = b1 ? m23 : m01, m47 = b1 ? m67 : m45, m = b2 ? m47 : m03;
                const u32 en = (pos >> 4) * 1024 + lane * 16 + (pos & 15) + (((m >> (8 * (pos & 3) + 2)) & 31u) << 11);
                ev |= (u32)__builtin_amdgcn_ds_permute((int)((bm != 0 && idx < 10 ? idx + 2 : 63u) * 4), (int)en);
                if (__builtin_expect(bm != 0 && idx >= 10, 0)) st2_sys(pool + ndw * 4 + (idx - 9) * 2, en);
                ++idx; bm &= bm - 1;
            }
            if (nov && lane == 0) st2_sys(pool + ndw * 4, nov);
        }
        if (lane == 0) ev = (u32)T0 | (raw || nov ? 32u : 0u) | (raw ? 64u : 0u);
        if (lane == 1) ev = start;
        if (lane < 12) st2_sys(blk + MAIN_B + wave * ENT_B + lane * 2, ev);
        LDS_AT(u32x4, amb + lane * 16) = u32x4{0, 0, 0, 0};
        LDS_AT(u32x4, amb + 1024 + lane * 16) = u32x4{0, 0, 0, 0};
        WAIT_VM(0);
        u32 a2 = 0;
        if (lane == 0) a2 = __hip_atomic_fetch_add((LAS u32*)(L + L_CTL) + C_ARR, 1u, __ATOMIC_RELAXED, __HIP_MEMORY_SCOPE_WORKGROUP);
        if (rfl(a2) == 3) {
            const u32 n16 = MIN16 + ((ctl[C_ALLOC] + 3) >> 2);
            if (lane < NR && lane != rank) {
                GAS u64* f = (GAS u64*)(fpeer + WFL_OFF + par * WFL_PAR) + ((cbe * 8 + rank) * 4 + kse);
                __hip_atomic_store(f, ((u64)call << 32) | n16, __ATOMIC_RELAXED, __HIP_MEMORY_SCOPE_SYSTEM);
            }
            ctl[C_TPUB] = (u32)rt() | 1u;
        }
        STAMP(2);
        } else {
            LDS_AT(u32x4, amb + lane * 16) = u32x4{0, 0, 0, 0};
            LDS_AT(u32x4, amb + 1024 + lane * 16) = u32x4{0, 0, 0, 0};
            WG_BARRIER(call);
            call = call + 1;
            scale = p.sx[0] * p.sw[0];
            asm volatile("" : "+v"(scale));
        }

        const int fw = wave & 3, grp = wave >> 2;
        f32x4 acc[8];
#pragma unroll
        for (int f = 0; f < 8; ++f) acc[f] = f32x4{0, 0, 0, 0};
        int avail = 0;
        const u32 aq0 = lbase + L_W + fw * FR_B + lane * 16, aq1 = lbase + L_W + fw * FR_B + 1024 + lane * 4;
        const u32 aen = lbase + L_W + MAIN_B + fw * ENT_B + (lane < 12 ? lane : 0) * 2, axx = lbase + L_X + lane * 16;
        const u32 scr = lbase + L_CTL + 64 + lane;
        auto wait_for = [&](int i) {
            u64 t0 = 0;
            for (u32 it = 1;; ++it) {
                const u32x2 fl = LDS_AT(u32x2, lbase + L_CTL);
                avail = (int)rfl(min(fl.x + 4, fl.y));
                if (i < avail) break;
                if (__builtin_expect((it & 1023) == 0, 0)) {
                    const u64 t = rt();
                    if (t0 == 0) t0 = t;
                    if (t - t0 > 2 * DEADLINE) { ctl[C_FAIL] = 1; avail = NSTEP; break; }
                }
                __builtin_amdgcn_s_sleep(1);
            }
            asm volatile("" ::: "memory");
        };
        auto mma8 = [&](int next, const i32x8& wf, const u32x4 (&xa)[16]) {
#pragma unroll
            for (int f = 0; f < 8; ++f) {
                const u32x4 a = xa[2 * f], b = xa[2 * f + 1];
                i32x8 xf;
                xf[0] = a.x; xf[1] = a.y; xf[2] = a.z; xf[3] = a.w; xf[4] = b.x; xf[5] = b.y; xf[6] = b.z; xf[7] = b.w;
                if (f == 3) {
                    asm volatile("" :: "v"(xa[15]) : "memory");
                    ctl[C_FREE + wave] = next;
                    __builtin_amdgcn_sched_barrier(0);
                }
                acc[f] = __builtin_amdgcn_mfma_scale_f32_16x16x128_f8f6f4(wf, xf, acc[f], 1  , 0  , 0, 0, 0, 0);
            }
        };
        if (grp == 1)
#pragma nounroll
        for (int i = 0; i < 4; ++i) {
            if (i >= avail) wait_for(i);
            u32x2 t0, t1, t2, t3;
            const u32 a = lbase + L_W + i * BLK_B + fw * 16 + (u32)(((lane >> 4) * 16 + ((lane & 15) >> 1)) * 64 + (lane & 1) * 8);
            asm volatile("ds_read_b64_tr_b8 %0, %4\n\tds_read_b64_tr_b8 %1, %4 offset:512\n\t"
                         "ds_read_b64_tr_b8 %2, %4 offset:4096\n\tds_read_b64_tr_b8 %3, %4 offset:4608\n\ts_waitcnt lgkmcnt(0)"
                         : "=&v"(t0), "=&v"(t1), "=&v"(t2), "=&v"(t3) : "v"(a) : "memory");
            u32x4 xa[16];
#pragma unroll
            for (int f = 0; f < 16; ++f) xa[f] = LDS_AT(u32x4, axx + i * XS_B + f * 1024);
            __builtin_amdgcn_sched_barrier(0);
            mma8(i < 3 ? i + 1 : 5, i32x8{(int)t0.x, (int)t0.y, (int)t1.x, (int)t1.y, (int)t2.x, (int)t2.y, (int)t3.x, (int)t3.y}, xa);
        }
        u32 xo = grp * XS_B, wo = (4 + grp) * BLK_B;
#pragma nounroll
        for (int i = 4 + grp; i < NSTEP; i += 2) {
            if (i >= avail) wait_for(i);
            const u32 ax = axx + xo;
            const u32 e0 = LDS_AT(unsigned short, aen + wo);
            const u32x4 q0 = LDS_AT(u32x4, aq0 + wo);
            const u32 q1 = LDS_AT(u32, aq1 + wo);
            u32x4 xa[16];
#pragma unroll
            for (int f = 0; f < 4; ++f) xa[f] = LDS_AT(u32x4, ax + f * 1024);
            __builtin_amdgcn_sched_barrier(0);
            const u32 hdr = rfl(e0), pool = lbase + L_W + POOL_B + wo + (rdl(e0, 1) & 1023u) * 4;
            const u32 ea = e0 >> 11 ? amb + (e0 & 2047u) : scr;
            LDS_AT(unsigned char, ea) = (unsigned char)(e0 >> 11 << 2);
            const u32 n0 = __builtin_popcount(~(q0.w | (q0.w >> 1)) & 0x55555555u), nn = n0 + __builtin_popcount(~(q1 | (q1 >> 1)) & 0x55555555u);
            const u32 incl = wave_scan_incl(nn), bp = 3 * (incl - nn);
            const u32 sa = pool + (bp >> 5) * 4;
            const u32 d0 = LDS_AT(u32, sa), d1 = LDS_AT(u32, sa + 4), d2 = LDS_AT(u32, sa + 8);
            if (__builtin_expect((hdr & 96u) == 32u, 0)) {
                const u32 ov = pool + ((3 * rdl(incl, 63) + 31) >> 5) * 4;
                const u32 n = rfl(LDS_AT(unsigned short, ov)) & 511u;
                for (u32 k = 0; k < n; k += 64) {
                    const u32 e = LDS_AT(unsigned short, ov + 2 + (k + lane) * 2);
                    if (k + lane < n) LDS_AT(unsigned char, amb + (e & 2047u)) = (unsigned char)(e >> 11 << 2);
                }
                __builtin_amdgcn_s_waitcnt(0xC07F);
            }
            const u32x4 m0 = LDS_AT(u32x4, amb + lane * 16);
            const u32x4 m1 = LDS_AT(u32x4, amb + 1024 + lane * 16);
            LDS_AT(unsigned char, ea) = 0;
            if (__builtin_expect(hdr & 32u, 0)) { LDS_AT(u32x4, amb + lane * 16) = u32x4{0, 0, 0, 0}; LDS_AT(u32x4, amb + 1024 + lane * 16) = u32x4{0, 0, 0, 0}; }
#pragma unroll
            for (int f = 4; f < 16; ++f) xa[f] = LDS_AT(u32x4, ax + f * 1024);
            __builtin_amdgcn_sched_barrier(0);
            const u32 T4 = (hdr & 31u) * 0x04040404u;
            const u32 LC = (T4 + 0x24201C00u) | 0xFFu, LKlo = T4 + 0x08040028u, LKhi = T4 + 0x1814100Cu;
            u32 pc[8], sel[8];
#pragma unroll
            for (int h = 0; h < 2; ++h) {
                const u32 C = h ? q1 : q0.w;
                u32 Cb = 0x07070707u;
#pragma unroll
                for (int i = 0; i < 4; ++i) {
                    const int j = 4 * h + i;
                    pc[j] = __builtin_amdgcn_perm(0u, LC, (C >> (2 * i)) & 0x03030303u);
                    const u32 e = pc[j] & 0x01010101u, t = e + (e << 8), u = (t << 16) + (t + Cb);
                    sel[j] = __builtin_amdgcn_bitop3_b32(u, pc[j], 0x07070707u, 0xB1);
                    Cb = __builtin_amdgcn_perm(u, u, 0x03030303u);
                }
            }
            const u32 w0 = __builtin_amdgcn_alignbit(d1, d0, bp), w1 = __builtin_amdgcn_alignbit(d2, d1, bp);
            i32x8 wf;
#pragma unroll
            for (int h = 0; h < 2; ++h) {
                const u32 w = h ? __builtin_amdgcn_alignbit(w1, w0, 3 * n0) : w0;
                const u32 ya = (w & 0x3Fu) | ((w << 10) & 0x3F0000u), yb = ((w >> 12) & 0x3Fu) | ((w >> 2) & 0x3F0000u);
                const u32 Ka = __builtin_amdgcn_perm(LKhi, LKlo, (ya | (ya << 5)) & 0x07070707u);
                const u32 Kb = __builtin_amdgcn_perm(LKhi, LKlo, (yb | (yb << 5)) & 0x07070707u);
                const u32x4 m = h ? m1 : m0;
#pragma unroll
                for (int i = 0; i < 4; ++i) wf[4 * h + i] = (__builtin_amdgcn_perm(Kb, Ka, sel[4 * h + i]) & pc[4 * h + i]) ^ m[i];
            }
            wf[0] |= q0.x & 0x83838383u; wf[1] |= __builtin_amdgcn_alignbit(q0.x, q0.x, 3) & 0x83838383u;
            wf[2] |= q0.y & 0x83838383u; wf[3] |= __builtin_amdgcn_alignbit(q0.y, q0.y, 3) & 0x83838383u;
            wf[4] |= q0.z & 0x83838383u; wf[5] |= __builtin_amdgcn_alignbit(q0.z, q0.z, 3) & 0x83838383u;
            wf[6] |= ((q0.x >> 5) & 0x03030303u) | ((q0.y << 1) & 0x80808080u);
            wf[7] |= ((q0.z >> 5) & 0x03030303u) | ((q0.y << 2) & 0x80808080u);
            if (__builtin_expect(hdr & 64u, 0)) {
                wf[0] = q0.x; wf[1] = q0.y; wf[2] = q0.z; wf[3] = q0.w; wf[4] = q1;
                wf[5] = LDS_AT(u32, pool + lane * 4); wf[6] = LDS_AT(u32, pool + 256 + lane * 4); wf[7] = LDS_AT(u32, pool + 512 + lane * 4);
                __builtin_amdgcn_s_waitcnt(0xC07F);
            }
            __builtin_amdgcn_sched_barrier(0);
            mma8(i + 2, wf, xa);
            xo ^= 2 * XS_B;
            wo = wo + 2 * BLK_B >= NSW * BLK_B ? wo + 2 * BLK_B - NSW * BLK_B : wo + 2 * BLK_B;
        }
        STAMP(5);
        constexpr u32 T_0 = L_X, T_1 = L_W + 5 * BLK_B;
        for (u32 it = 0; min_free() < NSTEP - 2 && it < (1u << 26); ++it) __builtin_amdgcn_s_sleep(0);
        const u32 tw = lbase + (grp ? T_1 : T_0) + (lane & 15) * 256 + (((fw * 4 + (lane >> 4)) ^ (lane & 15)) * 16);
#pragma unroll
        for (int f = 0; f < 8; ++f) LDS_AT(f32x4, tw + f * 4096) = acc[f];
        WAIT_LGKM0();
        ctl[C_FREE + wave] = 2 * NSTEP;
        for (u32 it = 0; min_free() < 2 * NSTEP && it < (1u << 26); ++it) __builtin_amdgcn_s_sleep(0);
        const float sc = ctl[C_FAIL] != 0 ? __builtin_nanf("") : scale;
        float* o = p.out + (size_t)(rb * 64 + wave * 16 + (lane >> 4)) * LDO + cb * 64 + (lane & 15) * 4;
        f32x4 y0[4], y1[4];
#pragma unroll
        for (int f = 0; f < 4; ++f) {
            const u32 tr = lbase + (wave * 16 + f * 4 + (lane >> 4)) * 256 + (((lane & 15) ^ (f * 4 + (lane >> 4))) * 16);
            y0[f] = LDS_AT(f32x4, tr + T_0); y1[f] = LDS_AT(f32x4, tr + T_1);
        }
#pragma unroll
        for (int f = 0; f < 4; ++f)
            st16_sys(o + (size_t)f * 4 * LDO, __builtin_bit_cast(u32x4, __builtin_elementwise_maximum((y0[f] + y1[f]) * sc, f32x4{0, 0, 0, 0})));
        STAMP(6);
    } else if (wave == 8) {
        WG_BARRIER(call);
        call = rfl(call) + 1;
        par = call & 1;
        const int R = lane < NREM ? lane : 0, src = src_of_rem(R, rank, cb), ks = R / 7;
        const u64 baddr = (((u64)(u32)__shfl((int)((u64)fpeer >> 32), src) << 32) | (u32)__shfl((int)(u32)(u64)fpeer, src)) + WST_OFF + par * WST_PAR + (u64)(cb * 4 + ks) * BLK_G;
        const u32 blo = (u32)baddr, bhi = (u32)(baddr >> 32);
        GAS u64* fl = (GAS u64*)(p.ws + WFL_OFF + par * WFL_PAR) + ((cb * 8 + src) * 4 + ks);
        const u32 voff = lane * 16;
        u32 n16v = MIN16;
        bool dead = false;
        if (FENCE_MODE == 3) {
            const u32 xcc = __builtin_amdgcn_s_getreg((3 << 11) | 20) & 15;
            GAS u64* inv = (GAS u64*)(p.ws + INV_OFF + xcc * 256);
            u32 prev = 0;
            if (lane == 0) prev = __hip_atomic_fetch_max((GAS u32*)(inv + 16), call, __ATOMIC_RELAXED, __HIP_MEMORY_SCOPE_AGENT);
            if (rfl(prev) < call) {
                __builtin_amdgcn_fence(__ATOMIC_ACQUIRE, "");
                WAIT_VM(0);
                if (lane == 0) __hip_atomic_store(inv, (u64)call << 32, __ATOMIC_RELAXED, __HIP_MEMORY_SCOPE_AGENT);
            }
            if (lane == 63) fl = inv;
            STAMP(3);
        }
        if (lane >= 28 && lane < 56) {
            const int q = lane - 28, src2 = (rank + 1 + q % 7) & 7, j = q / 7;
            fl = (GAS u64*)(p.ws + XFL_OFF + par * XFL_PAR) + (NPUSH <= 16 ? ((rb + (j >> 1)) * 8 + src2) * 4 + (j & 1) : (rb * 8 + src2) * 4 + j);
        }
        if (lane == 62) fl = (GAS u64*)(p.ws + TMO_OFF);
        u64 ready = 0;
        auto wait_flags = [&](u64 need, int xth) {
            const u64 t0 = rt();
            for (u32 it = 1;; ++it) {
                const u64 v = __hip_atomic_load(fl, __ATOMIC_RELAXED, __HIP_MEMORY_SCOPE_SYSTEM);
                n16v = (u32)v;
                ready = __ballot((u32)(v >> 32) >= call);
                if ((ready >> 62) & 1) { dead = rdl((u32)(v >> 32), 62) == call; if (dead) break; }
                if ((ready & need) == need && (__builtin_popcount((u32)(ready >> 28) & 0xFFFFFFFu) >= xth || rt() - t0 > 2000)) break;
                if ((it & 15) == 0 && expired(t0)) { dead = true; break; }
                __builtin_amdgcn_s_sleep(1);
            }
            n16v = n16v < MIN16 ? MIN16 : (n16v > MAX16 ? MAX16 : n16v);
        };
        wait_flags((FENCE_MODE == 3 ? 1ull << 63 : 0) | (LAZY ? 1 : 0xFFFFFFFull), FENCE_MODE == 0 ? 0 : XTH);
        STAMP(1);
        if (FENCE_MODE == 0) { __builtin_amdgcn_fence(__ATOMIC_ACQUIRE, ""); WAIT_VM(0); wait_flags(0, XTH); }
        for (u32 it = 0; rfl(ctl[C_TPUB]) == 0 && it < (1u << 24); ++it) __builtin_amdgcn_s_sleep(1);
        ctl[C_GO] = 1;
        STAMP(4);
        __builtin_amdgcn_s_setprio(PRIO_W);
        constexpr int RP = FENCE_MODE == 3 ? 2 : 0;
        int pub = 0, freed = 0;
        u32 lo = lbase + L_W + 4 * BLK_B;
        auto publish = [&](int upto) { if (upto > pub) { pub = upto; ctl[C_FULLW] = upto; } };
        u64 g[LAST + 1]; u32 l[LAST + 1];
#pragma unroll
        for (int j = 0; j <= LAST; ++j) { g[j] = 0; l[j] = 0; }
#define W_LOAD(OFF, J, K, NOP) do { dma16<OFF, RP, NOP>(g[J], voff, l[J]); WAIT_VM(W); after(std::integral_constant<int, K>{}); } while (0)
        auto rest_of_slot = [&](int n, bool mid, auto win, auto&& after) {
            constexpr int W = decltype(win)::value;
            if (mid) { W_LOAD(2048, LAG, 3, 0); W_LOAD(3072, LAG, 4, 0); }
            if (n >= LAST) {
                g[LAST] += 4096; l[LAST] += 4096;
                W_LOAD(0, LAST, 5, 0);
                const u32 n16 = rdl(n16v, n - LAST);
                if (__builtin_expect(n16 > 64, 0))
                    for (u32 i = 64; i < n16; i += 64) { if (lane + i < n16) dma16<1024, RP>(g[LAST] + i * 16, voff, l[LAST] + i * 16); WAIT_VM(W); }
                if ((u32)lane < n16) dma16<1024, RP>(g[LAST], voff, l[LAST]);
                WAIT_VM(W); after(std::integral_constant<int, SL>{});
            }
#pragma unroll
            for (int j = LAST; j > 0; --j) { g[j] = g[j - 1]; l[j] = l[j - 1]; }
        };
        auto open = [&](int n, int done) {
            if (LAZY && !((ready >> n) & 1)) { WAIT_VM(0); publish(done); wait_flags(1ull << n, 0); if (dead) return; }
            if (freed <= n + 4 - NSW) {
                const int need = n + 4 - NSW;
                const u64 t0 = rt();
                bool drained = false;
                for (;;) {
                    freed = min_free();
                    if (freed > need) break;
                    if (!drained) { WAIT_VM(0); publish(done); drained = true; continue; }
                    if (rt() - t0 > 2 * DEADLINE) { dead = true; break; }
                    __builtin_amdgcn_s_sleep(1);
                }
                if (dead) return;
            }
            g[0] = ((u64)rdl(bhi, n) << 32) | rdl(blo, n);
            l[0] = lo;
            lo = lo + BLK_B == lbase + L_W + NSW * BLK_B ? lbase + L_W : lo + BLK_B;
        };
        auto slot = [&](int n, auto win) {
            constexpr int W = decltype(win)::value;
            constexpr int PK = (W - 1) % SL + 1, PL = LAST + 1 + (W - PK) / SL;
            auto after = [&](auto k) { if constexpr (decltype(k)::value == PK) publish(n - PL + 1); };
            open(n, n - LAST);
            if (dead) return;
            W_LOAD(0, 0, 1, 4); W_LOAD(1024, 0, 2, 0);
            rest_of_slot(n, n >= LAG, win, after);
        };
        for (int n = 0; n < NB - NW2 && !dead; ++n) slot(n, std::integral_constant<int, WIN>{});
        for (int n = NB - NW2; n < NB && !dead; ++n) slot(n, std::integral_constant<int, WIN2>{});
        if (!dead) {
            static_for<LAST>([&](auto dd) {
                constexpr int d = decltype(dd)::value;
                rest_of_slot(NB + d, d < LAG, std::integral_constant<int, WIN2>{},
                             [&](auto k) { publish(std::integral_constant<int, w_landed(w_cum(d) - (SL - decltype(k)::value), WIN2)>::value); });
            });
            open(NB, NB);
        }
        if (dead) { ctl[C_FAIL] = 1; WAIT_VM(0); ctl[C_FULLW] = NREM; }
        else {
            constexpr int W = WIN2;
            auto after = [&](auto k) { publish(std::integral_constant<int, w_landed(w_cum(LAST) - (SL - decltype(k)::value), WIN2)>::value); };
            W_LOAD(0, 0, 1, 4); W_LOAD(1024, 0, 2, 0);
            g[LAG] = g[LAST] = g[0]; l[LAG] = l[LAST] = l[0];
            rest_of_slot(NB + LAST, true, std::integral_constant<int, WIN2>{}, after);
            STAMP(5);
            static_for<LAST + 4>([&](auto dd) {
                constexpr int upto = NREM - LAST - 3 + decltype(dd)::value, behind = w_behind(upto - 1, w_cum(LAST));
                if constexpr (behind < WIN2) { WAIT_VM(behind); publish(upto); }
            });
        }
        STAMP(6);
    } else if (wave == 9) {
        WG_BARRIER(call);
        call = rfl(call) + 1;
        par = call & 1;
        const int R = max((lane & 31) - 4, 0), src = src_of_rem(R, rank, cb), ks = R / 7;
        const u32 toff = (u32)((src * 4 + ks) * 8 + rb) * XT_B;
        const u64 xbase = (u64)p.ws + XRV_OFF + par * XRV_PAR;
        GAS u64* fl = (GAS u64*)(p.ws + XFL_OFF + par * XFL_PAR) + (((rb + (lane >> 5)) * 8 + src) * 4 + ks);
        constexpr u32 OWN = 0xFu;
        const u32 voff = lane * 16;
        const u32 xoff = (u32)(lane & 15) * LDX + (lane >> 4) * 16;
        u32 ready = OWN;
        u32 it = 0, la = lbase + L_X;
        bool dead = false;
        const u64 t0 = rt();
        int pub = 0, freed = 0;
        for (int s = 0; s < NSTEP && !dead; ++s) {
            if (!((ready >> s) & 1)) {
                WAIT_VM(0);
                if (pub < s) { pub = s; ctl[C_FULLX] = s; }
                for (;;) {
                    const u64 v = __hip_atomic_load(fl, __ATOMIC_RELAXED, __HIP_MEMORY_SCOPE_SYSTEM);
                    const u64 ok = __ballot((u32)(v >> 32) >= call);
                    ready = ((u32)ok & (u32)(ok >> 32)) | OWN;
                    if (ready == ~0u) { dead = rfl(__hip_atomic_load(tmo, __ATOMIC_RELAXED, __HIP_MEMORY_SCOPE_AGENT)) == call; break; }
                    if ((ready >> s) & 1) break;
                    if ((++it & 15) == 0 && expired(t0)) { dead = true; break; }
                    __builtin_amdgcn_s_sleep(2);
                }
            }
            while (!dead && freed <= s - NSX) {
                freed = min_free();
                if (freed > s - NSX) break;
                if ((++it & 1023) == 0 && rt() - t0 > 2 * DEADLINE) dead = true;
                __builtin_amdgcn_s_sleep(1);
            }
            if (dead) break;
            if (s < 4) {
                const u64 b = (u64)p.x + (u64)(rank * 512 + rb * 64) * LDX + s * 128;
#pragma unroll
                for (int i = 0; i < 16; ++i) dma16<0, 0>(b + (i >> 1) * 16 * LDX + (i & 1) * 64, xoff, la + i * 1024);
#pragma unroll
                for (int i = 0; i < 8; ++i)
                    dma16<0, 0>((u64)p.w + (u64)(s * 128 + i * 16) * LDW + cb * 64, (u32)(lane >> 2) * LDW + (lane & 3) * 16, lbase + L_W + s * BLK_B + i * 1024);
            } else {
                const u64 b = xbase + rdl(toff, s);
#pragma unroll
                for (int i = 0; i < 4; ++i) {
                    dma16<0, 1, 4>(b + i * 4096, voff, la + i * 4096); dma16<1024, 1>(b + i * 4096, voff, la + i * 4096);
                    dma16<2048, 1>(b + i * 4096, voff, la + i * 4096); dma16<3072, 1>(b + i * 4096, voff, la + i * 4096);
                }
            }
            la = la + XS_B == lbase + L_X + NSX * XS_B ? lbase + L_X : la + XS_B;
            WAIT_VM(16 * (XD - 1));
            if (pub < s - XD + 2) { pub = s - XD + 2; ctl[C_FULLX] = pub; }
        }
        if (dead) { ctl[C_FAIL] = 1; WAIT_VM(0); }
        else {
            WAIT_VM(32); ctl[C_FULLX] = max(pub, NSTEP - 2);
            WAIT_VM(16); ctl[C_FULLX] = max(pub, NSTEP - 1);
            WAIT_VM(0);
        }
        ctl[C_FULLX] = NSTEP;
        STAMP(6);
    } else {
        const int id = bid / 7, d = (rank + 1 + bid % 7) & 7, s = (rank + 7 - bid % 7) & 7;
        const bool push = id < NPUSH;
        u32x4 v[8];
        if (id < 32) {
            const unsigned char* g = p.x + (size_t)(d * 512 + (id & 7) * 64 + (lane & 15)) * LDX + (id >> 3) * 128 + (lane >> 4) * 16;
#pragma unroll
            for (int i = 0; i < 8; ++i) v[i] = *(const GAS u32x4*)(g + (size_t)(i >> 1) * 16 * LDX + (i & 1) * 64);
        }
        WG_BARRIER(call);
        call = rfl(call) + 1;
        par = call & 1;
        if (id < 32) {
            asm volatile("" : "+v"(v[0]), "+v"(v[1]), "+v"(v[2]), "+v"(v[3]), "+v"(v[4]), "+v"(v[5]), "+v"(v[6]), "+v"(v[7]), "+v"(fpeer));
            char* peer = (char*)(((u64)rdl((u32)((u64)fpeer >> 32), d) << 32) | rdl((u32)(u64)fpeer, d));
            char* dst = push ? peer + XRV_OFF + par * XRV_PAR + (size_t)(rank * 32 + id) * XT_B + lane * 16
                             : p.ws + XST_OFF + par * XST_PAR + (size_t)(d * 32 + id) * XT_B + lane * 16;
            bool open = false;
#pragma unroll
            for (int i = 0; i < 8; ++i) {
                st16_sys(dst + i * 1024, v[i]);
                if (push) {
                    if (!open) { const u32 tp = rfl(ctl[C_TPUB]); open = tp != 0 && (u32)rt() - tp >= PUSH_LAG; }
                    if (open) WAIT_VM(PUSH_W1 - 1); else WAIT_VM(PUSH_W0 - 1);
                }
            }
            STAMP(1);
            WAIT_VM(0);
            if (lane == 0) {
                GAS u64* f = push ? (GAS u64*)(peer + XFL_OFF + par * XFL_PAR) + (((id & 7) * 8 + rank) * 4 + (id >> 3))
                                  : (GAS u64*)(peer + XSF_OFF + par * XSF_PAR) + (rank * 32 + id);
                __hip_atomic_store(f, (u64)call << 32, __ATOMIC_RELAXED, __HIP_MEMORY_SCOPE_SYSTEM);
            }
            STAMP(2);
            if (!push) {
                GAS u64* f = (GAS u64*)(p.ws + XSF_OFF + par * XSF_PAR) + (s * 32 + id);
                const u64 t0 = rt();
                bool dead = false;
                for (u32 it = 1; (u32)(__hip_atomic_load(f, __ATOMIC_RELAXED, __HIP_MEMORY_SCOPE_SYSTEM) >> 32) < call; ++it) {
                    if ((it & 15) == 0 && expired(t0)) { dead = true; break; }
                    __builtin_amdgcn_s_sleep(4);
                }
                while (!dead && rfl(ctl[C_GO]) == 0 && rt() - t0 < 4000) __builtin_amdgcn_s_sleep(4);
                if (!dead) {
                    const char* from = (char*)(((u64)rdl((u32)((u64)fpeer >> 32), s) << 32) | rdl((u32)(u64)fpeer, s))
                                       + XST_OFF + par * XST_PAR + (size_t)(rank * 32 + id) * XT_B + lane * 16;
#pragma unroll
                    for (int i = 0; i < 8; i += PULL_W) {
                        if constexpr (PULL_W == 1)
                            asm volatile("global_load_dwordx4 %0, %1, off sc0 sc1\n\ts_waitcnt vmcnt(0)" : "=&v"(v[i]) : "v"(from + i * 1024) : "memory");
                        else
                            asm volatile("global_load_dwordx4 %0, %2, off sc0 sc1\n\tglobal_load_dwordx4 %1, %2, off offset:1024 sc0 sc1\n\ts_waitcnt vmcnt(0)"
                                         : "=&v"(v[i]), "=&v"(v[i + 1]) : "v"(from + i * 1024) : "memory");
                    }
                    char* to = p.ws + XRV_OFF + par * XRV_PAR + (size_t)(s * 32 + id) * XT_B + lane * 16;
#pragma unroll
                    for (int i = 0; i < 8; ++i) st16_sys(to + i * 1024, v[i]);
                    WAIT_VM(0);
                    if (lane == 0)
                        __hip_atomic_store((GAS u64*)(p.ws + XFL_OFF + par * XFL_PAR) + (((id & 7) * 8 + s) * 4 + (id >> 3)), (u64)call << 32,
                                           __ATOMIC_RELAXED, __HIP_MEMORY_SCOPE_SYSTEM);
                }
                STAMP(3);
            }
        }
    }
    if (lane == 0) __hip_atomic_store(cntp, call, __ATOMIC_RELAXED, __HIP_MEMORY_SCOPE_AGENT);
}

extern "C" void kernel_launch(int rank, int nranks, ncclComm_t comm,
                              hipStream_t stream,
                              void* const* d_in, const int* in_sizes, int n_in,
                              void* d_out, int out_size,
                              const int* in_dtypes, int out_dtype,
                              void* const* d_out_all,
                              void* const* d_ws_all, size_t ws_bytes) {
    (void)comm; (void)in_dtypes; (void)out_dtype; (void)d_out_all;
    if (nranks != NR || (unsigned)rank >= (unsigned)NR || n_in != 4 || ws_bytes < WS_NEED ||
        in_sizes[0] != 4096 * 512 || in_sizes[1] != 512 * 4096 || out_size != 512 * 4096) {
        fprintf(stderr, "unexpected problem shape\n");
        return;
    }
    HIP_CHECK(hipFuncSetAttribute((const void*)gemm_xchg, hipFuncAttributeMaxDynamicSharedMemorySize, LDS_BYTES));
    Params p;
    p.x = (const unsigned char*)d_in[0];
    p.w = (const unsigned char*)d_in[1];
    p.sx = (const float*)d_in[2];
    p.sw = (const float*)d_in[3];
    p.out = (float*)d_out;
    p.ws_all = d_ws_all;
    p.ws = (char*)d_ws_all[rank];
    p.rank = rank;
    p.pad = 0;
    hipLaunchKernelGGL(gemm_xchg, dim3(NWG), dim3(NTHR), LDS_BYTES, stream, p);
    HIP_CHECK(hipGetLastError());
}
```

```cpp
#include <hip/hip_runtime.h>
#include <rccl/rccl.h>
#include <cstdio>

#define HIP_CHECK(expr) do { hipError_t _e = (expr); if (_e != hipSuccess) { \
    fprintf(stderr, "HIP error %s at %s:%d\n", hipGetErrorString(_e), __FILE__, __LINE__); \
    return; } } while (0)

typedef unsigned u32;
typedef unsigned long long u64;
typedef unsigned u32x4 __attribute__((ext_vector_type(4)));
typedef unsigned u32x2 __attribute__((ext_vector_type(2)));
typedef int i32x8 __attribute__((ext_vector_type(8)));
typedef float f32x4 __attribute__((ext_vector_type(4)));
#define GAS __attribute__((address_space(1)))
#define LAS __attribute__((address_space(3)))

#define WIN 4
#define NU 10
#define ESLEEP 1
#define PRIO_C 1
#define PRIO_W 3
#define XTH 10
#define PUSH_W 2
#define XD 2
#define NSW 9
#define NSX 4

constexpr int NR = 8;
constexpr int NWG = 256, NTHR = 704;
constexpr int LDX = 512, LDW = 4096, LDO = 4096;
constexpr int MAIN_B = 5120, FR_B = 1280, ENT_B = 24, POOL_B = MAIN_B + 4 * ENT_B;
constexpr int MIN16 = 4 * ENT_B / 16, MAX16 = (4 * ENT_B + 4 * 768) / 16;
constexpr int BLK_B = 8320;
constexpr int BLK_G = 8448;
constexpr int XT_B = 8192;
constexpr int XS_B = 2 * XT_B;
constexpr int NSTEP = 32, NREM = 28;
static_assert(MAIN_B + MAX16 * 16 + 16 <= BLK_B && BLK_B <= BLK_G, "block");

constexpr size_t WST_OFF = 0, WST_PAR = 4u << 20;
constexpr size_t XRV_OFF = 8u << 20, XRV_PAR = 2u << 20;
constexpr size_t WFL_OFF = 12u << 20, WFL_PAR = 16u << 10;
constexpr size_t XFL_OFF = WFL_OFF + (32u << 10), XFL_PAR = 4u << 10;
constexpr size_t CNT_OFF = WFL_OFF + (64u << 10);
constexpr size_t TMO_OFF = WFL_OFF + (128u << 10);
constexpr size_t INV_OFF = TMO_OFF + 1024;
constexpr size_t DBG_OFF = WFL_OFF + (256u << 10);
constexpr size_t WS_NEED = 13u << 20;
static_assert((size_t)256 * BLK_G <= WST_PAR, "w staging");

constexpr int L_W = 0;
constexpr int L_X = L_W + NSW * BLK_B;
constexpr int L_MASK = L_X + NSX * XS_B;
constexpr int L_CTL = L_MASK + 16384;
constexpr int LDS_BYTES = L_CTL + 192;
static_assert(NSW == 9 && NSX == 4, "the output staging areas are chosen for these rings");
static_assert(LDS_BYTES <= 160 * 1024 && NSW >= 8 && XD <= NSX && XD <= 4, "LDS");
constexpr int C_FULLW = 0, C_FULLX = 1, C_FAIL = 2, C_ALLOC = 3, C_ARR = 4, C_PUB = 5, C_FREE = 8  , C_FREX = 16  ;

constexpr u64 DEADLINE = 30000000ull;

struct Params {
    const unsigned char* x;
    const unsigned char* w;
    const float* sx;
    const float* sw;
    float* out;
    void* const* ws_all;
    char* ws;
    int rank;
    int pad;
};

__device__ __forceinline__ u64 rt() { return __builtin_amdgcn_s_memrealtime(); }
__device__ __forceinline__ u32 rfl(u32 v) { return (u32)__builtin_amdgcn_readfirstlane(v); }
__device__ __forceinline__ u32 rdl(u32 v, int l) { return (u32)__builtin_amdgcn_readlane(v, l); }
#define WAIT_VM(n) asm volatile("s_waitcnt vmcnt(%0)" :: "i"(n) : "memory")
#define WG_BARRIER(call) do { u32 c_; asm volatile("s_barrier" ::: "memory"); \
    asm volatile("v_readfirstlane_b32 %0, %1\n\ts_nop 4" : "=s"(c_) : "v"(call) : "memory"); call = c_; } while (0)
#define WAIT_LGKM0() asm volatile("s_waitcnt lgkmcnt(0)" ::: "memory")

template <int OFF, int POL, int NOP = 0>
__device__ __forceinline__ void dma16(u64 base, u32 voff, u32 ldsaddr) {
    if constexpr (POL == 1)
        asm volatile("s_mov_b32 m0, %2\n\ts_nop %4\n\tglobal_load_lds_dwordx4 %0, %1 offset:%3 sc0 sc1"
                     :: "v"(voff), "s"(base), "s"(ldsaddr), "i"(OFF), "i"(NOP) : "memory");
    else if constexpr (POL == 2)
        asm volatile("s_mov_b32 m0, %2\n\ts_nop %4\n\tglobal_load_lds_dwordx4 %0, %1 offset:%3 nt"
                     :: "v"(voff), "s"(base), "s"(ldsaddr), "i"(OFF), "i"(NOP) : "memory");
    else
        asm volatile("s_mov_b32 m0, %2\n\ts_nop %4\n\tglobal_load_lds_dwordx4 %0, %1 offset:%3"
                     :: "v"(voff), "s"(base), "s"(ldsaddr), "i"(OFF), "i"(NOP) : "memory");
}
__device__ __forceinline__ void st4_sys(void* p, u32 v) {
    asm volatile("global_store_dword %0, %1, off sc0 sc1\n\ts_nop 1" :: "v"(p), "v"(v) : "memory");
}
__device__ __forceinline__ void st2_sys(void* p, u32 v) {
    asm volatile("global_store_short %0, %1, off sc0 sc1\n\ts_nop 1" :: "v"(p), "v"(v) : "memory");
}
__device__ __forceinline__ void st16_sys(void* p, u32x4 v) {
    asm volatile("global_store_dwordx4 %0, %1, off sc0 sc1\n\ts_nop 1" :: "v"(p), "v"(v) : "memory");
}
__device__ __forceinline__ void st8_sys(void* p, u32x2 v) {
    asm volatile("global_store_dwordx2 %0, %1, off sc0 sc1\n\ts_nop 1" :: "v"(p), "v"(v) : "memory");
}
__device__ __forceinline__ u32 wave_scan_incl(u32 v) {
    v += __builtin_amdgcn_update_dpp(0, v, 0x111, 0xf, 0xf, true);
    v += __builtin_amdgcn_update_dpp(0, v, 0x112, 0xf, 0xf, true);
    v += __builtin_amdgcn_update_dpp(0, v, 0x114, 0xf, 0xf, true);
    v += __builtin_amdgcn_update_dpp(0, v, 0x118, 0xf, 0xf, true);
    v += __builtin_amdgcn_update_dpp(0, v, 0x142, 0xa, 0xf, false);
    v += __builtin_amdgcn_update_dpp(0, v, 0x143, 0xc, 0xf, false);
    return v;
}

__device__ __forceinline__ int src_of_rem(int R, int rank, int cb) { return (rank + 1 + (cb + R) % 7) & 7; }

constexpr int LAG = 2, LAST = 2 * LAG, SL = 6, NB = NREM - NU;
static_assert(WIN >= 1 && WIN <= 18 && NSW > LAST + 2 && NU >= 1 && NB >= LAST, "window");
constexpr int w_cum(int d) { int c = 0; for (int t = 0; t <= d; ++t) c += t >= LAST ? SL : (t < LAG ? 4 : 2); return c; }
constexpr int w_behind(int b, int q) { return b >= NB - LAST ? q - w_cum(b - (NB - LAST)) : q + SL * (NB - LAST - 1 - b); }
constexpr int w_landed(int q, int win) { int u = 0; while (u < NREM && w_behind(u, q) >= win) ++u; return u; }
template <int N, int I = 0, class F> __device__ __forceinline__ void static_for(F&& f) {
    if constexpr (I < N) { f(std::integral_constant<int, I>{}); static_for<N, I + 1>(f); }
}

#define STAMP(i) do {} while (0)
#define LDS_AT(T, a) (*(volatile LAS T*)(size_t)(a))

__global__ void __launch_bounds__(NTHR, 3) gemm_xchg(Params p) {
    extern __shared__ __attribute__((aligned(16))) unsigned char lds_raw[];
    LAS unsigned char* L = (LAS unsigned char*)lds_raw;
    const u32 lbase = (u32)(size_t)L;
    volatile LAS u32* ctl = (volatile LAS u32*)(L + L_CTL);
    const int tid = threadIdx.x, lane = tid & 63;
    const int wave = rfl(tid >> 6);
    const int bid = blockIdx.x, xc = bid & 7, jj = bid >> 3;
    const int rank = p.rank;
    const int rb = (jj & 3) * 2, cb = xc * 8 + (jj >> 2);
    GAS u32* tmo = (GAS u32*)(p.ws + TMO_OFF) + 1;
    STAMP(0);

    GAS u32* cntp = (GAS u32*)(p.ws + CNT_OFF) + bid * 16 + wave;
    u32 call = __hip_atomic_load(cntp, __ATOMIC_RELAXED, __HIP_MEMORY_SCOPE_AGENT);
    char* fpeer = (char*)p.ws_all[lane & 7];
    u32 par;
    auto expired = [&](u64 t0) {
        if (rfl(__hip_atomic_load(tmo, __ATOMIC_RELAXED, __HIP_MEMORY_SCOPE_AGENT)) != call && rt() - t0 <= DEADLINE) return false;
        __hip_atomic_store(tmo, call, __ATOMIC_RELAXED, __HIP_MEMORY_SCOPE_AGENT);
        return true;
    };
    auto min_free = [&](int c = C_FREE) {
        const u32x4 a = LDS_AT(u32x4, lbase + L_CTL + c * 4), b = LDS_AT(u32x4, lbase + L_CTL + c * 4 + 16);
        return (int)rfl(min(min(min(a.x, a.y), min(a.z, a.w)), min(min(b.x, b.y), min(b.z, b.w))));
    };

    if (wave < 8) {
        const u32 amb = lbase + L_MASK + wave * 2048;
        float scale;
        if (wave < 4) {
        const int cbe = (jj & 1) + 2 * (xc + 8 * ((jj >> 1) & 3)), kse = jj >> 3;
        {
            const u64 src = (u64)p.w + (u64)(kse * 128) * LDW + cbe * 64 + wave * 16;
            dma16<0, 0, 4>(src, (u32)lane * LDW, amb);
            dma16<0, 0, 4>(src, (u32)(lane + 64) * LDW, amb + 1024);
        }
        if (wave == 0 && lane < 32) ctl[lane] = lane >= C_FREE && (lane & 7) < 4 ? 4 : 0;
        WAIT_LGKM0();
        WG_BARRIER(call);
        scale = p.sx[0] * p.sw[0];
        asm volatile("" : "+v"(scale), "+v"(fpeer));
        call = call + 1;
        par = call & 1;
        WAIT_VM(0);
        u32 x[8];
        {
            u32x2 t0, t1, t2, t3;
            const u32 a = amb + (u32)(((lane >> 4) * 16 + ((lane & 15) >> 1)) * 16 + (lane & 1) * 8);
            asm volatile("ds_read_b64_tr_b8 %0, %4\n\tds_read_b64_tr_b8 %1, %4 offset:128\n\t"
                         "ds_read_b64_tr_b8 %2, %4 offset:1024\n\tds_read_b64_tr_b8 %3, %4 offset:1152\n\ts_waitcnt lgkmcnt(0)"
                         : "=&v"(t0), "=&v"(t1), "=&v"(t2), "=&v"(t3) : "v"(a) : "memory");
            x[0] = t0.x; x[1] = t0.y; x[2] = t1.x; x[3] = t1.y; x[4] = t2.x; x[5] = t2.y; x[6] = t3.x; x[7] = t3.y;
        }
        STAMP(1);
        const u32 sbuf = lbase + L_W + (NSW - 1) * BLK_B + wave * 1024;
        LDS_AT(u32x4, sbuf + lane * 16) = u32x4{0, 0, 0, 0};
        u32 sum = 0;
#pragma unroll
        for (int i = 0; i < 8; ++i) sum = __builtin_amdgcn_sad_u8(x[i] & 0x7F7F7F7Fu, 0u, sum);
        sum = rdl(wave_scan_incl(sum), 63);
        int T0 = ((int)sum - 60211) >> 13;
        T0 = T0 < 0 ? 0 : (T0 > 21 ? 21 : T0);
        const u32 Q = (u32)(121 - T0) * 0x01010101u, T4 = (u32)T0 * 0x04040404u;
        const u32 LC = (T4 + 0x24201C00u) | 0xFFu, LKlo = T4 + 0x08040028u, LKhi = T4 + 0x1814100Cu;
        u32 mk[8], Cw[2], esc[2], ne[2], bm = 0;
#pragma unroll
        for (int h = 0; h < 2; ++h) {
            u32 Cb = 0, sh = 0;
            Cw[h] = 0; esc[h] = 0;
#pragma unroll
            for (int i = 0; i < 4; ++i) {
                const int j = 4 * h + i;
                const u32 q = ((x[j] >> 2) & 0x1F1F1F1Fu) + Q;
                const u32 g7 = q & 0x80808080u, gm = (g7 >> 4) - (g7 >> 7);
                u32 c = (q + 0x01010101u) & gm & 0x03030303u, k = q & ~gm & 0x07070707u;
                const u32 e = ~(c | (c >> 1)) & 0x01010101u;
                const u32 t = e + (e << 8), v = t + (t << 16), u = v + Cb;
                if (i >= 2) { const u32 ov = e & ((u + 0x77777777u) >> 7); c |= ov; k &= ~((ov << 3) - ov); }
                const u32 four = (k >> 15) & 0xE00u & (u32)__builtin_amdgcn_sbfe((int)v, 26, 1);
                esc[h] |= __builtin_amdgcn_udot4(k, __builtin_amdgcn_perm(0u, 0x40080100u, v), four, false) << sh;
                sh += 3 * (v >> 24);
                Cb = __builtin_amdgcn_perm(u, u, 0x03030303u);
                Cw[h] |= c << (2 * i);
                const u32 pc = __builtin_amdgcn_perm(0u, LC, c);
                const u32 pk = __builtin_amdgcn_perm(LKhi, LKlo, k | (~pc & 0x80808080u));
                mk[j] = (x[j] ^ (pk & pc)) & 0x7C7C7C7Cu;
                const u32 nz = __builtin_amdgcn_udot4((mk[j] + 0x7C7C7C7Cu) & 0x80808080u, 0x08040201u, 0u, false);
                bm |= j < 2 ? nz >> (7 - 4 * j) : nz << (4 * j - 7);
            }
            ne[h] = min(sh, 24u);
        }
        const u32 cnt = __builtin_popcount(bm), nb = ne[0] + ne[1];
        const u32 both = wave_scan_incl(nb | (cnt << 16)), tot = rdl(both, 63);
        const u32 ndw = ((tot & 0xFFFFu) + 31) >> 5, nex = tot >> 16, nov = nex > 10 ? nex - 10 : 0;
        const u32 need = ndw + (nov ? (nov + 2) >> 1 : 0);
        const bool raw = need > 192;
        u32 start = 0;
        if (lane == 0) start = __hip_atomic_fetch_add((LAS u32*)(L + L_CTL) + C_ALLOC, raw ? 192u : need, __ATOMIC_RELAXED, __HIP_MEMORY_SCOPE_WORKGROUP);
        char* blk = p.ws + WST_OFF + par * WST_PAR + (size_t)(cbe * 4 + kse) * BLK_G;
        u32x4 P0; u32 P1;
        P0.x = (x[0] & 0x83838383u) | (__builtin_amdgcn_alignbit(x[1], x[1], 29) & 0x1C1C1C1Cu) | ((x[6] << 5) & 0x60606060u);
        P0.y = (x[2] & 0x83838383u) | (__builtin_amdgcn_alignbit(x[3], x[3], 29) & 0x1C1C1C1Cu) | ((x[6] >> 1) & 0x40404040u) | ((x[7] >> 2) & 0x20202020u);
        P0.z = (x[4] & 0x83838383u) | (__builtin_amdgcn_alignbit(x[5], x[5], 29) & 0x1C1C1C1Cu) | ((x[7] << 5) & 0x60606060u);
        P0.w = Cw[0]; P1 = Cw[1];
        if (__builtin_expect(raw, 0)) { P0 = u32x4{x[0], x[1], x[2], x[3]}; P1 = x[4]; }
        st16_sys(blk + wave * FR_B + lane * 16, P0);
        st4_sys(blk + wave * FR_B + 1024 + lane * 4, P1);
        start = rfl(start);
        char* pool = blk + POOL_B + start * 4;
        u32 ev = 0;
        if (__builtin_expect(raw, 0)) {
            st4_sys(pool + lane * 4, x[5]); st4_sys(pool + 256 + lane * 4, x[6]); st4_sys(pool + 512 + lane * 4, x[7]);
        } else {
            const u32 bp = (both & 0xFFFFu) - nb;
            const u64 V = (u64)esc[0] | ((u64)esc[1] << ne[0]), U = V << (bp & 31);
            LAS u32* sp = (LAS u32*)(size_t)(sbuf + (bp >> 5) * 4);
            __hip_atomic_fetch_or(sp, (u32)U, __ATOMIC_RELAXED, __HIP_MEMORY_SCOPE_WORKGROUP);
            __hip_atomic_fetch_or(sp + 1, (u32)(U >> 32), __ATOMIC_RELAXED, __HIP_MEMORY_SCOPE_WORKGROUP);
            __hip_atomic_fetch_or(sp + 2, ((u32)(V >> 32) >> 1) >> (31 - (bp & 31)), __ATOMIC_RELAXED, __HIP_MEMORY_SCOPE_WORKGROUP);
            for (u32 i = lane; i < ndw; i += 64) st4_sys(pool + i * 4, LDS_AT(u32, sbuf + i * 4));
            u32 idx = (both >> 16) - cnt;
            while (__any(bm != 0)) {
                const u32 pos = __builtin_ctz(bm | 0x80000000u);
                const bool b0 = pos & 4, b1 = pos & 8, b2 = pos & 16;
                const u32 m01 = b0 ? mk[1] : mk[0], m23 = b0 ? mk[3] : mk[2], m45 = b0 ? mk[5] : mk[4], m67 = b0 ? mk[7] : mk[6];
                const u32 m03 = b1 ? m23 : m01, m47 = b1 ? m67 : m45, m = b2 ? m47 : m03;
                const u32 en = (pos >> 4) * 1024 + lane * 16 + (pos & 15) + (((m >> (8 * (pos & 3) + 2)) & 31u) << 11);
                ev |= (u32)__builtin_amdgcn_ds_permute((int)((bm != 0 && idx < 10 ? idx + 2 : 63u) * 4), (int)en);
                if (__builtin_expect(bm != 0 && idx >= 10, 0)) st2_sys(pool + ndw * 4 + (idx - 9) * 2, en);
                ++idx; bm &= bm - 1;
            }
            if (nov && lane == 0) st2_sys(pool + ndw * 4, nov);
        }
        if (lane == 0) ev = (u32)T0 | (raw || nov ? 32u : 0u) | (raw ? 64u : 0u);
        if (lane == 1) ev = start;
        if (lane < 12) st2_sys(blk + MAIN_B + wave * ENT_B + lane * 2, ev);
        LDS_AT(u32x4, amb + lane * 16) = u32x4{0, 0, 0, 0};
        LDS_AT(u32x4, amb + 1024 + lane * 16) = u32x4{0, 0, 0, 0};
        WAIT_VM(0);
        u32 a2 = 0;
        if (lane == 0) a2 = __hip_atomic_fetch_add((LAS u32*)(L + L_CTL) + C_ARR, 1u, __ATOMIC_RELAXED, __HIP_MEMORY_SCOPE_WORKGROUP);
        if (rfl(a2) == 3) {
            const u32 n16 = MIN16 + ((ctl[C_ALLOC] + 3) >> 2);
            if (lane < NR && lane != rank) {
                GAS u64* f = (GAS u64*)(fpeer + WFL_OFF + par * WFL_PAR) + ((cbe * 8 + rank) * 4 + kse);
                __hip_atomic_store(f, ((u64)call << 32) | n16, __ATOMIC_RELAXED, __HIP_MEMORY_SCOPE_SYSTEM);
            }
            ctl[C_PUB] = 1;
        }
        STAMP(2);
        } else {
            LDS_AT(u32x4, amb + lane * 16) = u32x4{0, 0, 0, 0};
            LDS_AT(u32x4, amb + 1024 + lane * 16) = u32x4{0, 0, 0, 0};
            WG_BARRIER(call);
            call = call + 1;
            scale = p.sx[0] * p.sw[0];
            asm volatile("" : "+v"(scale));
        }

        const int fw = wave & 3, grp = wave >> 2;
        f32x4 acc[8];
#pragma unroll
        for (int f = 0; f < 8; ++f) acc[f] = f32x4{0, 0, 0, 0};
        int avw = 0, avx = 0;
        const u32 aq0 = lbase + L_W + fw * FR_B + lane * 16, aq1 = lbase + L_W + fw * FR_B + 1024 + lane * 4;
        const u32 aen = lbase + L_W + MAIN_B + fw * ENT_B + (lane < 12 ? lane : 0) * 2, axx = lbase + L_X + lane * 16;
        const u32 scr = lbase + L_CTL + 128 + lane;
        auto wait_for = [&](int i, int& av) {
            u64 t0 = 0;
            for (u32 it = 1;; ++it) {
                const u32x2 fl = LDS_AT(u32x2, lbase + L_CTL);
                avw = (int)rfl(fl.x) + 4; avx = (int)rfl(fl.y);
                if (i < av) break;
                if (__builtin_expect((it & 1023) == 0, 0)) {
                    const u64 t = rt();
                    if (t0 == 0) t0 = t;
                    if (t - t0 > 2 * DEADLINE) { ctl[C_FAIL] = 1; avw = avx = NSTEP; break; }
                }
                __builtin_amdgcn_s_sleep(1);
            }
            asm volatile("" ::: "memory");
        };
        auto mma8 = [&](int next, const i32x8& wf, const u32x4 (&xa)[16]) {
#pragma unroll
            for (int f = 0; f < 8; ++f) {
                const u32x4 a = xa[2 * f], b = xa[2 * f + 1];
                i32x8 xf;
                xf[0] = a.x; xf[1] = a.y; xf[2] = a.z; xf[3] = a.w; xf[4] = b.x; xf[5] = b.y; xf[6] = b.z; xf[7] = b.w;
                if (f == 1) {
                    asm volatile("" :: "v"(xa[15]), "v"(wf) : "memory");
                    ctl[C_FREE + wave] = next;
                    __builtin_amdgcn_sched_barrier(0);
                }
                acc[f] = __builtin_amdgcn_mfma_scale_f32_16x16x128_f8f6f4(wf, xf, acc[f], 1  , 0  , 0, 0, 0, 0);
            }
        };
        if (grp == 1)
#pragma nounroll
        for (int i = 0; i < 4; ++i) {
            if (i >= avx) wait_for(i, avx);
            u32x2 t0, t1, t2, t3;
            const u32 a = lbase + L_W + i * BLK_B + fw * 16 + (u32)(((lane >> 4) * 16 + ((lane & 15) >> 1)) * 64 + (lane & 1) * 8);
            asm volatile("ds_read_b64_tr_b8 %0, %4\n\tds_read_b64_tr_b8 %1, %4 offset:512\n\t"
                         "ds_read_b64_tr_b8 %2, %4 offset:4096\n\tds_read_b64_tr_b8 %3, %4 offset:4608\n\ts_waitcnt lgkmcnt(0)"
                         : "=&v"(t0), "=&v"(t1), "=&v"(t2), "=&v"(t3) : "v"(a) : "memory");
            u32x4 xa[16];
#pragma unroll
            for (int f = 0; f < 16; ++f) xa[f] = LDS_AT(u32x4, axx + i * XS_B + f * 1024);
            ctl[C_FREX + wave] = i < 3 ? i + 1 : 5;
            __builtin_amdgcn_sched_barrier(0);
            mma8(i < 3 ? i + 1 : 5, i32x8{(int)t0.x, (int)t0.y, (int)t1.x, (int)t1.y, (int)t2.x, (int)t2.y, (int)t3.x, (int)t3.y}, xa);
        }
        u32 xo = grp * XS_B, wo = (4 + grp) * BLK_B;
#pragma nounroll
        for (int i = 4 + grp; i < NSTEP; i += 2) {
            if (i >= avx) wait_for(i, avx);
            u32x4 xa[16];
#pragma unroll
            for (int f = 0; f < 16; ++f) xa[f] = LDS_AT(u32x4, axx + xo + f * 1024);
            ctl[C_FREX + wave] = i + 2;
            if (i >= avw) { __builtin_amdgcn_s_setprio(0); wait_for(i, avw); __builtin_amdgcn_s_setprio(PRIO_C); }
            const u32 e0 = LDS_AT(unsigned short, aen + wo);
            const u32x4 q0 = LDS_AT(u32x4, aq0 + wo);
            const u32 q1 = LDS_AT(u32, aq1 + wo);
            __builtin_amdgcn_sched_barrier(0);
            const u32 hdr = rfl(e0), pool = lbase + L_W + POOL_B + wo + (rdl(e0, 1) & 1023u) * 4;
            const u32 ea = e0 >> 11 ? amb + (e0 & 2047u) : scr;
            LDS_AT(unsigned char, ea) = (unsigned char)(e0 >> 11 << 2);
            const u32 n0 = __builtin_popcount(~(q0.w | (q0.w >> 1)) & 0x55555555u), nn = n0 + __builtin_popcount(~(q1 | (q1 >> 1)) & 0x55555555u);
            const u32 incl = wave_scan_incl(nn), bp = 3 * (incl - nn);
            const u32 sa = pool + (bp >> 5) * 4;
            const u32 d0 = LDS_AT(u32, sa), d1 = LDS_AT(u32, sa + 4), d2 = LDS_AT(u32, sa + 8);
            if (__builtin_expect((hdr & 96u) == 32u, 0)) {
                const u32 ov = pool + ((3 * rdl(incl, 63) + 31) >> 5) * 4;
                const u32 n = rfl(LDS_AT(unsigned short, ov)) & 511u;
                for (u32 k = 0; k < n; k += 64) {
                    const u32 e = LDS_AT(unsigned short, ov + 2 + (k + lane) * 2);
                    if (k + lane < n) LDS_AT(unsigned char, amb + (e & 2047u)) = (unsigned char)(e >> 11 << 2);
                }
                __builtin_amdgcn_s_waitcnt(0xC07F);
            }
            const u32x4 m0 = LDS_AT(u32x4, amb + lane * 16);
            const u32x4 m1 = LDS_AT(u32x4, amb + 1024 + lane * 16);
            LDS_AT(unsigned char, ea) = 0;
            if (__builtin_expect(hdr & 32u, 0)) { LDS_AT(u32x4, amb + lane * 16) = u32x4{0, 0, 0, 0}; LDS_AT(u32x4, amb + 1024 + lane * 16) = u32x4{0, 0, 0, 0}; }
            __builtin_amdgcn_sched_barrier(0);
            const u32 T4 = (hdr & 31u) * 0x04040404u;
            const u32 LC = (T4 + 0x24201C00u) | 0xFFu, LKlo = T4 + 0x08040028u, LKhi = T4 + 0x1814100Cu;
            u32 pc[8], sel[8];
#pragma unroll
            for (int h = 0; h < 2; ++h) {
                const u32 C = h ? q1 : q0.w;
                u32 Cb = 0x07070707u;
#pragma unroll
                for (int i = 0; i < 4; ++i) {
                    const int j = 4 * h + i;
                    pc[j] = __builtin_amdgcn_perm(0u, LC, (C >> (2 * i)) & 0x03030303u);
                    const u32 e = pc[j] & 0x01010101u, t = e + (e << 8), u = (t << 16) + (t + Cb);
                    sel[j] = __builtin_amdgcn_bitop3_b32(u, pc[j], 0x07070707u, 0xB1);
                    Cb = __builtin_amdgcn_perm(u, u, 0x03030303u);
                }
            }
            const u32 w0 = __builtin_amdgcn_alignbit(d1, d0, bp), w1 = __builtin_amdgcn_alignbit(d2, d1, bp);
            i32x8 wf;
#pragma unroll
            for (int h = 0; h < 2; ++h) {
                const u32 w = h ? __builtin_amdgcn_alignbit(w1, w0, 3 * n0) : w0;
                const u32 ya = (w & 0x3Fu) | ((w << 10) & 0x3F0000u), yb = ((w >> 12) & 0x3Fu) | ((w >> 2) & 0x3F0000u);
                const u32 Ka = __builtin_amdgcn_perm(LKhi, LKlo, (ya | (ya << 5)) & 0x07070707u);
                const u32 Kb = __builtin_amdgcn_perm(LKhi, LKlo, (yb | (yb << 5)) & 0x07070707u);
                const u32x4 m = h ? m1 : m0;
#pragma unroll
                for (int i = 0; i < 4; ++i) wf[4 * h + i] = (__builtin_amdgcn_perm(Kb, Ka, sel[4 * h + i]) & pc[4 * h + i]) ^ m[i];
            }
            wf[0] |= q0.x & 0x83838383u; wf[1] |= __builtin_amdgcn_alignbit(q0.x, q0.x, 3) & 0x83838383u;
            wf[2] |= q0.y & 0x83838383u; wf[3] |= __builtin_amdgcn_alignbit(q0.y, q0.y, 3) & 0x83838383u;
            wf[4] |= q0.z & 0x83838383u; wf[5] |= __builtin_amdgcn_alignbit(q0.z, q0.z, 3) & 0x83838383u;
            wf[6] |= ((q0.x >> 5) & 0x03030303u) | ((q0.y << 1) & 0x80808080u);
            wf[7] |= ((q0.z >> 5) & 0x03030303u) | ((q0.y << 2) & 0x80808080u);
            if (__builtin_expect(hdr & 64u, 0)) {
                wf[0] = q0.x; wf[1] = q0.y; wf[2] = q0.z; wf[3] = q0.w; wf[4] = q1;
                wf[5] = LDS_AT(u32, pool + lane * 4); wf[6] = LDS_AT(u32, pool + 256 + lane * 4); wf[7] = LDS_AT(u32, pool + 512 + lane * 4);
                __builtin_amdgcn_s_waitcnt(0xC07F);
            }
            __builtin_amdgcn_sched_barrier(0);
            if (i >= NSTEP - 2)
                for (u32 it = 0; min_free() < NSTEP - 2 && it < (1u << 26); ++it) __builtin_amdgcn_s_sleep(0);
            mma8(i + 2, wf, xa);
            xo ^= 2 * XS_B;
            wo = wo + 2 * BLK_B >= NSW * BLK_B ? wo + 2 * BLK_B - NSW * BLK_B : wo + 2 * BLK_B;
        }
        STAMP(5);
        constexpr u32 T_0 = L_X, T_1 = L_W + 5 * BLK_B;
        const u32 tw = lbase + (grp ? T_1 : T_0) + (lane & 15) * 256 + (((fw * 4 + (lane >> 4)) ^ (lane & 15)) * 16);
#pragma unroll
        for (int f = 0; f < 8; ++f) LDS_AT(f32x4, tw + f * 4096) = acc[f];
        WAIT_LGKM0();
        ctl[C_FREE + wave] = 2 * NSTEP;
        if (wave >= 4) STAMP(2);
        __builtin_amdgcn_s_setprio(0);
        for (u32 it = 0; min_free() < 2 * NSTEP && it < (1u << 24); ++it) __builtin_amdgcn_s_sleep(ESLEEP);
        if (wave >= 4) STAMP(3);
        const float sc = ctl[C_FAIL] != 0 ? __builtin_nanf("") : scale;
        float* o = p.out + (size_t)(rb * 64 + wave * 16 + (lane >> 4)) * LDO + cb * 64 + (lane & 15) * 4;
        f32x4 y0[4], y1[4];
#pragma unroll
        for (int f = 0; f < 4; ++f) {
            const u32 tr = lbase + (wave * 16 + f * 4 + (lane >> 4)) * 256 + (((lane & 15) ^ (f * 4 + (lane >> 4))) * 16);
            y0[f] = LDS_AT(f32x4, tr + T_0); y1[f] = LDS_AT(f32x4, tr + T_1);
        }
#pragma unroll
        for (int f = 0; f < 4; ++f)
            st16_sys(o + (size_t)f * 4 * LDO, __builtin_bit_cast(u32x4, __builtin_elementwise_maximum((y0[f] + y1[f]) * sc, f32x4{0, 0, 0, 0})));
        STAMP(6);
    } else if (wave == 8) {
        WG_BARRIER(call);
        call = call + 1;
        par = call & 1;
        const int R = lane < NREM ? lane : 0, src = src_of_rem(R, rank, cb), ks = R / 7;
        const u64 baddr = (((u64)(u32)__shfl((int)((u64)fpeer >> 32), src) << 32) | (u32)__shfl((int)(u32)(u64)fpeer, src)) + WST_OFF + par * WST_PAR + (u64)(cb * 4 + ks) * BLK_G;
        const u32 blo = (u32)baddr, bhi = (u32)(baddr >> 32);
        GAS u64* fl = (GAS u64*)(p.ws + WFL_OFF + par * WFL_PAR) + ((cb * 8 + src) * 4 + ks);
        const u32 voff = lane * 16;
        u32 n16v = MIN16;
        bool dead = false;
        const u32 xcc = __builtin_amdgcn_s_getreg((3 << 11) | 20) & 15;
        GAS u64* inv = (GAS u64*)(p.ws + INV_OFF + xcc * 256);
        u32 prev = 0;
        if (lane == 0) prev = __hip_atomic_fetch_max((GAS u32*)(inv + 16), call, __ATOMIC_RELAXED, __HIP_MEMORY_SCOPE_AGENT);
        if (rfl(prev) < call) {
            __builtin_amdgcn_fence(__ATOMIC_ACQUIRE, "");
            WAIT_VM(0);
            if (lane == 0) __hip_atomic_store(inv, (u64)call << 32, __ATOMIC_RELAXED, __HIP_MEMORY_SCOPE_AGENT);
        }
        STAMP(3);
        if (lane >= 28 && lane < 56) fl = (GAS u64*)(p.ws + XFL_OFF + par * XFL_PAR) + ((rb * 8 + src_of_rem(lane, rank, 0)) * 4 + (lane - 28) / 7);
        if (lane == 62) fl = (GAS u64*)(p.ws + TMO_OFF);
        if (lane == 63) fl = inv;
        {
            const u64 t0 = rt();
            for (u32 it = 1;; ++it) {
                const u64 v = __hip_atomic_load(fl, __ATOMIC_RELAXED, __HIP_MEMORY_SCOPE_SYSTEM);
                n16v = (u32)v;
                const u64 up = __ballot((u32)(v >> 32) >= call);
                if (((up >> 62) & 1) && rdl((u32)(v >> 32), 62) == call) { dead = true; break; }
                if ((up | 0x7FFFFFFFF0000000ull) == ~0ull && (__builtin_popcount((u32)(up >> 28) & 0xFFFFFFFu) >= XTH || rt() - t0 > 2000)) break;
                if ((it & 15) == 0 && expired(t0)) { dead = true; break; }
                __builtin_amdgcn_s_sleep(1);
            }
            n16v = n16v < MIN16 ? MIN16 : (n16v > MAX16 ? MAX16 : n16v);
        }
        STAMP(1);
        for (u32 it = 0; rfl(ctl[C_PUB]) == 0 && it < (1u << 24); ++it) __builtin_amdgcn_s_sleep(1);
        STAMP(4);
        __builtin_amdgcn_s_setprio(PRIO_W);
        int pub = 0, freed = 0;
        u32 lo = lbase + L_W + 4 * BLK_B;
        auto publish = [&](int upto) { if (upto > pub) { pub = upto; ctl[C_FULLW] = upto; } };
        u64 g[LAST + 1]; u32 l[LAST + 1];
#pragma unroll
        for (int j = 0; j <= LAST; ++j) { g[j] = 0; l[j] = 0; }
#define W_LOAD(OFF, J, K, NOP) do { dma16<OFF, 0, NOP>(g[J], voff, l[J]); WAIT_VM(WIN); after(std::integral_constant<int, K>{}); } while (0)
        auto rest_of_slot = [&](int n, bool mid, auto&& after) {
            if (mid) { W_LOAD(2048, LAG, 3, 0); W_LOAD(3072, LAG, 4, 0); }
            if (n >= LAST) {
                g[LAST] += 4096; l[LAST] += 4096;
                W_LOAD(0, LAST, 5, 0);
                const u32 n16 = rdl(n16v, n - LAST);
                if (__builtin_expect(n16 > 64, 0))
                    for (u32 i = 64; i < n16; i += 64) { if (lane + i < n16) dma16<1024, 0>(g[LAST] + i * 16, voff, l[LAST] + i * 16); WAIT_VM(WIN); }
                if ((u32)lane < n16) dma16<1024, 0>(g[LAST], voff, l[LAST]);
                WAIT_VM(WIN); after(std::integral_constant<int, SL>{});
            }
#pragma unroll
            for (int j = LAST; j > 0; --j) { g[j] = g[j - 1]; l[j] = l[j - 1]; }
        };
        auto open = [&](int n, int done) {
            if (freed <= n + 4 - NSW) {
                const int need = n + 4 - NSW;
                const u64 t0 = rt();
                bool drained = false;
                for (;;) {
                    freed = min_free();
                    if (freed > need) break;
                    if (!drained) { WAIT_VM(0); publish(done); drained = true; continue; }
                    if (rt() - t0 > 2 * DEADLINE) { dead = true; break; }
                    __builtin_amdgcn_s_sleep(1);
                }
                if (dead) return;
            }
            g[0] = ((u64)rdl(bhi, n) << 32) | rdl(blo, n);
            l[0] = lo;
            lo = lo + BLK_B == lbase + L_W + NSW * BLK_B ? lbase + L_W : lo + BLK_B;
        };
        for (int n = 0; n < NB; ++n) {
            constexpr int PK = (WIN - 1) % SL + 1, PL = LAST + 1 + (WIN - PK) / SL;
            auto after = [&](auto k) { if constexpr (decltype(k)::value == PK) publish(n - PL + 1); };
            if (!dead) open(n, n - LAST);
            if (dead) break;
            W_LOAD(0, 0, 1, 4); W_LOAD(1024, 0, 2, 0);
            rest_of_slot(n, n >= LAG, after);
        }
        if (!dead) {
            static_for<LAST>([&](auto dd) {
                constexpr int d = decltype(dd)::value;
                rest_of_slot(NB + d, d < LAG, [&](auto k) { publish(std::integral_constant<int, w_landed(w_cum(d) - (SL - decltype(k)::value), WIN)>::value); });
            });
            static_for<NU>([&](auto uu) {
                constexpr int u = decltype(uu)::value;
                auto after = [&](auto k) { publish(std::integral_constant<int, w_landed(w_cum(LAST + u) - (SL - decltype(k)::value), WIN)>::value); };
                if (!dead) open(NB + u, NB + u);
                if (dead) return;
                W_LOAD(0, 0, 1, 4); W_LOAD(1024, 0, 2, 0);
                g[LAG] = g[LAST] = g[0]; l[LAG] = l[LAST] = l[0];
                rest_of_slot(NB + u + LAST, true, after);
            });
        }
        STAMP(5);
        if (dead) { ctl[C_FAIL] = 1; WAIT_VM(0); ctl[C_FULLW] = NREM; }
        else
            static_for<8>([&](auto dd) {
                constexpr int upto = NREM - 7 + decltype(dd)::value, behind = w_behind(upto - 1, w_cum(LAST + NU - 1));
                if constexpr (behind < WIN) { WAIT_VM(behind); publish(upto); }
            });
        STAMP(6);
    } else if (wave == 9) {
        WG_BARRIER(call);
        call = call + 1;
        par = call & 1;
        const int R = max((lane & 31) - 4, 0), src = src_of_rem(R, rank, cb), ks = R / 7;
        const u32 toff = (u32)((src * 4 + ks) * 8 + rb) * XT_B;
        const u64 xbase = (u64)p.ws + XRV_OFF + par * XRV_PAR;
        GAS u64* fl = (GAS u64*)(p.ws + XFL_OFF + par * XFL_PAR) + (((rb + (lane >> 5)) * 8 + src) * 4 + ks);
        constexpr u32 OWN = 0xFu;
        const u32 voff = lane * 16;
        const u32 xoff = (u32)(lane & 15) * LDX + (lane >> 4) * 16;
        u32 ready = OWN;
        u32 it = 0, la = lbase + L_X;
        bool dead = false;
        const u64 t0 = rt();
        int pub = 0, freed = 0;
        for (u32 k = 0; rfl(ctl[C_PUB]) == 0 && k < (1u << 22); ++k) __builtin_amdgcn_s_sleep(4);
        for (int s = 0; s < NSTEP && !dead; ++s) {
            if (!((ready >> s) & 1)) {
                WAIT_VM(0);
                if (pub < s) { pub = s; ctl[C_FULLX] = s; }
                for (;;) {
                    const u64 v = __hip_atomic_load(fl, __ATOMIC_RELAXED, __HIP_MEMORY_SCOPE_SYSTEM);
                    const u64 ok = __ballot((u32)(v >> 32) >= call);
                    ready = ((u32)ok & (u32)(ok >> 32)) | OWN;
                    if (ready == ~0u) { dead = rfl(__hip_atomic_load(tmo, __ATOMIC_RELAXED, __HIP_MEMORY_SCOPE_AGENT)) == call; break; }
                    if ((ready >> s) & 1) break;
                    if ((++it & 15) == 0 && expired(t0)) { dead = true; break; }
                    __builtin_amdgcn_s_sleep(2);
                }
            }
            while (!dead && freed <= s - NSX) {
                freed = min_free(C_FREX);
                if (freed > s - NSX) break;
                if ((++it & 1023) == 0 && rt() - t0 > 2 * DEADLINE) dead = true;
                __builtin_amdgcn_s_sleep(1);
            }
            if (dead) break;
            if (s < 4) {
                const u64 b = (u64)p.x + (u64)(rank * 512 + rb * 64) * LDX + s * 128;
#pragma unroll
                for (int i = 0; i < 16; ++i) dma16<0, 0>(b + (i >> 1) * 16 * LDX + (i & 1) * 64, xoff, la + i * 1024);
#pragma unroll
                for (int i = 0; i < 8; ++i)
                    dma16<0, 0>((u64)p.w + (u64)(s * 128 + i * 16) * LDW + cb * 64, (u32)(lane >> 2) * LDW + (lane & 3) * 16, lbase + L_W + s * BLK_B + i * 1024);
            } else {
                const u64 b = xbase + rdl(toff, s);
#pragma unroll
                for (int i = 0; i < 4; ++i) {
                    dma16<0, 1, 4>(b + i * 4096, voff, la + i * 4096); dma16<1024, 1>(b + i * 4096, voff, la + i * 4096);
                    dma16<2048, 1>(b + i * 4096, voff, la + i * 4096); dma16<3072, 1>(b + i * 4096, voff, la + i * 4096);
                }
            }
            la = la + XS_B == lbase + L_X + NSX * XS_B ? lbase + L_X : la + XS_B;
            WAIT_VM(16 * (XD - 1));
            if (pub < s - XD + 2) { pub = s - XD + 2; ctl[C_FULLX] = pub; }
        }
        if (dead) { ctl[C_FAIL] = 1; WAIT_VM(0); }
        else {
            WAIT_VM(32); ctl[C_FULLX] = max(pub, NSTEP - 2);
            WAIT_VM(16); ctl[C_FULLX] = max(pub, NSTEP - 1);
            WAIT_VM(0);
        }
        ctl[C_FULLX] = NSTEP;
        STAMP(6);
    } else {
        const int id = bid / 7, d = (rank + 1 + bid % 7) & 7;
        u32x4 v[8];
        if (id < 32) {
            const unsigned char* g = p.x + (size_t)(d * 512 + (id & 7) * 64 + (lane & 15)) * LDX + (id >> 3) * 128 + (lane >> 4) * 16;
#pragma unroll
            for (int i = 0; i < 8; ++i) v[i] = *(const GAS u32x4*)(g + (size_t)(i >> 1) * 16 * LDX + (i & 1) * 64);
        }
        WG_BARRIER(call);
        call = call + 1;
        par = call & 1;
        if (id < 32) {
            asm volatile("" : "+v"(v[0]), "+v"(v[1]), "+v"(v[2]), "+v"(v[3]), "+v"(v[4]), "+v"(v[5]), "+v"(v[6]), "+v"(v[7]), "+v"(fpeer));
            char* peer = (char*)(((u64)rdl((u32)((u64)fpeer >> 32), d) << 32) | rdl((u32)(u64)fpeer, d));
            char* dst = peer + XRV_OFF + par * XRV_PAR + (size_t)(rank * 32 + id) * XT_B + lane * 16;
#pragma unroll
            for (int i = 0; i < 8; ++i) { st16_sys(dst + i * 1024, v[i]); WAIT_VM(PUSH_W - 1); }
            STAMP(1);
            WAIT_VM(0);
            if (lane == 0)
                __hip_atomic_store((GAS u64*)(peer + XFL_OFF + par * XFL_PAR) + (((id & 7) * 8 + rank) * 4 + (id >> 3)), (u64)call << 32,
                                   __ATOMIC_RELAXED, __HIP_MEMORY_SCOPE_SYSTEM);
            STAMP(2);
        }
    }
    if (lane == 0) __hip_atomic_store(cntp, call, __ATOMIC_RELAXED, __HIP_MEMORY_SCOPE_AGENT);
}

extern "C" void kernel_launch(int rank, int nranks, ncclComm_t comm,
                              hipStream_t stream,
                              void* const* d_in, const int* in_sizes, int n_in,
                              void* d_out, int out_size,
                              const int* in_dtypes, int out_dtype,
                              void* const* d_out_all,
                              void* const* d_ws_all, size_t ws_bytes) {
    (void)comm; (void)in_dtypes; (void)out_dtype; (void)d_out_all;
    if (nranks != NR || (unsigned)rank >= (unsigned)NR || n_in != 4 || ws_bytes < WS_NEED ||
        in_sizes[0] != 4096 * 512 || in_sizes[1] != 512 * 4096 || out_size != 512 * 4096) {
        fprintf(stderr, "unexpected problem shape\n");
        return;
    }
    HIP_CHECK(hipFuncSetAttribute((const void*)gemm_xchg, hipFuncAttributeMaxDynamicSharedMemorySize, LDS_BYTES));
    Params p;
    p.x = (const unsigned char*)d_in[0];
    p.w = (const unsigned char*)d_in[1];
    p.sx = (const float*)d_in[2];
    p.sw = (const float*)d_in[3];
    p.out = (float*)d_out;
    p.ws_all = d_ws_all;
    p.ws = (char*)d_ws_all[rank];
    p.rank = rank;
    p.pad = 0;
    hipLaunchKernelGGL(gemm_xchg, dim3(NWG), dim3(NTHR), LDS_BYTES, stream, p);
    HIP_CHECK(hipGetLastError());
}
```

```cpp
#include <hip/hip_runtime.h>
#include <rccl/rccl.h>
#include <cstdio>

#define HIP_CHECK(expr) do { hipError_t _e = (expr); if (_e != hipSuccess) { \
    fprintf(stderr, "HIP error %s at %s:%d\n", hipGetErrorString(_e), __FILE__, __LINE__); \
    return; } } while (0)

typedef unsigned u32;
typedef unsigned long long u64;
typedef unsigned u32x4 __attribute__((ext_vector_type(4)));
typedef unsigned u32x2 __attribute__((ext_vector_type(2)));
typedef int i32x8 __attribute__((ext_vector_type(8)));
typedef float f32x4 __attribute__((ext_vector_type(4)));
#define GAS __attribute__((address_space(1)))
#define LAS __attribute__((address_space(3)))

#define WIN 4
#define NU 10
#define ESLEEP 1
#define PRIO_C 1
#define NG 2
#define PRIO_W 3
#define XTH 5
#define PUSH_W 2
#define XD 2
#define NSW 9
#define NSX 4

constexpr int NR = 8;
constexpr int NWG = 256, NTHR = 64 * (4 * NG + 3);
constexpr int LDX = 512, LDW = 4096, LDO = 4096;
constexpr int MAIN_B = 5120, FR_B = 1280, ENT_B = 32, NENT = 6, POOL_B = MAIN_B + 4 * ENT_B;
constexpr int MIN16 = 4 * ENT_B / 16, MAX16 = (4 * ENT_B + 4 * 768) / 16;
constexpr int BLK_B = 8448;
constexpr int BLK_G = 8448;
constexpr int XT_B = 8192;
constexpr int XS_B = 2 * XT_B;
constexpr int NSTEP = 32, NREM = 28;
static_assert(MAIN_B + MAX16 * 16 + 16 <= BLK_B && BLK_B <= BLK_G, "block");

constexpr size_t WST_OFF = 0, WST_PAR = 4u << 20;
constexpr size_t XRV_OFF = 8u << 20, XRV_PAR = 2u << 20;
constexpr size_t WFL_OFF = 12u << 20, WFL_PAR = 16u << 10;
constexpr size_t XFL_OFF = WFL_OFF + (32u << 10), XFL_PAR = 4u << 10;
constexpr size_t CNT_OFF = WFL_OFF + (64u << 10);
constexpr size_t TMO_OFF = WFL_OFF + (128u << 10);
constexpr size_t INV_OFF = TMO_OFF + 1024;
constexpr size_t DBG_OFF = WFL_OFF + (256u << 10);
constexpr size_t WS_NEED = 13u << 20;
static_assert((size_t)256 * BLK_G <= WST_PAR, "w staging");

constexpr int L_W = 0;
constexpr int L_X = L_W + NSW * BLK_B;
constexpr int L_MASK = L_X + NSX * XS_B;
constexpr int L_CTL = L_MASK + 16384;
constexpr int LDS_BYTES = L_CTL + 128;
static_assert(NSW == 9 && NSX == 4, "the output staging areas are chosen for these rings");
static_assert(LDS_BYTES <= 160 * 1024 && NSW >= 8 && XD <= NSX && XD <= 4, "LDS");
constexpr int C_FULLW = 0, C_FULLX = 1, C_FAIL = 2, C_ALLOC = 3, C_ARR = 4, C_PUB = 5, C_FREE = 8  ;

constexpr u64 DEADLINE = 30000000ull;

struct Params {
    const unsigned char* x;
    const unsigned char* w;
    const float* sx;
    const float* sw;
    float* out;
    void* const* ws_all;
    char* ws;
    int rank;
    int pad;
};

__device__ __forceinline__ u64 rt() { return __builtin_amdgcn_s_memrealtime(); }
__device__ __forceinline__ u32 rfl(u32 v) { return (u32)__builtin_amdgcn_readfirstlane(v); }
__device__ __forceinline__ u32 rdl(u32 v, int l) { return (u32)__builtin_amdgcn_readlane(v, l); }
#define WAIT_VM(n) asm volatile("s_waitcnt vmcnt(%0)" :: "i"(n) : "memory")
#define WG_BARRIER(call) do { u32 c_; asm volatile("s_barrier" ::: "memory"); \
    asm volatile("v_readfirstlane_b32 %0, %1\n\ts_nop 4" : "=s"(c_) : "v"(call) : "memory"); call = c_; } while (0)
#define WAIT_LGKM0() asm volatile("s_waitcnt lgkmcnt(0)" ::: "memory")

template <int OFF, int POL, int NOP = 0>
__device__ __forceinline__ void dma16(u64 base, u32 voff, u32 ldsaddr) {
    if constexpr (POL == 1)
        asm volatile("s_mov_b32 m0, %2\n\ts_nop %4\n\tglobal_load_lds_dwordx4 %0, %1 offset:%3 sc0 sc1"
                     :: "v"(voff), "s"(base), "s"(ldsaddr), "i"(OFF), "i"(NOP) : "memory");
    else if constexpr (POL == 2)
        asm volatile("s_mov_b32 m0, %2\n\ts_nop %4\n\tglobal_load_lds_dwordx4 %0, %1 offset:%3 nt"
                     :: "v"(voff), "s"(base), "s"(ldsaddr), "i"(OFF), "i"(NOP) : "memory");
    else
        asm volatile("s_mov_b32 m0, %2\n\ts_nop %4\n\tglobal_load_lds_dwordx4 %0, %1 offset:%3"
                     :: "v"(voff), "s"(base), "s"(ldsaddr), "i"(OFF), "i"(NOP) : "memory");
}
__device__ __forceinline__ void st4_sys(void* p, u32 v) {
    asm volatile("global_store_dword %0, %1, off sc0 sc1\n\ts_nop 1" :: "v"(p), "v"(v) : "memory");
}
__device__ __forceinline__ void st2_sys(void* p, u32 v) {
    asm volatile("global_store_short %0, %1, off sc0 sc1\n\ts_nop 1" :: "v"(p), "v"(v) : "memory");
}
__device__ __forceinline__ void st16_sys(void* p, u32x4 v) {
    asm volatile("global_store_dwordx4 %0, %1, off sc0 sc1\n\ts_nop 1" :: "v"(p), "v"(v) : "memory");
}
__device__ __forceinline__ void st8_sys(void* p, u32x2 v) {
    asm volatile("global_store_dwordx2 %0, %1, off sc0 sc1\n\ts_nop 1" :: "v"(p), "v"(v) : "memory");
}
__device__ __forceinline__ u32 wave_scan_incl(u32 v) {
    v += __builtin_amdgcn_update_dpp(0, v, 0x111, 0xf, 0xf, true);
    v += __builtin_amdgcn_update_dpp(0, v, 0x112, 0xf, 0xf, true);
    v += __builtin_amdgcn_update_dpp(0, v, 0x114, 0xf, 0xf, true);
    v += __builtin_amdgcn_update_dpp(0, v, 0x118, 0xf, 0xf, true);
    v += __builtin_amdgcn_update_dpp(0, v, 0x142, 0xa, 0xf, false);
    v += __builtin_amdgcn_update_dpp(0, v, 0x143, 0xc, 0xf, false);
    return v;
}

__device__ __forceinline__ int src_of_rem(int R, int rank, int cb) { return (rank + 1 + (cb + R) % 7) & 7; }

constexpr int LAG = 2, LAST = 2 * LAG, SL = 6, NB = NREM - NU;
static_assert(WIN >= 1 && WIN <= 18 && NSW > LAST + 2 && NU >= 1 && NB >= LAST, "window");
constexpr int w_cum(int d) { int c = 0; for (int t = 0; t <= d; ++t) c += t >= LAST ? SL : (t < LAG ? 4 : 2); return c; }
constexpr int w_behind(int b, int q) { return b >= NB - LAST ? q - w_cum(b - (NB - LAST)) : q + SL * (NB - LAST - 1 - b); }
constexpr int w_landed(int q, int win) { int u = 0; while (u < NREM && w_behind(u, q) >= win) ++u; return u; }
template <int N, int I = 0, class F> __device__ __forceinline__ void static_for(F&& f) {
    if constexpr (I < N) { f(std::integral_constant<int, I>{}); static_for<N, I + 1>(f); }
}

#define STAMP(i) do {} while (0)
#define LDS_AT(T, a) (*(volatile LAS T*)(size_t)(a))

__global__ void __launch_bounds__(NTHR, 3) gemm_xchg(Params p) {
    extern __shared__ __attribute__((aligned(16))) unsigned char lds_raw[];
    LAS unsigned char* L = (LAS unsigned char*)lds_raw;
    const u32 lbase = (u32)(size_t)L;
    volatile LAS u32* ctl = (volatile LAS u32*)(L + L_CTL);
    const int tid = threadIdx.x, lane = tid & 63;
    const int wave = rfl(tid >> 6) == 0 ? 10 : (rfl(tid >> 6) <= 4 * NG ? rfl(tid >> 6) - 1 : rfl(tid >> 6) - 4 * NG + 7);
    const int bid = blockIdx.x, xc = bid & 7, jj = bid >> 3;
    const int rank = p.rank;
    const int rb = (jj & 3) * 2, cb = xc * 8 + (jj >> 2);
    GAS u32* tmo = (GAS u32*)(p.ws + TMO_OFF) + 1;
    STAMP(0);

    GAS u32* cntp = (GAS u32*)(p.ws + CNT_OFF) + bid * 16 + wave;
    u32 call = __hip_atomic_load(cntp, __ATOMIC_RELAXED, __HIP_MEMORY_SCOPE_AGENT);
    char* fpeer = (char*)p.ws_all[lane & 7];
    u32 par;
    auto expired = [&](u64 t0) {
        if (rfl(__hip_atomic_load(tmo, __ATOMIC_RELAXED, __HIP_MEMORY_SCOPE_AGENT)) != call && rt() - t0 <= DEADLINE) return false;
        __hip_atomic_store(tmo, call, __ATOMIC_RELAXED, __HIP_MEMORY_SCOPE_AGENT);
        return true;
    };
    auto min_free = [&]() {
        const u32x4 a = LDS_AT(u32x4, lbase + L_CTL + C_FREE * 4), b = LDS_AT(u32x4, lbase + L_CTL + C_FREE * 4 + 16);
        return (int)rfl(min(min(min(a.x, a.y), min(a.z, a.w)), min(min(b.x, b.y), min(b.z, b.w))));
    };

    if (wave < 8) {
        const u32 amb = lbase + L_MASK + wave * 2048;
        float scale;
        if (wave < 4) {
        const int cbe = (jj & 1) + 2 * (xc + 8 * ((jj >> 1) & 3)), kse = jj >> 3;
        {
            const u64 src = (u64)p.w + (u64)(kse * 128) * LDW + cbe * 64 + wave * 16;
            dma16<0, 0, 4>(src, (u32)lane * LDW, amb);
            dma16<0, 0, 4>(src, (u32)(lane + 64) * LDW, amb + 1024);
        }
        if (wave == 0 && lane < 16) ctl[lane] = NG == 2 ? (lane >= C_FREE && lane < C_FREE + 4 ? 4 : 0) : (lane >= C_FREE + 4 ? 1000 : 0);
        WAIT_LGKM0();
        WG_BARRIER(call);
        scale = p.sx[0] * p.sw[0];
        asm volatile("" : "+v"(scale), "+v"(fpeer));
        call = call + 1;
        par = call & 1;
        WAIT_VM(0);
        u32 x[8];
        {
            u32x2 t0, t1, t2, t3;
            const u32 a = amb + (u32)(((lane >> 4) * 16 + ((lane & 15) >> 1)) * 16 + (lane & 1) * 8);
            asm volatile("ds_read_b64_tr_b8 %0, %4\n\tds_read_b64_tr_b8 %1, %4 offset:128\n\t"
                         "ds_read_b64_tr_b8 %2, %4 offset:1024\n\tds_read_b64_tr_b8 %3, %4 offset:1152\n\ts_waitcnt lgkmcnt(0)"
                         : "=&v"(t0), "=&v"(t1), "=&v"(t2), "=&v"(t3) : "v"(a) : "memory");
            x[0] = t0.x; x[1] = t0.y; x[2] = t1.x; x[3] = t1.y; x[4] = t2.x; x[5] = t2.y; x[6] = t3.x; x[7] = t3.y;
        }
        STAMP(1);
        const u32 sbuf = lbase + L_W + (NSW - 1) * BLK_B + wave * 1024;
        LDS_AT(u32x4, sbuf + lane * 16) = u32x4{0, 0, 0, 0};
        u32 sum = 0;
#pragma unroll
        for (int i = 0; i < 8; ++i) sum = __builtin_amdgcn_sad_u8(x[i] & 0x7F7F7F7Fu, 0u, sum);
        sum = rdl(wave_scan_incl(sum), 63);
        int T0 = ((int)sum - 60211) >> 13;
        T0 = T0 < 0 ? 0 : (T0 > 21 ? 21 : T0);
        const u32 Q = (u32)(121 - T0) * 0x01010101u, T4 = (u32)T0 * 0x04040404u;
        const u32 LC = (T4 + 0x24201C00u) | 0xFFu, LKlo = T4 + 0x10141828u, LKhi = T4 + 0x0004080Cu, K7 = (u32)(T0 + 39) * 0x01010101u;
        u32 mk[8], Cw[2], esc[2], ne[2], bm = 0;
        u64 wide[2];
#pragma unroll
        for (int h = 0; h < 2; ++h) {
            u32 Cb = 0, k[4], v[4];
            Cw[h] = 0;
#pragma unroll
            for (int i = 0; i < 4; ++i) {
                const int j = 4 * h + i;
                const u32 ex = (x[j] >> 2) & 0x1F1F1F1Fu, q = ex + Q;
                const u32 g7 = q & 0x80808080u, gm = (g7 >> 4) - (g7 >> 7);
                u32 c = (q + 0x01010101u) & gm & 0x03030303u;
                k[i] = (K7 - ex) & ~gm & 0x07070707u;
                const u32 e = ~(c | (c >> 1)) & 0x01010101u;
                const u32 t = e + (e << 8);
                v[i] = t + (t << 16);
                const u32 u = v[i] + Cb;
                if (i >= 2) { const u32 ov = e & ((u + 0x77777777u) >> 7); c |= ov; k[i] &= ~((ov << 3) - ov); }
                Cb = __builtin_amdgcn_perm(u, u, 0x03030303u);
                Cw[h] |= c << (2 * i);
                const u32 pc = __builtin_amdgcn_perm(0u, LC, c);
                const u32 pk = __builtin_amdgcn_perm(LKhi, LKlo, k[i] | (~pc & 0x80808080u));
                mk[j] = (x[j] ^ (pk & pc)) & 0x7C7C7C7Cu;
                const u32 nz = __builtin_amdgcn_udot4((mk[j] + 0x7C7C7C7Cu) & 0x80808080u, 0x08040201u, 0u, false);
                bm |= j < 2 ? nz >> (7 - 4 * j) : nz << (4 * j - 7);
            }
            const bool w3 = ((k[0] | k[1] | k[2] | k[3]) & 0x04040404u) != 0;
            wide[h] = __ballot(w3);
            const u32 b = w3 ? 3 : 2, wlo = w3 ? 0x40080100u : 0x10040100u, whi = w3 ? 0u : 0x40u, m4 = w3 ? 0xE00u : 0u;
            u32 sh = 0;
            esc[h] = 0;
#pragma unroll
            for (int i = 0; i < 4; ++i) {
                const u32 four = (k[i] >> 15) & m4 & (u32)__builtin_amdgcn_sbfe((int)v[i], 26, 1);
                esc[h] |= __builtin_amdgcn_udot4(k[i], __builtin_amdgcn_perm(whi, wlo, v[i]), four, false) << sh;
                sh += b * (v[i] >> 24);
            }
            ne[h] = min(sh, 8 * b);
        }
        const u32 cnt = __builtin_popcount(bm), nb = ne[0] + ne[1];
        const u32 both = wave_scan_incl(nb | (cnt << 16)), tot = rdl(both, 63);
        const u32 ndw = ((tot & 0xFFFFu) + 31) >> 5, nex = tot >> 16, nov = nex > NENT ? nex - NENT : 0;
        const u32 need = ndw + (nov ? (nov + 2) >> 1 : 0);
        const bool raw = need > 192;
        u32 start = 0;
        if (lane == 0) start = __hip_atomic_fetch_add((LAS u32*)(L + L_CTL) + C_ALLOC, raw ? 192u : need, __ATOMIC_RELAXED, __HIP_MEMORY_SCOPE_WORKGROUP);
        char* blk = p.ws + WST_OFF + par * WST_PAR + (size_t)(cbe * 4 + kse) * BLK_G;
        u32x4 P0; u32 P1;
        P0.x = (x[0] & 0x83838383u) | (__builtin_amdgcn_alignbit(x[1], x[1], 29) & 0x1C1C1C1Cu) | ((x[6] << 5) & 0x60606060u);
        P0.y = (x[2] & 0x83838383u) | (__builtin_amdgcn_alignbit(x[3], x[3], 29) & 0x1C1C1C1Cu) | ((x[6] >> 1) & 0x40404040u) | ((x[7] >> 2) & 0x20202020u);
        P0.z = (x[4] & 0x83838383u) | (__builtin_amdgcn_alignbit(x[5], x[5], 29) & 0x1C1C1C1Cu) | ((x[7] << 5) & 0x60606060u);
        P0.w = Cw[0]; P1 = Cw[1];
        if (__builtin_expect(raw, 0)) { P0 = u32x4{x[0], x[1], x[2], x[3]}; P1 = x[4]; }
        st16_sys(blk + wave * FR_B + lane * 16, P0);
        st4_sys(blk + wave * FR_B + 1024 + lane * 4, P1);
        start = rfl(start);
        char* pool = blk + POOL_B + start * 4;
        u32 ev = 0;
        if (__builtin_expect(raw, 0)) {
            st4_sys(pool + lane * 4, x[5]); st4_sys(pool + 256 + lane * 4, x[6]); st4_sys(pool + 512 + lane * 4, x[7]);
        } else {
            const u32 bp = (both & 0xFFFFu) - nb;
            const u64 V = (u64)esc[0] | ((u64)esc[1] << ne[0]), U = V << (bp & 31);
            LAS u32* sp = (LAS u32*)(size_t)(sbuf + (bp >> 5) * 4);
            __hip_atomic_fetch_or(sp, (u32)U, __ATOMIC_RELAXED, __HIP_MEMORY_SCOPE_WORKGROUP);
            __hip_atomic_fetch_or(sp + 1, (u32)(U >> 32), __ATOMIC_RELAXED, __HIP_MEMORY_SCOPE_WORKGROUP);
            __hip_atomic_fetch_or(sp + 2, ((u32)(V >> 32) >> 1) >> (31 - (bp & 31)), __ATOMIC_RELAXED, __HIP_MEMORY_SCOPE_WORKGROUP);
            for (u32 i = lane; i < ndw; i += 64) st4_sys(pool + i * 4, LDS_AT(u32, sbuf + i * 4));
            u32 idx = (both >> 16) - cnt;
            while (__any(bm != 0)) {
                const u32 pos = __builtin_ctz(bm | 0x80000000u);
                const bool b0 = pos & 4, b1 = pos & 8, b2 = pos & 16;
                const u32 m01 = b0 ? mk[1] : mk[0], m23 = b0 ? mk[3] : mk[2], m45 = b0 ? mk[5] : mk[4], m67 = b0 ? mk[7] : mk[6];
                const u32 m03 = b1 ? m23 : m01, m47 = b1 ? m67 : m45, m = b2 ? m47 : m03;
                const u32 en = (pos >> 4) * 1024 + lane * 16 + (pos & 15) + (((m >> (8 * (pos & 3) + 2)) & 31u) << 11);
                ev |= (u32)__builtin_amdgcn_ds_permute((int)((bm != 0 && idx < NENT ? idx + 10 : 63u) * 4), (int)en);
                if (__builtin_expect(bm != 0 && idx >= NENT, 0)) st2_sys(pool + ndw * 4 + (idx - NENT + 1) * 2, en);
                ++idx; bm &= bm - 1;
            }
            if (nov && lane == 0) st2_sys(pool + ndw * 4, nov);
        }
        if (lane < 8) ev = (u32)((lane < 4 ? wide[0] : wide[1]) >> (16 * (lane & 3)));
        if (lane == 8) ev = (u32)T0 | (raw || nov ? 32u : 0u) | (raw ? 64u : 0u);
        if (lane == 9) ev = start;
        if (lane < 16) st2_sys(blk + MAIN_B + wave * ENT_B + lane * 2, ev);
        LDS_AT(u32x4, amb + lane * 16) = u32x4{0, 0, 0, 0};
        LDS_AT(u32x4, amb + 1024 + lane * 16) = u32x4{0, 0, 0, 0};
        WAIT_VM(0);
        u32 a2 = 0;
        if (lane == 0) a2 = __hip_atomic_fetch_add((LAS u32*)(L + L_CTL) + C_ARR, 1u, __ATOMIC_RELAXED, __HIP_MEMORY_SCOPE_WORKGROUP);
        if (rfl(a2) == 3) {
            const u32 n16 = MIN16 + ((ctl[C_ALLOC] + 3) >> 2);
            if (lane < NR && lane != rank) {
                GAS u64* f = (GAS u64*)(fpeer + WFL_OFF + par * WFL_PAR) + ((cbe * 8 + rank) * 4 + kse);
                __hip_atomic_store(f, ((u64)call << 32) | n16, __ATOMIC_RELAXED, __HIP_MEMORY_SCOPE_SYSTEM);
            }
            ctl[C_PUB] = 1;
        }
        STAMP(2);
        } else {
            LDS_AT(u32x4, amb + lane * 16) = u32x4{0, 0, 0, 0};
            LDS_AT(u32x4, amb + 1024 + lane * 16) = u32x4{0, 0, 0, 0};
            WG_BARRIER(call);
            call = call + 1;
            scale = p.sx[0] * p.sw[0];
            asm volatile("" : "+v"(scale));
        }

        const int fw = wave & 3, grp = wave >> 2;
        f32x4 acc[8];
#pragma unroll
        for (int f = 0; f < 8; ++f) acc[f] = f32x4{0, 0, 0, 0};
        int avw = 0, avx = 0;
        const u32 aq0 = lbase + L_W + fw * FR_B + lane * 16, aq1 = lbase + L_W + fw * FR_B + 1024 + lane * 4;
        const u32 awd = lbase + L_W + MAIN_B + fw * ENT_B, aen = awd + 16 + (lane < 8 ? lane : 0) * 2, axx = lbase + L_X + lane * 16;
        const u32 scr = lbase + L_CTL + 64 + lane;
        auto wait_for = [&](int i, int& av) {
            u64 t0 = 0;
            for (u32 it = 1;; ++it) {
                const u32x2 fl = LDS_AT(u32x2, lbase + L_CTL);
                avw = (int)rfl(fl.x) + 4; avx = (int)rfl(fl.y);
                if (i < av) break;
                if (__builtin_expect((it & 1023) == 0, 0)) {
                    const u64 t = rt();
                    if (t0 == 0) t0 = t;
                    if (t - t0 > 2 * DEADLINE) { ctl[C_FAIL] = 1; avw = avx = NSTEP; break; }
                }
                __builtin_amdgcn_s_sleep(1);
            }
            asm volatile("" ::: "memory");
        };
        auto mma8 = [&](int next, const i32x8& wf, const u32x4 (&xa)[16]) {
#pragma unroll
            for (int f = 0; f < 8; ++f) {
                const u32x4 a = xa[2 * f], b = xa[2 * f + 1];
                i32x8 xf;
                xf[0] = a.x; xf[1] = a.y; xf[2] = a.z; xf[3] = a.w; xf[4] = b.x; xf[5] = b.y; xf[6] = b.z; xf[7] = b.w;
                if (f == 1) {
                    asm volatile("" :: "v"(xa[15]), "v"(wf) : "memory");
                    ctl[C_FREE + wave] = next;
                    __builtin_amdgcn_sched_barrier(0);
                }
                acc[f] = __builtin_amdgcn_mfma_scale_f32_16x16x128_f8f6f4(wf, xf, acc[f], 1  , 0  , 0, 0, 0, 0);
            }
        };
        if (grp == NG - 1)
#pragma nounroll
        for (int i = 0; i < 4; ++i) {
            if (i >= avx) wait_for(i, avx);
            u32x2 t0, t1, t2, t3;
            const u32 a = lbase + L_W + i * BLK_B + fw * 16 + (u32)(((lane >> 4) * 16 + ((lane & 15) >> 1)) * 64 + (lane & 1) * 8);
            asm volatile("ds_read_b64_tr_b8 %0, %4\n\tds_read_b64_tr_b8 %1, %4 offset:512\n\t"
                         "ds_read_b64_tr_b8 %2, %4 offset:4096\n\tds_read_b64_tr_b8 %3, %4 offset:4608\n\ts_waitcnt lgkmcnt(0)"
                         : "=&v"(t0), "=&v"(t1), "=&v"(t2), "=&v"(t3) : "v"(a) : "memory");
            u32x4 xa[16];
#pragma unroll
            for (int f = 0; f < 16; ++f) xa[f] = LDS_AT(u32x4, axx + i * XS_B + f * 1024);
            __builtin_amdgcn_sched_barrier(0);
            mma8(i < 3 ? i + 1 : 3 + NG, i32x8{(int)t0.x, (int)t0.y, (int)t1.x, (int)t1.y, (int)t2.x, (int)t2.y, (int)t3.x, (int)t3.y}, xa);
        }
        u32 xo = grp * XS_B, wo = (4 + grp) * BLK_B;
#pragma nounroll
        for (int i = 4 + grp; i < NSTEP; i += NG) {
            if (i >= avx) wait_for(i, avx);
            u32x4 xa[16];
#pragma unroll
            for (int f = 0; f < 16; ++f) xa[f] = LDS_AT(u32x4, axx + xo + f * 1024);
            if (i >= avw) { __builtin_amdgcn_s_setprio(0); wait_for(i, avw); __builtin_amdgcn_s_setprio(PRIO_C); }
            const u32 e0 = LDS_AT(unsigned short, aen + wo);
            const u32x4 q0 = LDS_AT(u32x4, aq0 + wo);
            const u32 q1 = LDS_AT(u32, aq1 + wo);
            const u32x4 wd = LDS_AT(u32x4, awd + wo);
            __builtin_amdgcn_sched_barrier(0);
            const u32 hdr = rfl(e0), pool = lbase + L_W + POOL_B + wo + (rdl(e0, 1) & 1023u) * 4;
            const u32 ea = e0 >> 11 ? amb + (e0 & 2047u) : scr;
            LDS_AT(unsigned char, ea) = (unsigned char)(e0 >> 11 << 2);
            const bool w3[2] = {__builtin_amdgcn_inverse_ballot_w64(((u64)rfl(wd.y) << 32) | rfl(wd.x)),
                                __builtin_amdgcn_inverse_ballot_w64(((u64)rfl(wd.w) << 32) | rfl(wd.z))};
            const u32 cw[2] = {w3[0] ? 3u : 2u, w3[1] ? 3u : 2u};
            const u32 n0 = cw[0] * __builtin_popcount(~(q0.w | (q0.w >> 1)) & 0x55555555u), nn = n0 + cw[1] * __builtin_popcount(~(q1 | (q1 >> 1)) & 0x55555555u);
            const u32 incl = wave_scan_incl(nn), bp = incl - nn;
            const u32 sa = pool + (bp >> 5) * 4;
            const u32 d0 = LDS_AT(u32, sa), d1 = LDS_AT(u32, sa + 4), d2 = LDS_AT(u32, sa + 8);
            if (__builtin_expect((hdr & 96u) == 32u, 0)) {
                const u32 ov = pool + ((rdl(incl, 63) + 31) >> 5) * 4;
                const u32 n = rfl(LDS_AT(unsigned short, ov)) & 511u;
                for (u32 k = 0; k < n; k += 64) {
                    const u32 e = LDS_AT(unsigned short, ov + 2 + (k + lane) * 2);
                    if (k + lane < n) LDS_AT(unsigned char, amb + (e & 2047u)) = (unsigned char)(e >> 11 << 2);
                }
                __builtin_amdgcn_s_waitcnt(0xC07F);
            }
            const u32x4 m0 = LDS_AT(u32x4, amb + lane * 16);
            const u32x4 m1 = LDS_AT(u32x4, amb + 1024 + lane * 16);
            LDS_AT(unsigned char, ea) = 0;
            if (__builtin_expect(hdr & 32u, 0)) { LDS_AT(u32x4, amb + lane * 16) = u32x4{0, 0, 0, 0}; LDS_AT(u32x4, amb + 1024 + lane * 16) = u32x4{0, 0, 0, 0}; }
            __builtin_amdgcn_sched_barrier(0);
            const u32 T4 = (hdr & 31u) * 0x04040404u;
            const u32 LC = (T4 + 0x24201C00u) | 0xFFu, LKlo = T4 + 0x10141828u, LKhi = T4 + 0x0004080Cu;
            u32 pc[8], sel[8];
#pragma unroll
            for (int h = 0; h < 2; ++h) {
                const u32 C = h ? q1 : q0.w;
                u32 Cb = 0x07070707u;
#pragma unroll
                for (int i = 0; i < 4; ++i) {
                    const int j = 4 * h + i;
                    pc[j] = __builtin_amdgcn_perm(0u, LC, (C >> (2 * i)) & 0x03030303u);
                    const u32 e = pc[j] & 0x01010101u, t = e + (e << 8), u = (t << 16) + (t + Cb);
                    sel[j] = __builtin_amdgcn_bitop3_b32(u, pc[j], 0x07070707u, 0xB1);
                    Cb = __builtin_amdgcn_perm(u, u, 0x03030303u);
                }
            }
            const u32 w0 = __builtin_amdgcn_alignbit(d1, d0, bp), w1 = __builtin_amdgcn_alignbit(d2, d1, bp);
            i32x8 wf;
#pragma unroll
            for (int h = 0; h < 2; ++h) {
                const u32 w = h ? __builtin_amdgcn_alignbit(w1, w0, n0) : w0;
                const u32 c4 = 4 * cw[h], s1 = w3[h] ? 10u : 12u, s2 = 8 - cw[h];
                const u32 k1 = w3[h] ? 0x003F003Fu : 0x000F000Fu, k2 = w3[h] ? 0x07070707u : 0x03030303u;
                const u32 wa = __builtin_amdgcn_ubfe(w, 0u, c4), wb = __builtin_amdgcn_ubfe(w, c4, c4);
                const u32 ya = (wa | (wa << s1)) & k1, yb = (wb | (wb << s1)) & k1;
                const u32 Ka = __builtin_amdgcn_perm(LKhi, LKlo, (ya | (ya << s2)) & k2);
                const u32 Kb = __builtin_amdgcn_perm(LKhi, LKlo, (yb | (yb << s2)) & k2);
                const u32x4 m = h ? m1 : m0;
#pragma unroll
                for (int i = 0; i < 4; ++i) wf[4 * h + i] = (__builtin_amdgcn_perm(Kb, Ka, sel[4 * h + i]) & pc[4 * h + i]) ^ m[i];
            }
            wf[0] |= q0.x & 0x83838383u; wf[1] |= __builtin_amdgcn_alignbit(q0.x, q0.x, 3) & 0x83838383u;
            wf[2] |= q0.y & 0x83838383u; wf[3] |= __builtin_amdgcn_alignbit(q0.y, q0.y, 3) & 0x83838383u;
            wf[4] |= q0.z & 0x83838383u; wf[5] |= __builtin_amdgcn_alignbit(q0.z, q0.z, 3) & 0x83838383u;
            wf[6] |= ((q0.x >> 5) & 0x03030303u) | ((q0.y << 1) & 0x80808080u);
            wf[7] |= ((q0.z >> 5) & 0x03030303u) | ((q0.y << 2) & 0x80808080u);
            if (__builtin_expect(hdr & 64u, 0)) {
                wf[0] = q0.x; wf[1] = q0.y; wf[2] = q0.z; wf[3] = q0.w; wf[4] = q1;
                wf[5] = LDS_AT(u32, pool + lane * 4); wf[6] = LDS_AT(u32, pool + 256 + lane * 4); wf[7] = LDS_AT(u32, pool + 512 + lane * 4);
                __builtin_amdgcn_s_waitcnt(0xC07F);
            }
            __builtin_amdgcn_sched_barrier(0);
            if (i >= NSTEP - NG)
                for (u32 it = 0; min_free() < NSTEP - 2 && it < (1u << 26); ++it) __builtin_amdgcn_s_sleep(0);
            mma8(i + NG, wf, xa);
            xo = (xo + NG * XS_B) & (NSX * XS_B - 1);
            wo = wo + NG * BLK_B >= NSW * BLK_B ? wo + NG * BLK_B - NSW * BLK_B : wo + NG * BLK_B;
        }
        STAMP(5);
        constexpr u32 T_0 = L_X, T_1 = L_W + 5 * BLK_B;
        const u32 tw = lbase + (grp ? T_1 : T_0) + (lane & 15) * 256 + (((fw * 4 + (lane >> 4)) ^ (lane & 15)) * 16);
#pragma unroll
        for (int f = 0; f < 8; ++f) LDS_AT(f32x4, tw + f * 4096) = acc[f];
        WAIT_LGKM0();
        ctl[C_FREE + wave] = 2 * NSTEP;
        if (wave >= 4) STAMP(2);
        __builtin_amdgcn_s_setprio(0);
        for (u32 it = 0; min_free() < 2 * NSTEP && it < (1u << 24); ++it) __builtin_amdgcn_s_sleep(ESLEEP);
        if (wave >= 4) STAMP(3);
        const float sc = ctl[C_FAIL] != 0 ? __builtin_nanf("") : scale;
        constexpr int RW = 32 / NG;
        float* o = p.out + (size_t)(rb * 64 + wave * RW + (lane >> 4)) * LDO + cb * 64 + (lane & 15) * 4;
        f32x4 y[RW / 4];
#pragma unroll
        for (int f = 0; f < RW / 4; ++f) {
            const u32 tr = lbase + (wave * RW + f * 4 + (lane >> 4)) * 256 + (((lane & 15) ^ ((f * 4 + (lane >> 4)) & 15)) * 16);
            y[f] = LDS_AT(f32x4, tr + T_0);
            if (NG == 2) y[f] += LDS_AT(f32x4, tr + T_1);
        }
#pragma unroll
        for (int f = 0; f < RW / 4; ++f)
            st16_sys(o + (size_t)f * 4 * LDO, __builtin_bit_cast(u32x4, __builtin_elementwise_maximum(y[f] * sc, f32x4{0, 0, 0, 0})));
        STAMP(6);
    } else if (wave == 8) {
        WG_BARRIER(call);
        call = call + 1;
        par = call & 1;
        const int R = lane < NREM ? lane : 0, src = src_of_rem(R, rank, cb), ks = R / 7;
        const u64 baddr = (((u64)(u32)__shfl((int)((u64)fpeer >> 32), src) << 32) | (u32)__shfl((int)(u32)(u64)fpeer, src)) + WST_OFF + par * WST_PAR + (u64)(cb * 4 + ks) * BLK_G;
        const u32 blo = (u32)baddr, bhi = (u32)(baddr >> 32);
        GAS u64* fl = (GAS u64*)(p.ws + WFL_OFF + par * WFL_PAR) + ((cb * 8 + src) * 4 + ks);
        const u32 voff = lane * 16;
        u32 n16v = MIN16;
        bool dead = false;
        const u32 xcc = __builtin_amdgcn_s_getreg((3 << 11) | 20) & 15;
        GAS u64* inv = (GAS u64*)(p.ws + INV_OFF + xcc * 256);
        u32 prev = 0;
        if (lane == 0) prev = __hip_atomic_fetch_max((GAS u32*)(inv + 16), call, __ATOMIC_RELAXED, __HIP_MEMORY_SCOPE_AGENT);
        if (rfl(prev) < call) {
            __builtin_amdgcn_fence(__ATOMIC_ACQUIRE, "");
            WAIT_VM(0);
            if (lane == 0) __hip_atomic_store(inv, (u64)call << 32, __ATOMIC_RELAXED, __HIP_MEMORY_SCOPE_AGENT);
        }
        STAMP(3);
        if (lane >= 28 && lane < 56) fl = (GAS u64*)(p.ws + XFL_OFF + par * XFL_PAR) + ((rb * 8 + src_of_rem(lane, rank, 0)) * 4 + (lane - 28) / 7);
        if (lane == 62) fl = (GAS u64*)(p.ws + TMO_OFF);
        if (lane == 63) fl = inv;
        {
            const u64 t0 = rt();
            for (u32 it = 1;; ++it) {
                const u64 v = __hip_atomic_load(fl, __ATOMIC_RELAXED, __HIP_MEMORY_SCOPE_SYSTEM);
                n16v = (u32)v;
                const u64 up = __ballot((u32)(v >> 32) >= call);
                if (((up >> 62) & 1) && rdl((u32)(v >> 32), 62) == call) { dead = true; break; }
                if ((up | 0x7FFFFFFFF0000000ull) == ~0ull && (__builtin_popcount((u32)(up >> 28) & 0xFFFFFFFu) >= XTH || rt() - t0 > 2000)) break;
                if ((it & 15) == 0 && expired(t0)) { dead = true; break; }
                __builtin_amdgcn_s_sleep(1);
            }
            n16v = n16v < MIN16 ? MIN16 : (n16v > MAX16 ? MAX16 : n16v);
        }
        STAMP(1);
        for (u32 it = 0; rfl(ctl[C_PUB]) == 0 && it < (1u << 24); ++it) __builtin_amdgcn_s_sleep(1);
        STAMP(4);
        __builtin_amdgcn_s_setprio(PRIO_W);
        int pub = 0, freed = 0;
        u32 lo = lbase + L_W + 4 * BLK_B;
        auto publish = [&](int upto) { if (upto > pub) { pub = upto; ctl[C_FULLW] = upto; } };
        u64 g[LAST + 1]; u32 l[LAST + 1];
#pragma unroll
        for (int j = 0; j <= LAST; ++j) { g[j] = 0; l[j] = 0; }
#define W_LOAD(OFF, J, K, NOP) do { dma16<OFF, 0, NOP>(g[J], voff, l[J]); WAIT_VM(WIN); after(std::integral_constant<int, K>{}); } while (0)
        auto rest_of_slot = [&](int n, bool mid, auto&& after) {
            if (mid) { W_LOAD(2048, LAG, 3, 0); W_LOAD(3072, LAG, 4, 0); }
            if (n >= LAST) {
                g[LAST] += 4096; l[LAST] += 4096;
                W_LOAD(0, LAST, 5, 0);
                const u32 n16 = rdl(n16v, n - LAST);
                if (__builtin_expect(n16 > 64, 0))
                    for (u32 i = 64; i < n16; i += 64) { if (lane + i < n16) dma16<1024, 0>(g[LAST] + i * 16, voff, l[LAST] + i * 16); WAIT_VM(WIN); }
                if ((u32)lane < n16) dma16<1024, 0>(g[LAST], voff, l[LAST]);
                WAIT_VM(WIN); after(std::integral_constant<int, SL>{});
            }
#pragma unroll
            for (int j = LAST; j > 0; --j) { g[j] = g[j - 1]; l[j] = l[j - 1]; }
        };
        auto open = [&](int n, int done) {
            if (freed <= n + 4 - NSW) {
                const int need = n + 4 - NSW;
                const u64 t0 = rt();
                bool drained = false;
                for (;;) {
                    freed = min_free();
                    if (freed > need) break;
                    if (!drained) { WAIT_VM(0); publish(done); drained = true; continue; }
                    if (rt() - t0 > 2 * DEADLINE) { dead = true; break; }
                    __builtin_amdgcn_s_sleep(1);
                }
                if (dead) return;
            }
            g[0] = ((u64)rdl(bhi, n) << 32) | rdl(blo, n);
            l[0] = lo;
            lo = lo + BLK_B == lbase + L_W + NSW * BLK_B ? lbase + L_W : lo + BLK_B;
        };
        for (int n = 0; n < NB; ++n) {
            constexpr int PK = (WIN - 1) % SL + 1, PL = LAST + 1 + (WIN - PK) / SL;
            auto after = [&](auto k) { if constexpr (decltype(k)::value == PK) publish(n - PL + 1); };
            if (!dead) open(n, n - LAST);
            if (dead) break;
            W_LOAD(0, 0, 1, 4); W_LOAD(1024, 0, 2, 0);
            rest_of_slot(n, n >= LAG, after);
        }
        if (!dead) {
            static_for<LAST>([&](auto dd) {
                constexpr int d = decltype(dd)::value;
                rest_of_slot(NB + d, d < LAG, [&](auto k) { publish(std::integral_constant<int, w_landed(w_cum(d) - (SL - decltype(k)::value), WIN)>::value); });
            });
            static_for<NU>([&](auto uu) {
                constexpr int u = decltype(uu)::value;
                auto after = [&](auto k) { publish(std::integral_constant<int, w_landed(w_cum(LAST + u) - (SL - decltype(k)::value), WIN)>::value); };
                if (!dead) open(NB + u, NB + u);
                if (dead) return;
                W_LOAD(0, 0, 1, 4); W_LOAD(1024, 0, 2, 0);
                g[LAG] = g[LAST] = g[0]; l[LAG] = l[LAST] = l[0];
                rest_of_slot(NB + u + LAST, true, after);
            });
        }
        STAMP(5);
        if (dead) { ctl[C_FAIL] = 1; WAIT_VM(0); ctl[C_FULLW] = NREM; }
        else
            static_for<8>([&](auto dd) {
                constexpr int upto = NREM - 7 + decltype(dd)::value, behind = w_behind(upto - 1, w_cum(LAST + NU - 1));
                if constexpr (behind < WIN) { WAIT_VM(behind); publish(upto); }
            });
        STAMP(6);
    } else if (wave == 9) {
        WG_BARRIER(call);
        call = call + 1;
        par = call & 1;
        const int R = max((lane & 31) - 4, 0), src = src_of_rem(R, rank, cb), ks = R / 7;
        const u32 toff = (u32)((src * 4 + ks) * 8 + rb) * XT_B;
        const u64 xbase = (u64)p.ws + XRV_OFF + par * XRV_PAR;
        GAS u64* fl = (GAS u64*)(p.ws + XFL_OFF + par * XFL_PAR) + (((rb + (lane >> 5)) * 8 + src) * 4 + ks);
        constexpr u32 OWN = 0xFu;
        const u32 voff = lane * 16;
        const u32 xoff = (u32)(lane & 15) * LDX + (lane >> 4) * 16;
        u32 ready = OWN;
        u32 it = 0, la = lbase + L_X;
        bool dead = false;
        const u64 t0 = rt();
        int pub = 0, freed = 0;
        for (u32 k = 0; rfl(ctl[C_PUB]) == 0 && k < (1u << 22); ++k) __builtin_amdgcn_s_sleep(4);
        for (int s = 0; s < NSTEP && !dead; ++s) {
            if (!((ready >> s) & 1)) {
                WAIT_VM(0);
                if (pub < s) { pub = s; ctl[C_FULLX] = s; }
                for (;;) {
                    const u64 v = __hip_atomic_load(fl, __ATOMIC_RELAXED, __HIP_MEMORY_SCOPE_SYSTEM);
                    const u64 ok = __ballot((u32)(v >> 32) >= call);
                    ready = ((u32)ok & (u32)(ok >> 32)) | OWN;
                    if (ready == ~0u) { dead = rfl(__hip_atomic_load(tmo, __ATOMIC_RELAXED, __HIP_MEMORY_SCOPE_AGENT)) == call; break; }
                    if ((ready >> s) & 1) break;
                    if ((++it & 15) == 0 && expired(t0)) { dead = true; break; }
                    __builtin_amdgcn_s_sleep(2);
                }
            }
            while (!dead && freed <= s - NSX) {
                freed = min_free();
                if (freed > s - NSX) break;
                if ((++it & 1023) == 0 && rt() - t0 > 2 * DEADLINE) dead = true;
                __builtin_amdgcn_s_sleep(1);
            }
            if (dead) break;
            if (s < 4) {
                const u64 b = (u64)p.x + (u64)(rank * 512 + rb * 64) * LDX + s * 128;
#pragma unroll
                for (int i = 0; i < 16; ++i) dma16<0, 0>(b + (i >> 1) * 16 * LDX + (i & 1) * 64, xoff, la + i * 1024);
#pragma unroll
                for (int i = 0; i < 8; ++i)
                    dma16<0, 0>((u64)p.w + (u64)(s * 128 + i * 16) * LDW + cb * 64, (u32)(lane >> 2) * LDW + (lane & 3) * 16, lbase + L_W + s * BLK_B + i * 1024);
            } else {
                const u64 b = xbase + rdl(toff, s);
#pragma unroll
                for (int i = 0; i < 4; ++i) {
                    dma16<0, 1, 4>(b + i * 4096, voff, la + i * 4096); dma16<1024, 1>(b + i * 4096, voff, la + i * 4096);
                    dma16<2048, 1>(b + i * 4096, voff, la + i * 4096); dma16<3072, 1>(b + i * 4096, voff, la + i * 4096);
                }
            }
            la = la + XS_B == lbase + L_X + NSX * XS_B ? lbase + L_X : la + XS_B;
            WAIT_VM(16 * (XD - 1));
            if (pub < s - XD + 2) { pub = s - XD + 2; ctl[C_FULLX] = pub; }
        }
        if (dead) { ctl[C_FAIL] = 1; WAIT_VM(0); }
        else {
            WAIT_VM(32); ctl[C_FULLX] = max(pub, NSTEP - 2);
            WAIT_VM(16); ctl[C_FULLX] = max(pub, NSTEP - 1);
            WAIT_VM(0);
        }
        ctl[C_FULLX] = NSTEP;
        STAMP(6);
    } else {
        const int id = bid / 7, d = (rank + 1 + bid % 7) & 7;
        u32x4 v[8];
        if (id < 32) {
            const unsigned char* g = p.x + (size_t)(d * 512 + (id & 7) * 64 + (lane & 15)) * LDX + (id >> 3) * 128 + (lane >> 4) * 16;
#pragma unroll
            for (int i = 0; i < 8; ++i) v[i] = *(const GAS u32x4*)(g + (size_t)(i >> 1) * 16 * LDX + (i & 1) * 64);
        }
        WG_BARRIER(call);
        call = call + 1;
        par = call & 1;
        if (id < 32) {
            asm volatile("" : "+v"(v[0]), "+v"(v[1]), "+v"(v[2]), "+v"(v[3]), "+v"(v[4]), "+v"(v[5]), "+v"(v[6]), "+v"(v[7]), "+v"(fpeer));
            char* peer = (char*)(((u64)rdl((u32)((u64)fpeer >> 32), d) << 32) | rdl((u32)(u64)fpeer, d));
            char* dst = peer + XRV_OFF + par * XRV_PAR + (size_t)(rank * 32 + id) * XT_B + lane * 16;
#pragma unroll
            for (int i = 0; i < 8; ++i) { st16_sys(dst + i * 1024, v[i]); WAIT_VM(PUSH_W - 1); }
            STAMP(1);
            WAIT_VM(0);
            if (lane == 0)
                __hip_atomic_store((GAS u64*)(peer + XFL_OFF + par * XFL_PAR) + (((id & 7) * 8 + rank) * 4 + (id >> 3)), (u64)call << 32,
                                   __ATOMIC_RELAXED, __HIP_MEMORY_SCOPE_SYSTEM);
            STAMP(2);
        }
    }
    if (lane == 0) __hip_atomic_store(cntp, call, __ATOMIC_RELAXED, __HIP_MEMORY_SCOPE_AGENT);
}

extern "C" void kernel_launch(int rank, int nranks, ncclComm_t comm,
                              hipStream_t stream,
                              void* const* d_in, const int* in_sizes, int n_in,
                              void* d_out, int out_size,
                              const int* in_dtypes, int out_dtype,
                              void* const* d_out_all,
                              void* const* d_ws_all, size_t ws_bytes) {
    (void)comm; (void)in_dtypes; (void)out_dtype; (void)d_out_all;
    if (nranks != NR || (unsigned)rank >= (unsigned)NR || n_in != 4 || ws_bytes < WS_NEED ||
        in_sizes[0] != 4096 * 512 || in_sizes[1] != 512 * 4096 || out_size != 512 * 4096) {
        fprintf(stderr, "unexpected problem shape\n");
        return;
    }
    HIP_CHECK(hipFuncSetAttribute((const void*)gemm_xchg, hipFuncAttributeMaxDynamicSharedMemorySize, LDS_BYTES));
    Params p;
    p.x = (const unsigned char*)d_in[0];
    p.w = (const unsigned char*)d_in[1];
    p.sx = (const float*)d_in[2];
    p.sw = (const float*)d_in[3];
    p.out = (float*)d_out;
    p.ws_all = d_ws_all;
    p.ws = (char*)d_ws_all[rank];
    p.rank = rank;
    p.pad = 0;
    hipLaunchKernelGGL(gemm_xchg, dim3(NWG), dim3(NTHR), LDS_BYTES, stream, p);
    HIP_CHECK(hipGetLastError());
}
```

```cpp
#include <hip/hip_runtime.h>
#include <rccl/rccl.h>
#include <cstdio>

#define HIP_CHECK(expr) do { hipError_t _e = (expr); if (_e != hipSuccess) { \
    fprintf(stderr, "HIP error %s at %s:%d\n", hipGetErrorString(_e), __FILE__, __LINE__); \
    return; } } while (0)

typedef unsigned u32;
typedef unsigned long long u64;
typedef unsigned u32x4 __attribute__((ext_vector_type(4)));
typedef unsigned u32x2 __attribute__((ext_vector_type(2)));
typedef int i32x8 __attribute__((ext_vector_type(8)));
typedef float f32x4 __attribute__((ext_vector_type(4)));
#define GAS __attribute__((address_space(1)))
#define LAS __attribute__((address_space(3)))

constexpr int WIN = 4;
constexpr int NU = 10;
constexpr int ESLEEP = 1;
constexpr int PRIO_C = 1;
constexpr int NG = 2;
constexpr int PSLEEP = 4;
constexpr int PDELAY = 40;
constexpr int XSLEEP = 16;
constexpr int PRIO_W = 3;
constexpr int XTH = 5;
constexpr int PUSH_W = 2;
constexpr int XD = 2;
constexpr int NSW = 9;
constexpr int NSX = 4;

constexpr int NR = 8;
constexpr int NWG = 256, NTHR = 64 * (4 * NG + 3);
constexpr int LDX = 512, LDW = 4096, LDO = 4096;
constexpr int MAIN_B = 5120, FR_B = 1280, ENT_B = 32, NENT = 6, POOL_B = MAIN_B + 4 * ENT_B;
constexpr int MIN16 = 4 * ENT_B / 16, MAX16 = (4 * ENT_B + 4 * 768) / 16;
constexpr int BLK_B = 8448;
constexpr int BLK_G = 8448;
constexpr int XT_B = 8192;
constexpr int XS_B = 2 * XT_B;
constexpr int NSTEP = 32, NREM = 28;
static_assert(MAIN_B + MAX16 * 16 + 16 <= BLK_B && BLK_B <= BLK_G, "block");

constexpr size_t WST_OFF = 0, WST_PAR = 4u << 20;
constexpr size_t XRV_OFF = 8u << 20, XRV_PAR = 2u << 20;
constexpr size_t WFL_OFF = 12u << 20, WFL_PAR = 16u << 10;
constexpr size_t XFL_OFF = WFL_OFF + (32u << 10), XFL_PAR = 4u << 10;
constexpr size_t CNT_OFF = WFL_OFF + (64u << 10);
constexpr size_t TMO_OFF = WFL_OFF + (128u << 10);
constexpr size_t INV_OFF = TMO_OFF + 1024;
constexpr size_t WS_NEED = 13u << 20;
static_assert((size_t)256 * BLK_G <= WST_PAR, "w staging");

constexpr int L_W = 0;
constexpr int L_X = L_W + NSW * BLK_B;
constexpr int L_MASK = L_X + NSX * XS_B;
constexpr int L_CTL = L_MASK + 16384;
constexpr int LDS_BYTES = L_CTL + 128;
static_assert(NSW == 9 && NSX == 4, "the output staging areas are chosen for these rings");
static_assert(LDS_BYTES <= 160 * 1024 && NSW >= 8 && XD <= NSX && XD <= 4, "LDS");
constexpr int C_FULLW = 0, C_FULLX = 1, C_FAIL = 2, C_ALLOC = 3, C_ARR = 4, C_PUB = 5, C_FREE = 8  ;

constexpr u64 DEADLINE = 30000000ull;

struct Params {
    const unsigned char* x;
    const unsigned char* w;
    const float* sx;
    const float* sw;
    float* out;
    void* const* ws_all;
    char* ws;
    int rank;
    int pad;
};

__device__ __forceinline__ u64 rt() { return __builtin_amdgcn_s_memrealtime(); }
__device__ __forceinline__ u32 rfl(u32 v) { return (u32)__builtin_amdgcn_readfirstlane(v); }
__device__ __forceinline__ u32 rdl(u32 v, int l) { return (u32)__builtin_amdgcn_readlane(v, l); }
#define WAIT_VM(n) asm volatile("s_waitcnt vmcnt(%0)" :: "i"(n) : "memory")
#define WG_BARRIER(call) do { u32 c_; asm volatile("s_barrier" ::: "memory"); \
    asm volatile("v_readfirstlane_b32 %0, %1\n\ts_nop 4" : "=s"(c_) : "v"(call) : "memory"); call = c_; } while (0)
#define WAIT_LGKM0() asm volatile("s_waitcnt lgkmcnt(0)" ::: "memory")

template <int OFF, int POL, int NOP = 0>
__device__ __forceinline__ void dma16(u64 base, u32 voff, u32 ldsaddr) {
    if constexpr (POL == 1)
        asm volatile("s_mov_b32 m0, %2\n\ts_nop %4\n\tglobal_load_lds_dwordx4 %0, %1 offset:%3 sc0 sc1"
                     :: "v"(voff), "s"(base), "s"(ldsaddr), "i"(OFF), "i"(NOP) : "memory");
    else
        asm volatile("s_mov_b32 m0, %2\n\ts_nop %4\n\tglobal_load_lds_dwordx4 %0, %1 offset:%3"
                     :: "v"(voff), "s"(base), "s"(ldsaddr), "i"(OFF), "i"(NOP) : "memory");
}
__device__ __forceinline__ void st4_sys(void* p, u32 v) {
    asm volatile("global_store_dword %0, %1, off sc0 sc1\n\ts_nop 1" :: "v"(p), "v"(v) : "memory");
}
__device__ __forceinline__ void st2_sys(void* p, u32 v) {
    asm volatile("global_store_short %0, %1, off sc0 sc1\n\ts_nop 1" :: "v"(p), "v"(v) : "memory");
}
__device__ __forceinline__ void st16_sys(void* p, u32x4 v) {
    asm volatile("global_store_dwordx4 %0, %1, off sc0 sc1\n\ts_nop 1" :: "v"(p), "v"(v) : "memory");
}
__device__ __forceinline__ u32 wave_scan_incl(u32 v) {
    v += __builtin_amdgcn_update_dpp(0, v, 0x111, 0xf, 0xf, true);
    v += __builtin_amdgcn_update_dpp(0, v, 0x112, 0xf, 0xf, true);
    v += __builtin_amdgcn_update_dpp(0, v, 0x114, 0xf, 0xf, true);
    v += __builtin_amdgcn_update_dpp(0, v, 0x118, 0xf, 0xf, true);
    v += __builtin_amdgcn_update_dpp(0, v, 0x142, 0xa, 0xf, false);
    v += __builtin_amdgcn_update_dpp(0, v, 0x143, 0xc, 0xf, false);
    return v;
}

__device__ __forceinline__ int src_of_rem(int R, int rank, int cb) { return (rank + 1 + (cb + R) % 7) & 7; }

constexpr int LAG = 2, LAST = 2 * LAG, SL = 6, NB = NREM - NU;
static_assert(WIN >= 1 && WIN <= 18 && NSW > LAST + 2 && NU >= 1 && NB >= LAST, "window");
constexpr int w_cum(int d) { int c = 0; for (int t = 0; t <= d; ++t) c += t >= LAST ? SL : (t < LAG ? 4 : 2); return c; }
constexpr int w_behind(int b, int q) { return b >= NB - LAST ? q - w_cum(b - (NB - LAST)) : q + SL * (NB - LAST - 1 - b); }
constexpr int w_landed(int q, int win) { int u = 0; while (u < NREM && w_behind(u, q) >= win) ++u; return u; }
template <int N, int I = 0, class F> __device__ __forceinline__ void static_for(F&& f) {
    if constexpr (I < N) { f(std::integral_constant<int, I>{}); static_for<N, I + 1>(f); }
}

#define LDS_AT(T, a) (*(volatile LAS T*)(size_t)(a))

__global__ void __launch_bounds__(NTHR, 3) gemm_xchg(Params p) {
    extern __shared__ __attribute__((aligned(16))) unsigned char lds_raw[];
    LAS unsigned char* L = (LAS unsigned char*)lds_raw;
    const u32 lbase = (u32)(size_t)L;
    volatile LAS u32* ctl = (volatile LAS u32*)(L + L_CTL);
    const int tid = threadIdx.x, lane = tid & 63;
    const int wave = rfl(tid >> 6) == 0 ? 10 : (rfl(tid >> 6) <= 4 * NG ? rfl(tid >> 6) - 1 : rfl(tid >> 6) - 4 * NG + 7);
    const int bid = blockIdx.x, xc = bid & 7, jj = bid >> 3;
    const int rank = p.rank;
    const int rb = (jj & 3) * 2, cb = xc * 8 + (jj >> 2);
    GAS u32* tmo = (GAS u32*)(p.ws + TMO_OFF) + 1;

    GAS u32* cntp = (GAS u32*)(p.ws + CNT_OFF) + bid * 16 + wave;
    u32 call = __hip_atomic_load(cntp, __ATOMIC_RELAXED, __HIP_MEMORY_SCOPE_AGENT);
    char* fpeer = (char*)p.ws_all[lane & 7];
    u32 par;
    auto expired = [&](u64 t0) {
        if (rfl(__hip_atomic_load(tmo, __ATOMIC_RELAXED, __HIP_MEMORY_SCOPE_AGENT)) != call && rt() - t0 <= DEADLINE) return false;
        __hip_atomic_store(tmo, call, __ATOMIC_RELAXED, __HIP_MEMORY_SCOPE_AGENT);
        return true;
    };
    auto min_free = [&]() {
        const u32x4 a = LDS_AT(u32x4, lbase + L_CTL + C_FREE * 4), b = LDS_AT(u32x4, lbase + L_CTL + C_FREE * 4 + 16);
        return (int)rfl(min(min(min(a.x, a.y), min(a.z, a.w)), min(min(b.x, b.y), min(b.z, b.w))));
    };

    if (wave < 8) {
        const u32 amb = lbase + L_MASK + wave * 2048;
        float scale;
        if (wave < 4) {
        const int cbe = (jj & 1) + 2 * (xc + 8 * ((jj >> 1) & 3)), kse = jj >> 3;
        {
            const u64 src = (u64)p.w + (u64)(kse * 128) * LDW + cbe * 64 + wave * 16;
            dma16<0, 0, 4>(src, (u32)lane * LDW, amb);
            dma16<0, 0, 4>(src, (u32)(lane + 64) * LDW, amb + 1024);
        }
        if (wave == 0 && lane < 16) ctl[lane] = NG == 2 ? (lane >= C_FREE && lane < C_FREE + 4 ? 4 : 0) : (lane >= C_FREE + 4 ? 1000 : 0);
        WAIT_LGKM0();
        WG_BARRIER(call);
        scale = p.sx[0] * p.sw[0];
        asm volatile("" : "+v"(scale), "+v"(fpeer));
        call = call + 1;
        par = call & 1;
        WAIT_VM(0);
        u32 x[8];
        {
            u32x2 t0, t1, t2, t3;
            const u32 a = amb + (u32)(((lane >> 4) * 16 + ((lane & 15) >> 1)) * 16 + (lane & 1) * 8);
            asm volatile("ds_read_b64_tr_b8 %0, %4\n\tds_read_b64_tr_b8 %1, %4 offset:128\n\t"
                         "ds_read_b64_tr_b8 %2, %4 offset:1024\n\tds_read_b64_tr_b8 %3, %4 offset:1152\n\ts_waitcnt lgkmcnt(0)"
                         : "=&v"(t0), "=&v"(t1), "=&v"(t2), "=&v"(t3) : "v"(a) : "memory");
            x[0] = t0.x; x[1] = t0.y; x[2] = t1.x; x[3] = t1.y; x[4] = t2.x; x[5] = t2.y; x[6] = t3.x; x[7] = t3.y;
        }
        const u32 sbuf = lbase + L_W + (NSW - 1) * BLK_B + wave * 1024;
        LDS_AT(u32x4, sbuf + lane * 16) = u32x4{0, 0, 0, 0};
        u32 sum = 0;
#pragma unroll
        for (int i = 0; i < 8; ++i) sum = __builtin_amdgcn_sad_u8(x[i] & 0x7F7F7F7Fu, 0u, sum);
        sum = rdl(wave_scan_incl(sum), 63);
        int T0 = ((int)sum - 60211) >> 13;
        T0 = T0 < 0 ? 0 : (T0 > 21 ? 21 : T0);
        const u32 Q = (u32)(121 - T0) * 0x01010101u, T4 = (u32)T0 * 0x04040404u;
        const u32 LC = (T4 + 0x24201C00u) | 0xFFu, LKlo = T4 + 0x10141828u, LKhi = T4 + 0x0004080Cu, K7 = (u32)(T0 + 39) * 0x01010101u;
        u32 mk[8], Cw[2], esc[2], ne[2], bm = 0;
        u64 wide[2];
#pragma unroll
        for (int h = 0; h < 2; ++h) {
            u32 Cb = 0, k[4], v[4];
            Cw[h] = 0;
#pragma unroll
            for (int i = 0; i < 4; ++i) {
                const int j = 4 * h + i;
                const u32 ex = (x[j] >> 2) & 0x1F1F1F1Fu, q = ex + Q;
                const u32 g7 = q & 0x80808080u, gm = (g7 >> 4) - (g7 >> 7);
                u32 c = (q + 0x01010101u) & gm & 0x03030303u;
                k[i] = (K7 - ex) & ~gm & 0x07070707u;
                const u32 e = ~(c | (c >> 1)) & 0x01010101u;
                const u32 t = e + (e << 8);
                v[i] = t + (t << 16);
                const u32 u = v[i] + Cb;
                if (i >= 2) { const u32 ov = e & ((u + 0x77777777u) >> 7); c |= ov; k[i] &= ~((ov << 3) - ov); }
                Cb = __builtin_amdgcn_perm(u, u, 0x03030303u);
                Cw[h] |= c << (2 * i);
                const u32 pc = __builtin_amdgcn_perm(0u, LC, c);
                const u32 pk = __builtin_amdgcn_perm(LKhi, LKlo, k[i] | (~pc & 0x80808080u));
                mk[j] = (x[j] ^ (pk & pc)) & 0x7C7C7C7Cu;
                const u32 nz = __builtin_amdgcn_udot4((mk[j] + 0x7C7C7C7Cu) & 0x80808080u, 0x08040201u, 0u, false);
                bm |= j < 2 ? nz >> (7 - 4 * j) : nz << (4 * j - 7);
            }
            const bool w3 = ((k[0] | k[1] | k[2] | k[3]) & 0x04040404u) != 0;
            wide[h] = __ballot(w3);
            const u32 b = w3 ? 3 : 2, wlo = w3 ? 0x40080100u : 0x10040100u, whi = w3 ? 0u : 0x40u, m4 = w3 ? 0xE00u : 0u;
            u32 sh = 0;
            esc[h] = 0;
#pragma unroll
            for (int i = 0; i < 4; ++i) {
                const u32 four = (k[i] >> 15) & m4 & (u32)__builtin_amdgcn_sbfe((int)v[i], 26, 1);
                esc[h] |= __builtin_amdgcn_udot4(k[i], __builtin_amdgcn_perm(whi, wlo, v[i]), four, false) << (sh & 31);
                sh += b * (v[i] >> 24);
            }
            ne[h] = min(sh, 8 * b);
        }
        const u32 cnt = __builtin_popcount(bm), nb = ne[0] + ne[1];
        const u32 both = wave_scan_incl(nb | (cnt << 16)), tot = rdl(both, 63);
        const u32 ndw = ((tot & 0xFFFFu) + 31) >> 5, nex = tot >> 16, nov = nex > NENT ? nex - NENT : 0;
        const u32 need = ndw + (nov ? (nov + 2) >> 1 : 0);
        const bool raw = need > 192;
        u32 start = 0;
        if (lane == 0) start = __hip_atomic_fetch_add((LAS u32*)(L + L_CTL) + C_ALLOC, raw ? 192u : need, __ATOMIC_RELAXED, __HIP_MEMORY_SCOPE_WORKGROUP);
        char* blk = p.ws + WST_OFF + par * WST_PAR + (size_t)(cbe * 4 + kse) * BLK_G;
        u32x4 P0; u32 P1;
        P0.x = (x[0] & 0x83838383u) | (__builtin_amdgcn_alignbit(x[1], x[1], 29) & 0x1C1C1C1Cu) | ((x[6] << 5) & 0x60606060u);
        P0.y = (x[2] & 0x83838383u) | (__builtin_amdgcn_alignbit(x[3], x[3], 29) & 0x1C1C1C1Cu) | ((x[6] >> 1) & 0x40404040u) | ((x[7] >> 2) & 0x20202020u);
        P0.z = (x[4] & 0x83838383u) | (__builtin_amdgcn_alignbit(x[5], x[5], 29) & 0x1C1C1C1Cu) | ((x[7] << 5) & 0x60606060u);
        P0.w = Cw[0]; P1 = Cw[1];
        if (__builtin_expect(raw, 0)) { P0 = u32x4{x[0], x[1], x[2], x[3]}; P1 = x[4]; }
        st16_sys(blk + wave * FR_B + lane * 16, P0);
        st4_sys(blk + wave * FR_B + 1024 + lane * 4, P1);
        start = rfl(start);
        char* pool = blk + POOL_B + start * 4;
        u32 ev = 0;
        if (__builtin_expect(raw, 0)) {
            st4_sys(pool + lane * 4, x[5]); st4_sys(pool + 256 + lane * 4, x[6]); st4_sys(pool + 512 + lane * 4, x[7]);
        } else {
            const u32 bp = (both & 0xFFFFu) - nb;
            const u64 V = (u64)esc[0] | ((u64)esc[1] << ne[0]), U = V << (bp & 31);
            LAS u32* sp = (LAS u32*)(size_t)(sbuf + (bp >> 5) * 4);
            __hip_atomic_fetch_or(sp, (u32)U, __ATOMIC_RELAXED, __HIP_MEMORY_SCOPE_WORKGROUP);
            __hip_atomic_fetch_or(sp + 1, (u32)(U >> 32), __ATOMIC_RELAXED, __HIP_MEMORY_SCOPE_WORKGROUP);
            __hip_atomic_fetch_or(sp + 2, ((u32)(V >> 32) >> 1) >> (31 - (bp & 31)), __ATOMIC_RELAXED, __HIP_MEMORY_SCOPE_WORKGROUP);
            for (u32 i = lane; i < ndw; i += 64) st4_sys(pool + i * 4, LDS_AT(u32, sbuf + i * 4));
            u32 idx = (both >> 16) - cnt;
            while (__any(bm != 0)) {
                const u32 pos = __builtin_ctz(bm | 0x80000000u);
                const bool b0 = pos & 4, b1 = pos & 8, b2 = pos & 16;
                const u32 m01 = b0 ? mk[1] : mk[0], m23 = b0 ? mk[3] : mk[2], m45 = b0 ? mk[5] : mk[4], m67 = b0 ? mk[7] : mk[6];
                const u32 m03 = b1 ? m23 : m01, m47 = b1 ? m67 : m45, m = b2 ? m47 : m03;
                const u32 en = (pos >> 4) * 1024 + lane * 16 + (pos & 15) + (((m >> (8 * (pos & 3) + 2)) & 31u) << 11);
                ev |= (u32)__builtin_amdgcn_ds_permute((int)((bm != 0 && idx < NENT ? idx + 10 : 63u) * 4), (int)en);
                if (__builtin_expect(bm != 0 && idx >= NENT, 0)) st2_sys(pool + ndw * 4 + (idx - NENT + 1) * 2, en);
                ++idx; bm &= bm - 1;
            }
            if (nov && lane == 0) st2_sys(pool + ndw * 4, nov);
        }
        if (lane < 8) ev = (u32)((lane < 4 ? wide[0] : wide[1]) >> (16 * (lane & 3)));
        if (lane == 8) ev = (u32)T0 | (raw || nov ? 32u : 0u) | (raw ? 64u : 0u);
        if (lane == 9) ev = start;
        if (lane < 16) st2_sys(blk + MAIN_B + wave * ENT_B + lane * 2, ev);
        LDS_AT(u32x4, amb + lane * 16) = u32x4{0, 0, 0, 0};
        LDS_AT(u32x4, amb + 1024 + lane * 16) = u32x4{0, 0, 0, 0};
        WAIT_VM(0);
        u32 a2 = 0;
        if (lane == 0) a2 = __hip_atomic_fetch_add((LAS u32*)(L + L_CTL) + C_ARR, 1u, __ATOMIC_RELAXED, __HIP_MEMORY_SCOPE_WORKGROUP);
        if (rfl(a2) == 3) {
            const u32 n16 = MIN16 + ((ctl[C_ALLOC] + 3) >> 2);
            if (lane < NR && lane != rank) {
                GAS u64* f = (GAS u64*)(fpeer + WFL_OFF + par * WFL_PAR) + ((cbe * 8 + rank) * 4 + kse);
                __hip_atomic_store(f, ((u64)call << 32) | n16, __ATOMIC_RELAXED, __HIP_MEMORY_SCOPE_SYSTEM);
            }
            ctl[C_PUB] = 1;
        }
        } else {
            LDS_AT(u32x4, amb + lane * 16) = u32x4{0, 0, 0, 0};
            LDS_AT(u32x4, amb + 1024 + lane * 16) = u32x4{0, 0, 0, 0};
            WG_BARRIER(call);
            call = call + 1;
            scale = p.sx[0] * p.sw[0];
            asm volatile("" : "+v"(scale));
        }

        const int fw = wave & 3, grp = wave >> 2;
        f32x4 acc[8];
#pragma unroll
        for (int f = 0; f < 8; ++f) acc[f] = f32x4{0, 0, 0, 0};
        int avw = 0, avx = 0;
        const u32 aq0 = lbase + L_W + fw * FR_B + lane * 16, aq1 = lbase + L_W + fw * FR_B + 1024 + lane * 4;
        const u32 awd = lbase + L_W + MAIN_B + fw * ENT_B, aen = awd + 16 + (lane < 8 ? lane : 0) * 2, axx = lbase + L_X + lane * 16;
        const u32 scr = lbase + L_CTL + 64 + lane;
        auto wait_for = [&](int i, int& av) {
            u64 t0 = 0;
            for (u32 it = 1;; ++it) {
                const u32x2 fl = LDS_AT(u32x2, lbase + L_CTL);
                avw = (int)rfl(fl.x) + 4; avx = (int)rfl(fl.y);
                if (i < av) break;
                if (__builtin_expect((it & 1023) == 0, 0)) {
                    const u64 t = rt();
                    if (t0 == 0) t0 = t;
                    if (t - t0 > 2 * DEADLINE) { ctl[C_FAIL] = 1; avw = avx = NSTEP; break; }
                }
                __builtin_amdgcn_s_sleep(1);
            }
            asm volatile("" ::: "memory");
        };
        auto mma8 = [&](int next, const i32x8& wf, const u32x4 (&xa)[16]) {
#pragma unroll
            for (int f = 0; f < 8; ++f) {
                const u32x4 a = xa[2 * f], b = xa[2 * f + 1];
                i32x8 xf;
                xf[0] = a.x; xf[1] = a.y; xf[2] = a.z; xf[3] = a.w; xf[4] = b.x; xf[5] = b.y; xf[6] = b.z; xf[7] = b.w;
                if (f == 1) {
                    asm volatile("" :: "v"(xa[15]), "v"(wf) : "memory");
                    ctl[C_FREE + wave] = next;
                    __builtin_amdgcn_sched_barrier(0);
                }
                acc[f] = __builtin_amdgcn_mfma_scale_f32_16x16x128_f8f6f4(wf, xf, acc[f], 1  , 0  , 0, 0, 0, 0);
            }
        };
        if (grp == NG - 1)
#pragma nounroll
        for (int i = 0; i < 4; ++i) {
            if (i >= avx) wait_for(i, avx);
            u32x2 t0, t1, t2, t3;
            const u32 a = lbase + L_W + i * BLK_B + fw * 16 + (u32)(((lane >> 4) * 16 + ((lane & 15) >> 1)) * 64 + (lane & 1) * 8);
            asm volatile("ds_read_b64_tr_b8 %0, %4\n\tds_read_b64_tr_b8 %1, %4 offset:512\n\t"
                         "ds_read_b64_tr_b8 %2, %4 offset:4096\n\tds_read_b64_tr_b8 %3, %4 offset:4608\n\ts_waitcnt lgkmcnt(0)"
                         : "=&v"(t0), "=&v"(t1), "=&v"(t2), "=&v"(t3) : "v"(a) : "memory");
            u32x4 xa[16];
#pragma unroll
            for (int f = 0; f < 16; ++f) xa[f] = LDS_AT(u32x4, axx + i * XS_B + f * 1024);
            __builtin_amdgcn_sched_barrier(0);
            mma8(i < 3 ? i + 1 : 3 + NG, i32x8{(int)t0.x, (int)t0.y, (int)t1.x, (int)t1.y, (int)t2.x, (int)t2.y, (int)t3.x, (int)t3.y}, xa);
        }
        u32 xo = grp * XS_B, wo = (4 + grp) * BLK_B;
#pragma nounroll
        for (int i = 4 + grp; i < NSTEP; i += NG) {
            if (i >= avx) wait_for(i, avx);
            u32x4 xa[16];
#pragma unroll
            for (int f = 0; f < 16; ++f) xa[f] = LDS_AT(u32x4, axx + xo + f * 1024);
            if (i >= avw) { __builtin_amdgcn_s_setprio(0); wait_for(i, avw); __builtin_amdgcn_s_setprio(PRIO_C); }
            const u32 e0 = LDS_AT(unsigned short, aen + wo);
            const u32x4 q0 = LDS_AT(u32x4, aq0 + wo);
            const u32 q1 = LDS_AT(u32, aq1 + wo);
            const u32x4 wd = LDS_AT(u32x4, awd + wo);
            __builtin_amdgcn_sched_barrier(0);
            const u32 hdr = rfl(e0), pool = lbase + L_W + POOL_B + wo + (rdl(e0, 1) & 1023u) * 4;
            const u32 ea = e0 >> 11 ? amb + (e0 & 2047u) : scr;
            LDS_AT(unsigned char, ea) = (unsigned char)(e0 >> 11 << 2);
            const bool w3[2] = {__builtin_amdgcn_inverse_ballot_w64(((u64)rfl(wd.y) << 32) | rfl(wd.x)),
                                __builtin_amdgcn_inverse_ballot_w64(((u64)rfl(wd.w) << 32) | rfl(wd.z))};
            const u32 cw[2] = {w3[0] ? 3u : 2u, w3[1] ? 3u : 2u};
            const u32 n0 = cw[0] * __builtin_popcount(~(q0.w | (q0.w >> 1)) & 0x55555555u), nn = n0 + cw[1] * __builtin_popcount(~(q1 | (q1 >> 1)) & 0x55555555u);
            const u32 incl = wave_scan_incl(nn), bp = incl - nn;
            const u32 sa = pool + (bp >> 5) * 4;
            const u32 d0 = LDS_AT(u32, sa), d1 = LDS_AT(u32, sa + 4), d2 = LDS_AT(u32, sa + 8);
            if (__builtin_expect((hdr & 96u) == 32u, 0)) {
                const u32 ov = pool + ((rdl(incl, 63) + 31) >> 5) * 4;
                const u32 n = rfl(LDS_AT(unsigned short, ov)) & 511u;
                for (u32 k = 0; k < n; k += 64) {
                    const u32 e = LDS_AT(unsigned short, ov + 2 + (k + lane) * 2);
                    if (k + lane < n) LDS_AT(unsigned char, amb + (e & 2047u)) = (unsigned char)(e >> 11 << 2);
                }
                __builtin_amdgcn_s_waitcnt(0xC07F);
            }
            const u32x4 m0 = LDS_AT(u32x4, amb + lane * 16);
            const u32x4 m1 = LDS_AT(u32x4, amb + 1024 + lane * 16);
            LDS_AT(unsigned char, ea) = 0;
            if (__builtin_expect(hdr & 32u, 0)) { LDS_AT(u32x4, amb + lane * 16) = u32x4{0, 0, 0, 0}; LDS_AT(u32x4, amb + 1024 + lane * 16) = u32x4{0, 0, 0, 0}; }
            __builtin_amdgcn_sched_barrier(0);
            const u32 T4 = (hdr & 31u) * 0x04040404u;
            const u32 LC = (T4 + 0x24201C00u) | 0xFFu, LKlo = T4 + 0x10141828u, LKhi = T4 + 0x0004080Cu;
            u32 pc[8], sel[8];
#pragma unroll
            for (int h = 0; h < 2; ++h) {
                const u32 C = h ? q1 : q0.w;
                u32 Cb = 0x07070707u;
#pragma unroll
                for (int i = 0; i < 4; ++i) {
                    const int j = 4 * h + i;
                    pc[j] = __builtin_amdgcn_perm(0u, LC, (C >> (2 * i)) & 0x03030303u);
                    const u32 e = pc[j] & 0x01010101u, t = e + (e << 8), u = (t << 16) + (t + Cb);
                    sel[j] = __builtin_amdgcn_bitop3_b32(u, pc[j], 0x07070707u, 0xB1);
                    Cb = __builtin_amdgcn_perm(u, u, 0x03030303u);
                }
            }
            const u32 w0 = __builtin_amdgcn_alignbit(d1, d0, bp), w1 = __builtin_amdgcn_alignbit(d2, d1, bp);
            i32x8 wf;
#pragma unroll
            for (int h = 0; h < 2; ++h) {
                const u32 w = h ? __builtin_amdgcn_alignbit(w1, w0, n0) : w0;
                const u32 c4 = 4 * cw[h], s1 = w3[h] ? 10u : 12u, s2 = 8 - cw[h];
                const u32 k1 = w3[h] ? 0x003F003Fu : 0x000F000Fu, k2 = w3[h] ? 0x07070707u : 0x03030303u;
                const u32 wa = __builtin_amdgcn_ubfe(w, 0u, c4), wb = __builtin_amdgcn_ubfe(w, c4, c4);
                const u32 ya = (wa | (wa << s1)) & k1, yb = (wb | (wb << s1)) & k1;
                const u32 Ka = __builtin_amdgcn_perm(LKhi, LKlo, (ya | (ya << s2)) & k2);
                const u32 Kb = __builtin_amdgcn_perm(LKhi, LKlo, (yb | (yb << s2)) & k2);
                const u32x4 m = h ? m1 : m0;
#pragma unroll
                for (int i = 0; i < 4; ++i) wf[4 * h + i] = (__builtin_amdgcn_perm(Kb, Ka, sel[4 * h + i]) & pc[4 * h + i]) ^ m[i];
            }
            wf[0] |= q0.x & 0x83838383u; wf[1] |= __builtin_amdgcn_alignbit(q0.x, q0.x, 3) & 0x83838383u;
            wf[2] |= q0.y & 0x83838383u; wf[3] |= __builtin_amdgcn_alignbit(q0.y, q0.y, 3) & 0x83838383u;
            wf[4] |= q0.z & 0x83838383u; wf[5] |= __builtin_amdgcn_alignbit(q0.z, q0.z, 3) & 0x83838383u;
            wf[6] |= ((q0.x >> 5) & 0x03030303u) | ((q0.y << 1) & 0x80808080u);
            wf[7] |= ((q0.z >> 5) & 0x03030303u) | ((q0.y << 2) & 0x80808080u);
            if (__builtin_expect(hdr & 64u, 0)) {
                wf[0] = q0.x; wf[1] = q0.y; wf[2] = q0.z; wf[3] = q0.w; wf[4] = q1;
                wf[5] = LDS_AT(u32, pool + lane * 4); wf[6] = LDS_AT(u32, pool + 256 + lane * 4); wf[7] = LDS_AT(u32, pool + 512 + lane * 4);
                __builtin_amdgcn_s_waitcnt(0xC07F);
            }
            __builtin_amdgcn_sched_barrier(0);
            if (i >= NSTEP - NG)
                for (u32 it = 0; min_free() < NSTEP - 2 && it < (1u << 26); ++it) __builtin_amdgcn_s_sleep(0);
            mma8(i + NG, wf, xa);
            xo = (xo + NG * XS_B) & (NSX * XS_B - 1);
            wo = wo + NG * BLK_B >= NSW * BLK_B ? wo + NG * BLK_B - NSW * BLK_B : wo + NG * BLK_B;
        }
        constexpr u32 T_0 = L_X, T_1 = L_W + 5 * BLK_B;
        const u32 tw = lbase + (grp ? T_1 : T_0) + (lane & 15) * 256 + (((fw * 4 + (lane >> 4)) ^ (lane & 15)) * 16);
#pragma unroll
        for (int f = 0; f < 8; ++f) LDS_AT(f32x4, tw + f * 4096) = acc[f];
        WAIT_LGKM0();
        ctl[C_FREE + wave] = 2 * NSTEP;
        __builtin_amdgcn_s_setprio(0);
        for (u32 it = 0; min_free() < 2 * NSTEP && it < (1u << 24); ++it) __builtin_amdgcn_s_sleep(ESLEEP);
        const float sc = ctl[C_FAIL] != 0 ? __builtin_nanf("") : scale;
        constexpr int RW = 32 / NG;
        float* o = p.out + (size_t)(rb * 64 + wave * RW + (lane >> 4)) * LDO + cb * 64 + (lane & 15) * 4;
        f32x4 y[RW / 4];
#pragma unroll
        for (int f = 0; f < RW / 4; ++f) {
            const u32 tr = lbase + (wave * RW + f * 4 + (lane >> 4)) * 256 + (((lane & 15) ^ ((f * 4 + (lane >> 4)) & 15)) * 16);
            y[f] = LDS_AT(f32x4, tr + T_0);
            if (NG == 2) y[f] += LDS_AT(f32x4, tr + T_1);
        }
#pragma unroll
        for (int f = 0; f < RW / 4; ++f)
            st16_sys(o + (size_t)f * 4 * LDO, __builtin_bit_cast(u32x4, __builtin_elementwise_maximum(y[f] * sc, f32x4{0, 0, 0, 0})));
    } else if (wave == 8) {
        WG_BARRIER(call);
        call = call + 1;
        par = call & 1;
        const int R = lane < NREM ? lane : 0, src = src_of_rem(R, rank, cb), ks = R / 7;
        const u64 baddr = (((u64)(u32)__shfl((int)((u64)fpeer >> 32), src) << 32) | (u32)__shfl((int)(u32)(u64)fpeer, src)) + WST_OFF + par * WST_PAR + (u64)(cb * 4 + ks) * BLK_G;
        const u32 blo = (u32)baddr, bhi = (u32)(baddr >> 32);
        GAS u64* fl = (GAS u64*)(p.ws + WFL_OFF + par * WFL_PAR) + ((cb * 8 + src) * 4 + ks);
        const u32 voff = lane * 16;
        u32 n16v = MIN16;
        bool dead = false;
        const u32 xcc = __builtin_amdgcn_s_getreg((3 << 11) | 20) & 15;
        GAS u64* inv = (GAS u64*)(p.ws + INV_OFF + xcc * 256);
        u32 prev = 0;
        if (lane == 0) prev = __hip_atomic_fetch_max((GAS u32*)(inv + 16), call, __ATOMIC_RELAXED, __HIP_MEMORY_SCOPE_AGENT);
        if (rfl(prev) < call) {
            __builtin_amdgcn_fence(__ATOMIC_ACQUIRE, "");
            WAIT_VM(0);
            if (lane == 0) __hip_atomic_store(inv, (u64)call << 32, __ATOMIC_RELAXED, __HIP_MEMORY_SCOPE_AGENT);
        }
        if (lane >= 28 && lane < 56) fl = (GAS u64*)(p.ws + XFL_OFF + par * XFL_PAR) + ((rb * 8 + src_of_rem(lane, rank, 0)) * 4 + (lane - 28) / 7);
        if (lane == 62) fl = (GAS u64*)(p.ws + TMO_OFF);
        if (lane == 63) fl = inv;
        for (u32 it = 0; rfl(ctl[C_PUB]) == 0 && it < (1u << 22); ++it) __builtin_amdgcn_s_sleep(4);
        if (PDELAY) __builtin_amdgcn_s_sleep(PDELAY);
        {
            const u64 t0 = rt();
            for (u32 it = 1;; ++it) {
                const u64 v = __hip_atomic_load(fl, __ATOMIC_RELAXED, __HIP_MEMORY_SCOPE_SYSTEM);
                n16v = (u32)v;
                const u64 up = __ballot((u32)(v >> 32) >= call);
                if (((up >> 62) & 1) && rdl((u32)(v >> 32), 62) == call) { dead = true; break; }
                if ((up | 0x7FFFFFFFF0000000ull) == ~0ull && (__builtin_popcount((u32)(up >> 28) & 0xFFFFFFFu) >= XTH || rt() - t0 > 2000)) break;
                if ((it & 15) == 0 && expired(t0)) { dead = true; break; }
                __builtin_amdgcn_s_sleep(PSLEEP);
            }
            n16v = n16v < MIN16 ? MIN16 : (n16v > MAX16 ? MAX16 : n16v);
        }
        __builtin_amdgcn_s_setprio(PRIO_W);
        int pub = 0, freed = 0;
        u32 lo = lbase + L_W + 4 * BLK_B;
        auto publish = [&](int upto) { if (upto > pub) { pub = upto; ctl[C_FULLW] = upto; } };
        u64 g[LAST + 1]; u32 l[LAST + 1];
#pragma unroll
        for (int j = 0; j <= LAST; ++j) { g[j] = 0; l[j] = 0; }
#define W_LOAD(OFF, J, K, NOP) do { dma16<OFF, 0, NOP>(g[J], voff, l[J]); WAIT_VM(WIN); after(std::integral_constant<int, K>{}); } while (0)
        auto rest_of_slot = [&](int n, bool mid, auto&& after) {
            if (mid) { W_LOAD(2048, LAG, 3, 0); W_LOAD(3072, LAG, 4, 0); }
            if (n >= LAST) {
                g[LAST] += 4096; l[LAST] += 4096;
                W_LOAD(0, LAST, 5, 0);
                const u32 n16 = rdl(n16v, n - LAST);
                if (__builtin_expect(n16 > 64, 0))
                    for (u32 i = 64; i < n16; i += 64) { if (lane + i < n16) dma16<1024, 0>(g[LAST] + i * 16, voff, l[LAST] + i * 16); WAIT_VM(WIN); }
                if ((u32)lane < n16) dma16<1024, 0>(g[LAST], voff, l[LAST]);
                WAIT_VM(WIN); after(std::integral_constant<int, SL>{});
            }
#pragma unroll
            for (int j = LAST; j > 0; --j) { g[j] = g[j - 1]; l[j] = l[j - 1]; }
        };
        auto open = [&](int n, int done) {
            if (freed <= n + 4 - NSW) {
                const int need = n + 4 - NSW;
                const u64 t0 = rt();
                bool drained = false;
                for (;;) {
                    freed = min_free();
                    if (freed > need) break;
                    if (!drained) { WAIT_VM(0); publish(done); drained = true; continue; }
                    if (rt() - t0 > 2 * DEADLINE) { dead = true; break; }
                    __builtin_amdgcn_s_sleep(1);
                }
                if (dead) return;
            }
            g[0] = ((u64)rdl(bhi, n) << 32) | rdl(blo, n);
            l[0] = lo;
            lo = lo + BLK_B == lbase + L_W + NSW * BLK_B ? lbase + L_W : lo + BLK_B;
        };
        for (int n = 0; n < NB; ++n) {
            constexpr int PK = (WIN - 1) % SL + 1, PL = LAST + 1 + (WIN - PK) / SL;
            auto after = [&](auto k) { if constexpr (decltype(k)::value == PK) publish(n - PL + 1); };
            if (!dead) open(n, n - LAST);
            if (dead) break;
            W_LOAD(0, 0, 1, 4); W_LOAD(1024, 0, 2, 0);
            rest_of_slot(n, n >= LAG, after);
        }
        if (!dead) {
            static_for<LAST>([&](auto dd) {
                constexpr int d = decltype(dd)::value;
                rest_of_slot(NB + d, d < LAG, [&](auto k) { publish(std::integral_constant<int, w_landed(w_cum(d) - (SL - decltype(k)::value), WIN)>::value); });
            });
            static_for<NU>([&](auto uu) {
                constexpr int u = decltype(uu)::value;
                auto after = [&](auto k) { publish(std::integral_constant<int, w_landed(w_cum(LAST + u) - (SL - decltype(k)::value), WIN)>::value); };
                if (!dead) open(NB + u, NB + u);
                if (dead) return;
                W_LOAD(0, 0, 1, 4); W_LOAD(1024, 0, 2, 0);
                g[LAG] = g[LAST] = g[0]; l[LAG] = l[LAST] = l[0];
                rest_of_slot(NB + u + LAST, true, after);
            });
        }
        if (dead) { ctl[C_FAIL] = 1; WAIT_VM(0); ctl[C_FULLW] = NREM; }
        else
            static_for<8>([&](auto dd) {
                constexpr int upto = NREM - 7 + decltype(dd)::value, behind = w_behind(upto - 1, w_cum(LAST + NU - 1));
                if constexpr (behind < WIN) { WAIT_VM(behind); publish(upto); }
            });
    } else if (wave == 9) {
        WG_BARRIER(call);
        call = call + 1;
        par = call & 1;
        const int R = max((lane & 31) - 4, 0), src = src_of_rem(R, rank, cb), ks = R / 7;
        const u32 toff = (u32)((src * 4 + ks) * 8 + rb) * XT_B;
        const u64 xbase = (u64)p.ws + XRV_OFF + par * XRV_PAR;
        GAS u64* fl = (GAS u64*)(p.ws + XFL_OFF + par * XFL_PAR) + (((rb + (lane >> 5)) * 8 + src) * 4 + ks);
        constexpr u32 OWN = 0xFu;
        const u32 voff = lane * 16;
        const u32 xoff = (u32)(lane & 15) * LDX + (lane >> 4) * 16;
        u32 ready = OWN;
        u32 it = 0, la = lbase + L_X;
        bool dead = false;
        const u64 t0 = rt();
        int pub = 0, freed = 0;
        for (u32 k = 0; rfl(ctl[C_PUB]) == 0 && k < (1u << 22); ++k) __builtin_amdgcn_s_sleep(4);
        for (int s = 0; s < NSTEP && !dead; ++s) {
            if (!((ready >> s) & 1)) {
                WAIT_VM(0);
                if (pub < s) { pub = s; ctl[C_FULLX] = s; }
                for (;;) {
                    const u64 v = __hip_atomic_load(fl, __ATOMIC_RELAXED, __HIP_MEMORY_SCOPE_SYSTEM);
                    const u64 ok = __ballot((u32)(v >> 32) >= call);
                    ready = ((u32)ok & (u32)(ok >> 32)) | OWN;
                    if (ready == ~0u) { dead = rfl(__hip_atomic_load(tmo, __ATOMIC_RELAXED, __HIP_MEMORY_SCOPE_AGENT)) == call; break; }
                    if ((ready >> s) & 1) break;
                    if ((++it & 15) == 0 && expired(t0)) { dead = true; break; }
                    __builtin_amdgcn_s_sleep(XSLEEP);
                }
            }
            while (!dead && freed <= s - NSX) {
                freed = min_free();
                if (freed > s - NSX) break;
                if ((++it & 1023) == 0 && rt() - t0 > 2 * DEADLINE) dead = true;
                __builtin_amdgcn_s_sleep(1);
            }
            if (dead) break;
            if (s < 4) {
                const u64 b = (u64)p.x + (u64)(rank * 512 + rb * 64) * LDX + s * 128;
#pragma unroll
                for (int i = 0; i < 16; ++i) dma16<0, 0>(b + (i >> 1) * 16 * LDX + (i & 1) * 64, xoff, la + i * 1024);
#pragma unroll
                for (int i = 0; i < 8; ++i)
                    dma16<0, 0>((u64)p.w + (u64)(s * 128 + i * 16) * LDW + cb * 64, (u32)(lane >> 2) * LDW + (lane & 3) * 16, lbase + L_W + s * BLK_B + i * 1024);
            } else {
                const u64 b = xbase + rdl(toff, s);
#pragma unroll
                for (int i = 0; i < 4; ++i) {
                    dma16<0, 1, 4>(b + i * 4096, voff, la + i * 4096); dma16<1024, 1>(b + i * 4096, voff, la + i * 4096);
                    dma16<2048, 1>(b + i * 4096, voff, la + i * 4096); dma16<3072, 1>(b + i * 4096, voff, la + i * 4096);
                }
            }
            la = la + XS_B == lbase + L_X + NSX * XS_B ? lbase + L_X : la + XS_B;
            WAIT_VM(16 * (XD - 1));
            if (pub < s - XD + 2) { pub = s - XD + 2; ctl[C_FULLX] = pub; }
        }
        if (dead) { ctl[C_FAIL] = 1; WAIT_VM(0); }
        else {
            WAIT_VM(32); ctl[C_FULLX] = max(pub, NSTEP - 2);
            WAIT_VM(16); ctl[C_FULLX] = max(pub, NSTEP - 1);
            WAIT_VM(0);
        }
        ctl[C_FULLX] = NSTEP;
    } else {
        const int id = bid / 7, d = (rank + 1 + bid % 7) & 7;
        u32x4 v[8];
        if (id < 32) {
            const unsigned char* g = p.x + (size_t)(d * 512 + (id & 7) * 64 + (lane & 15)) * LDX + (id >> 3) * 128 + (lane >> 4) * 16;
#pragma unroll
            for (int i = 0; i < 8; ++i) v[i] = *(const GAS u32x4*)(g + (size_t)(i >> 1) * 16 * LDX + (i & 1) * 64);
        }
        WG_BARRIER(call);
        call = call + 1;
        par = call & 1;
        if (id < 32) {
            asm volatile("" : "+v"(v[0]), "+v"(v[1]), "+v"(v[2]), "+v"(v[3]), "+v"(v[4]), "+v"(v[5]), "+v"(v[6]), "+v"(v[7]), "+v"(fpeer));
            char* peer = (char*)(((u64)rdl((u32)((u64)fpeer >> 32), d) << 32) | rdl((u32)(u64)fpeer, d));
            char* dst = peer + XRV_OFF + par * XRV_PAR + (size_t)(rank * 32 + id) * XT_B + lane * 16;
#pragma unroll
            for (int i = 0; i < 8; ++i) { st16_sys(dst + i * 1024, v[i]); WAIT_VM(PUSH_W - 1); }
            WAIT_VM(0);
            if (lane == 0)
                __hip_atomic_store((GAS u64*)(peer + XFL_OFF + par * XFL_PAR) + (((id & 7) * 8 + rank) * 4 + (id >> 3)), (u64)call << 32,
                                   __ATOMIC_RELAXED, __HIP_MEMORY_SCOPE_SYSTEM);
        }
    }
    if (lane == 0) __hip_atomic_store(cntp, call, __ATOMIC_RELAXED, __HIP_MEMORY_SCOPE_AGENT);
}

extern "C" void kernel_launch(int rank, int nranks, ncclComm_t comm,
                              hipStream_t stream,
                              void* const* d_in, const int* in_sizes, int n_in,
                              void* d_out, int out_size,
                              const int* in_dtypes, int out_dtype,
                              void* const* d_out_all,
                              void* const* d_ws_all, size_t ws_bytes) {
    (void)comm; (void)in_dtypes; (void)out_dtype; (void)d_out_all;
    if (nranks != NR || (unsigned)rank >= (unsigned)NR || n_in != 4 || ws_bytes < WS_NEED ||
        in_sizes[0] != 4096 * 512 || in_sizes[1] != 512 * 4096 || out_size != 512 * 4096) {
        fprintf(stderr, "unexpected problem shape\n");
        return;
    }
    HIP_CHECK(hipFuncSetAttribute((const void*)gemm_xchg, hipFuncAttributeMaxDynamicSharedMemorySize, LDS_BYTES));
    Params p;
    p.x = (const unsigned char*)d_in[0];
    p.w = (const unsigned char*)d_in[1];
    p.sx = (const float*)d_in[2];
    p.sw = (const float*)d_in[3];
    p.out = (float*)d_out;
    p.ws_all = d_ws_all;
    p.ws = (char*)d_ws_all[rank];
    p.rank = rank;
    p.pad = 0;
    hipLaunchKernelGGL(gemm_xchg, dim3(NWG), dim3(NTHR), LDS_BYTES, stream, p);
    HIP_CHECK(hipGetLastError());
}
```

```cpp
#include <hip/hip_runtime.h>
#include <rccl/rccl.h>
#include <cstdio>

#define HIP_CHECK(expr) do { hipError_t _e = (expr); if (_e != hipSuccess) { \
    fprintf(stderr, "HIP error %s at %s:%d\n", hipGetErrorString(_e), __FILE__, __LINE__); \
    return; } } while (0)

typedef unsigned u32;
typedef unsigned long long u64;
typedef unsigned u32x4 __attribute__((ext_vector_type(4)));
typedef unsigned u32x2 __attribute__((ext_vector_type(2)));
typedef int i32x8 __attribute__((ext_vector_type(8)));
typedef float f32x4 __attribute__((ext_vector_type(4)));
#define GAS __attribute__((address_space(1)))
#define LAS __attribute__((address_space(3)))

constexpr int WIN = 4;
constexpr int NU = 10;
constexpr int ESLEEP = 1;
constexpr int PRIO_C = 1;
constexpr int NG = 2;
constexpr int PSLEEP = 4;
constexpr int PTIME = 500;
constexpr int XSLEEP = 16;
constexpr int PRIO_W = 3;
constexpr int XTH = 5;
constexpr int PUSH_W = 2;
constexpr int XD = 2;
constexpr int NSW = 9;
constexpr int NSX = 4;

constexpr int NR = 8;
constexpr int NWG = 256, NTHR = 64 * (4 * NG + 3);
constexpr int LDX = 512, LDW = 4096, LDO = 4096;
constexpr int MAIN_B = 5120, FR_B = 1280, ENT_B = 32, NENT = 6, POOL_B = MAIN_B + 4 * ENT_B;
constexpr int MIN16 = 4 * ENT_B / 16, MAX16 = (4 * ENT_B + 4 * 768) / 16;
constexpr int BLK_B = 8448;
constexpr int BLK_G = 8448;
constexpr int XT_B = 8192;
constexpr int XS_B = 2 * XT_B;
constexpr int NSTEP = 32, NREM = 28;
static_assert(MAIN_B + MAX16 * 16 + 16 <= BLK_B && BLK_B <= BLK_G, "block");

constexpr size_t WST_OFF = 0, WST_PAR = 4u << 20;
constexpr size_t XRV_OFF = 8u << 20, XRV_PAR = 2u << 20;
constexpr size_t WFL_OFF = 12u << 20, WFL_PAR = 16u << 10;
constexpr size_t XFL_OFF = WFL_OFF + (32u << 10), XFL_PAR = 4u << 10;
constexpr size_t CNT_OFF = WFL_OFF + (64u << 10);
constexpr size_t TMO_OFF = WFL_OFF + (128u << 10);
constexpr size_t INV_OFF = TMO_OFF + 1024;
constexpr size_t WS_NEED = 13u << 20;
static_assert((size_t)256 * BLK_G <= WST_PAR, "w staging");

constexpr int L_W = 0;
constexpr int L_X = L_W + NSW * BLK_B;
constexpr int L_MASK = L_X + NSX * XS_B;
constexpr int L_CTL = L_MASK + 16384;
constexpr int LDS_BYTES = L_CTL + 128;
static_assert(NSW == 9 && NSX == 4, "the output staging areas are chosen for these rings");
static_assert(LDS_BYTES <= 160 * 1024 && NSW >= 8 && XD <= NSX && XD <= 4, "LDS");
constexpr int C_FULLW = 0, C_FULLX = 1, C_FAIL = 2, C_ALLOC = 3, C_ARR = 4, C_PUB = 5, C_FREE = 8  ;

constexpr u64 DEADLINE = 30000000ull;

struct Params {
    const unsigned char* x;
    const unsigned char* w;
    const float* sx;
    const float* sw;
    float* out;
    void* const* ws_all;
    char* ws;
    int rank;
    int pad;
};

__device__ __forceinline__ u64 rt() { return __builtin_amdgcn_s_memrealtime(); }
__device__ __forceinline__ u32 rfl(u32 v) { return (u32)__builtin_amdgcn_readfirstlane(v); }
__device__ __forceinline__ u32 rdl(u32 v, int l) { return (u32)__builtin_amdgcn_readlane(v, l); }
#define WAIT_VM(n) asm volatile("s_waitcnt vmcnt(%0)" :: "i"(n) : "memory")
#define WG_BARRIER(call) do { u32 c_; asm volatile("s_barrier" ::: "memory"); \
    asm volatile("v_readfirstlane_b32 %0, %1\n\ts_nop 4" : "=s"(c_) : "v"(call) : "memory"); call = c_; } while (0)
#define WAIT_LGKM0() asm volatile("s_waitcnt lgkmcnt(0)" ::: "memory")

template <int OFF, int POL, int NOP = 0>
__device__ __forceinline__ void dma16(u64 base, u32 voff, u32 ldsaddr) {
    if constexpr (POL == 1)
        asm volatile("s_mov_b32 m0, %2\n\ts_nop %4\n\tglobal_load_lds_dwordx4 %0, %1 offset:%3 sc0 sc1"
                     :: "v"(voff), "s"(base), "s"(ldsaddr), "i"(OFF), "i"(NOP) : "memory");
    else
        asm volatile("s_mov_b32 m0, %2\n\ts_nop %4\n\tglobal_load_lds_dwordx4 %0, %1 offset:%3"
                     :: "v"(voff), "s"(base), "s"(ldsaddr), "i"(OFF), "i"(NOP) : "memory");
}
__device__ __forceinline__ void st4_sys(void* p, u32 v) {
    asm volatile("global_store_dword %0, %1, off sc0 sc1\n\ts_nop 1" :: "v"(p), "v"(v) : "memory");
}
__device__ __forceinline__ void st2_sys(void* p, u32 v) {
    asm volatile("global_store_short %0, %1, off sc0 sc1\n\ts_nop 1" :: "v"(p), "v"(v) : "memory");
}
__device__ __forceinline__ void st16_sys(void* p, u32x4 v) {
    asm volatile("global_store_dwordx4 %0, %1, off sc0 sc1\n\ts_nop 1" :: "v"(p), "v"(v) : "memory");
}
__device__ __forceinline__ u32 wave_scan_incl(u32 v) {
    v += __builtin_amdgcn_update_dpp(0, v, 0x111, 0xf, 0xf, true);
    v += __builtin_amdgcn_update_dpp(0, v, 0x112, 0xf, 0xf, true);
    v += __builtin_amdgcn_update_dpp(0, v, 0x114, 0xf, 0xf, true);
    v += __builtin_amdgcn_update_dpp(0, v, 0x118, 0xf, 0xf, true);
    v += __builtin_amdgcn_update_dpp(0, v, 0x142, 0xa, 0xf, false);
    v += __builtin_amdgcn_update_dpp(0, v, 0x143, 0xc, 0xf, false);
    return v;
}

__device__ __forceinline__ int src_of_rem(int R, int rank, int cb) { return (rank + 1 + (cb + R) % 7) & 7; }

constexpr int LAG = 2, LAST = 2 * LAG, SL = 6, NB = NREM - NU;
static_assert(WIN >= 1 && WIN <= SL && NSW > LAST + 2 && NU >= 1 && NB >= LAST, "window");
constexpr int w_cum(int d) { int c = 0; for (int t = 0; t <= d; ++t) c += t >= LAST ? SL : (t < LAG ? 4 : 2); return c; }
constexpr int w_behind(int b, int q) { return b >= NB - LAST ? q - w_cum(b - (NB - LAST)) : q + SL * (NB - LAST - 1 - b); }
constexpr int w_landed(int q, int win) { int u = 0; while (u < NREM && w_behind(u, q) >= win) ++u; return u; }
template <int N, int I = 0, class F> __device__ __forceinline__ void static_for(F&& f) {
    if constexpr (I < N) { f(std::integral_constant<int, I>{}); static_for<N, I + 1>(f); }
}

#define LDS_AT(T, a) (*(volatile LAS T*)(size_t)(a))

__global__ void __launch_bounds__(NTHR, 3) gemm_xchg(Params p) {
    extern __shared__ __attribute__((aligned(16))) unsigned char lds_raw[];
    LAS unsigned char* L = (LAS unsigned char*)lds_raw;
    const u32 lbase = (u32)(size_t)L;
    volatile LAS u32* ctl = (volatile LAS u32*)(L + L_CTL);
    const int tid = threadIdx.x, lane = tid & 63;
    const int wave = rfl(tid >> 6) == 0 ? 10 : (rfl(tid >> 6) <= 4 * NG ? rfl(tid >> 6) - 1 : rfl(tid >> 6) - 4 * NG + 7);
    const int bid = blockIdx.x, xc = bid & 7, jj = bid >> 3;
    const int rank = p.rank;
    const int rb = (jj & 3) * 2, cb = xc * 8 + (jj >> 2);
    GAS u32* tmo = (GAS u32*)(p.ws + TMO_OFF) + 1;

    GAS u32* cntp = (GAS u32*)(p.ws + CNT_OFF) + bid * 16 + wave;
    u32 call = __hip_atomic_load(cntp, __ATOMIC_RELAXED, __HIP_MEMORY_SCOPE_AGENT);
    char* fpeer = (char*)p.ws_all[lane & 7];
    u32 par;
    auto expired = [&](u64 t0) {
        if (rfl(__hip_atomic_load(tmo, __ATOMIC_RELAXED, __HIP_MEMORY_SCOPE_AGENT)) != call && rt() - t0 <= DEADLINE) return false;
        __hip_atomic_store(tmo, call, __ATOMIC_RELAXED, __HIP_MEMORY_SCOPE_AGENT);
        return true;
    };
    auto min_free = [&]() {
        const u32x4 a = LDS_AT(u32x4, lbase + L_CTL + C_FREE * 4), b = LDS_AT(u32x4, lbase + L_CTL + C_FREE * 4 + 16);
        return (int)rfl(min(min(min(a.x, a.y), min(a.z, a.w)), min(min(b.x, b.y), min(b.z, b.w))));
    };

    if (wave < 8) {
        const u32 amb = lbase + L_MASK + wave * 2048;
        float scale;
        if (wave < 4) {
        const int cbe = (jj & 1) + 2 * (xc + 8 * ((jj >> 1) & 3)), kse = jj >> 3;
        {
            const u64 src = (u64)p.w + (u64)(kse * 128) * LDW + cbe * 64 + wave * 16;
            dma16<0, 0, 4>(src, (u32)lane * LDW, amb);
            dma16<0, 0, 4>(src, (u32)(lane + 64) * LDW, amb + 1024);
        }
        if (wave == 0 && lane < 16) ctl[lane] = NG == 2 ? (lane >= C_FREE && lane < C_FREE + 4 ? 4 : 0) : (lane >= C_FREE + 4 ? 1000 : 0);
        WAIT_LGKM0();
        WG_BARRIER(call);
        scale = p.sx[0] * p.sw[0];
        asm volatile("" : "+v"(scale), "+v"(fpeer));
        call = call + 1;
        par = call & 1;
        WAIT_VM(0);
        u32 x[8];
        {
            u32x2 t0, t1, t2, t3;
            const u32 a = amb + (u32)(((lane >> 4) * 16 + ((lane & 15) >> 1)) * 16 + (lane & 1) * 8);
            asm volatile("ds_read_b64_tr_b8 %0, %4\n\tds_read_b64_tr_b8 %1, %4 offset:128\n\t"
                         "ds_read_b64_tr_b8 %2, %4 offset:1024\n\tds_read_b64_tr_b8 %3, %4 offset:1152\n\ts_waitcnt lgkmcnt(0)"
                         : "=&v"(t0), "=&v"(t1), "=&v"(t2), "=&v"(t3) : "v"(a) : "memory");
            x[0] = t0.x; x[1] = t0.y; x[2] = t1.x; x[3] = t1.y; x[4] = t2.x; x[5] = t2.y; x[6] = t3.x; x[7] = t3.y;
        }
        const u32 sbuf = lbase + L_W + (NSW - 1) * BLK_B + wave * 1024;
        LDS_AT(u32x4, sbuf + lane * 16) = u32x4{0, 0, 0, 0};
        u32 sum = 0;
#pragma unroll
        for (int i = 0; i < 8; ++i) sum = __builtin_amdgcn_sad_u8(x[i] & 0x7F7F7F7Fu, 0u, sum);
        sum = rdl(wave_scan_incl(sum), 63);
        int T0 = ((int)sum - 60211) >> 13;
        T0 = T0 < 0 ? 0 : (T0 > 21 ? 21 : T0);
        const u32 Q = (u32)(121 - T0) * 0x01010101u, T4 = (u32)T0 * 0x04040404u;
        const u32 LC = (T4 + 0x24201C00u) | 0xFFu, LKlo = T4 + 0x10141828u, LKhi = T4 + 0x0004080Cu, K7 = (u32)(T0 + 39) * 0x01010101u;
        u32 mk[8], Cw[2], esc[2], ne[2], bm = 0;
        u64 wide[2];
#pragma unroll
        for (int h = 0; h < 2; ++h) {
            u32 Cb = 0, k[4], v[4];
            Cw[h] = 0;
#pragma unroll
            for (int i = 0; i < 4; ++i) {
                const int j = 4 * h + i;
                const u32 ex = (x[j] >> 2) & 0x1F1F1F1Fu, q = ex + Q;
                const u32 g7 = q & 0x80808080u, gm = (g7 >> 4) - (g7 >> 7);
                u32 c = (q + 0x01010101u) & gm & 0x03030303u;
                k[i] = (K7 - ex) & ~gm & 0x07070707u;
                const u32 e = ~(c | (c >> 1)) & 0x01010101u;
                const u32 t = e + (e << 8);
                v[i] = t + (t << 16);
                const u32 u = v[i] + Cb;
                if (i >= 2) { const u32 ov = e & ((u + 0x77777777u) >> 7); c |= ov; k[i] &= ~((ov << 3) - ov); }
                Cb = __builtin_amdgcn_perm(u, u, 0x03030303u);
                Cw[h] |= c << (2 * i);
                const u32 pc = __builtin_amdgcn_perm(0u, LC, c);
                const u32 pk = __builtin_amdgcn_perm(LKhi, LKlo, k[i] | (~pc & 0x80808080u));
                mk[j] = (x[j] ^ (pk & pc)) & 0x7C7C7C7Cu;
                const u32 nz = __builtin_amdgcn_udot4((mk[j] + 0x7C7C7C7Cu) & 0x80808080u, 0x08040201u, 0u, false);
                bm |= j < 2 ? nz >> (7 - 4 * j) : nz << (4 * j - 7);
            }
            const bool w3 = ((k[0] | k[1] | k[2] | k[3]) & 0x04040404u) != 0;
            wide[h] = __ballot(w3);
            const u32 b = w3 ? 3 : 2, wlo = w3 ? 0x40080100u : 0x10040100u, whi = w3 ? 0u : 0x40u, m4 = w3 ? 0xE00u : 0u;
            u32 sh = 0;
            esc[h] = 0;
#pragma unroll
            for (int i = 0; i < 4; ++i) {
                const u32 four = (k[i] >> 15) & m4 & (u32)__builtin_amdgcn_sbfe((int)v[i], 26, 1);
                esc[h] |= __builtin_amdgcn_udot4(k[i], __builtin_amdgcn_perm(whi, wlo, v[i]), four, false) << (sh & 31);
                sh += b * (v[i] >> 24);
            }
            ne[h] = min(sh, 8 * b);
        }
        const u32 cnt = __builtin_popcount(bm), nb = ne[0] + ne[1];
        const u32 both = wave_scan_incl(nb | (cnt << 16)), tot = rdl(both, 63);
        const u32 ndw = ((tot & 0xFFFFu) + 31) >> 5, nex = tot >> 16, nov = nex > NENT ? nex - NENT : 0;
        const u32 need = ndw + (nov ? (nov + 2) >> 1 : 0);
        const bool raw = need > 192;
        u32 start = 0;
        if (lane == 0) start = __hip_atomic_fetch_add((LAS u32*)(L + L_CTL) + C_ALLOC, raw ? 192u : need, __ATOMIC_RELAXED, __HIP_MEMORY_SCOPE_WORKGROUP);
        char* blk = p.ws + WST_OFF + par * WST_PAR + (size_t)(cbe * 4 + kse) * BLK_G;
        u32x4 P0; u32 P1;
        P0.x = (x[0] & 0x83838383u) | (__builtin_amdgcn_alignbit(x[1], x[1], 29) & 0x1C1C1C1Cu) | ((x[6] << 5) & 0x60606060u);
        P0.y = (x[2] & 0x83838383u) | (__builtin_amdgcn_alignbit(x[3], x[3], 29) & 0x1C1C1C1Cu) | ((x[6] >> 1) & 0x40404040u) | ((x[7] >> 2) & 0x20202020u);
        P0.z = (x[4] & 0x83838383u) | (__builtin_amdgcn_alignbit(x[5], x[5], 29) & 0x1C1C1C1Cu) | ((x[7] << 5) & 0x60606060u);
        P0.w = Cw[0]; P1 = Cw[1];
        if (__builtin_expect(raw, 0)) { P0 = u32x4{x[0], x[1], x[2], x[3]}; P1 = x[4]; }
        st16_sys(blk + wave * FR_B + lane * 16, P0);
        st4_sys(blk + wave * FR_B + 1024 + lane * 4, P1);
        start = rfl(start);
        char* pool = blk + POOL_B + start * 4;
        u32 ev = 0;
        if (__builtin_expect(raw, 0)) {
            st4_sys(pool + lane * 4, x[5]); st4_sys(pool + 256 + lane * 4, x[6]); st4_sys(pool + 512 + lane * 4, x[7]);
        } else {
            const u32 bp = (both & 0xFFFFu) - nb;
            const u64 V = (u64)esc[0] | ((u64)esc[1] << ne[0]), U = V << (bp & 31);
            LAS u32* sp = (LAS u32*)(size_t)(sbuf + (bp >> 5) * 4);
            __hip_atomic_fetch_or(sp, (u32)U, __ATOMIC_RELAXED, __HIP_MEMORY_SCOPE_WORKGROUP);
            __hip_atomic_fetch_or(sp + 1, (u32)(U >> 32), __ATOMIC_RELAXED, __HIP_MEMORY_SCOPE_WORKGROUP);
            __hip_atomic_fetch_or(sp + 2, ((u32)(V >> 32) >> 1) >> (31 - (bp & 31)), __ATOMIC_RELAXED, __HIP_MEMORY_SCOPE_WORKGROUP);
            for (u32 i = lane; i < ndw; i += 64) st4_sys(pool + i * 4, LDS_AT(u32, sbuf + i * 4));
            u32 idx = (both >> 16) - cnt;
            while (__any(bm != 0)) {
                const u32 pos = __builtin_ctz(bm | 0x80000000u);
                const bool b0 = pos & 4, b1 = pos & 8, b2 = pos & 16;
                const u32 m01 = b0 ? mk[1] : mk[0], m23 = b0 ? mk[3] : mk[2], m45 = b0 ? mk[5] : mk[4], m67 = b0 ? mk[7] : mk[6];
                const u32 m03 = b1 ? m23 : m01, m47 = b1 ? m67 : m45, m = b2 ? m47 : m03;
                const u32 en = (pos >> 4) * 1024 + lane * 16 + (pos & 15) + (((m >> (8 * (pos & 3) + 2)) & 31u) << 11);
                ev |= (u32)__builtin_amdgcn_ds_permute((int)((bm != 0 && idx < NENT ? idx + 10 : 63u) * 4), (int)en);
                if (__builtin_expect(bm != 0 && idx >= NENT, 0)) st2_sys(pool + ndw * 4 + (idx - NENT + 1) * 2, en);
                ++idx; bm &= bm - 1;
            }
            if (nov && lane == 0) st2_sys(pool + ndw * 4, nov);
        }
        if (lane < 8) ev = (u32)((lane < 4 ? wide[0] : wide[1]) >> (16 * (lane & 3)));
        if (lane == 8) ev = (u32)T0 | (raw || nov ? 32u : 0u) | (raw ? 64u : 0u);
        if (lane == 9) ev = start;
        if (lane < 16) st2_sys(blk + MAIN_B + wave * ENT_B + lane * 2, ev);
        LDS_AT(u32x4, amb + lane * 16) = u32x4{0, 0, 0, 0};
        LDS_AT(u32x4, amb + 1024 + lane * 16) = u32x4{0, 0, 0, 0};
        WAIT_VM(0);
        u32 a2 = 0;
        if (lane == 0) a2 = __hip_atomic_fetch_add((LAS u32*)(L + L_CTL) + C_ARR, 1u, __ATOMIC_RELAXED, __HIP_MEMORY_SCOPE_WORKGROUP);
        if (rfl(a2) == 3) {
            const u32 n16 = MIN16 + ((ctl[C_ALLOC] + 3) >> 2);
            if (lane < NR && lane != rank) {
                GAS u64* f = (GAS u64*)(fpeer + WFL_OFF + par * WFL_PAR) + ((cbe * 8 + rank) * 4 + kse);
                __hip_atomic_store(f, ((u64)call << 32) | n16, __ATOMIC_RELAXED, __HIP_MEMORY_SCOPE_SYSTEM);
            }
            ctl[C_PUB] = 1;
        }
        } else {
            LDS_AT(u32x4, amb + lane * 16) = u32x4{0, 0, 0, 0};
            LDS_AT(u32x4, amb + 1024 + lane * 16) = u32x4{0, 0, 0, 0};
            WG_BARRIER(call);
            call = call + 1;
            scale = p.sx[0] * p.sw[0];
            asm volatile("" : "+v"(scale));
        }

        const int fw = wave & 3, grp = wave >> 2;
        f32x4 acc[8];
#pragma unroll
        for (int f = 0; f < 8; ++f) acc[f] = f32x4{0, 0, 0, 0};
        int avw = 0, avx = 0;
        const u32 aq0 = lbase + L_W + fw * FR_B + lane * 16, aq1 = lbase + L_W + fw * FR_B + 1024 + lane * 4;
        const u32 awd = lbase + L_W + MAIN_B + fw * ENT_B, aen = awd + 16 + (lane < 8 ? lane : 0) * 2, axx = lbase + L_X + lane * 16;
        const u32 scr = lbase + L_CTL + 64 + lane;
        auto wait_for = [&](int i, int& av) {
            u64 t0 = 0;
            for (u32 it = 1;; ++it) {
                const u32x2 fl = LDS_AT(u32x2, lbase + L_CTL);
                avw = (int)rfl(fl.x) + 4; avx = (int)rfl(fl.y);
                if (i < av) break;
                if (__builtin_expect((it & 1023) == 0, 0)) {
                    const u64 t = rt();
                    if (t0 == 0) t0 = t;
                    if (t - t0 > 2 * DEADLINE) { ctl[C_FAIL] = 1; avw = avx = NSTEP; break; }
                }
                __builtin_amdgcn_s_sleep(1);
            }
            asm volatile("" ::: "memory");
        };
        auto mma8 = [&](int next, const i32x8& wf, const u32x4 (&xa)[16]) {
#pragma unroll
            for (int f = 0; f < 8; ++f) {
                const u32x4 a = xa[2 * f], b = xa[2 * f + 1];
                i32x8 xf;
                xf[0] = a.x; xf[1] = a.y; xf[2] = a.z; xf[3] = a.w; xf[4] = b.x; xf[5] = b.y; xf[6] = b.z; xf[7] = b.w;
                if (f == 1) {
                    asm volatile("" :: "v"(xa[15]), "v"(wf) : "memory");
                    ctl[C_FREE + wave] = next;
                    __builtin_amdgcn_sched_barrier(0);
                }
                acc[f] = __builtin_amdgcn_mfma_scale_f32_16x16x128_f8f6f4(wf, xf, acc[f], 1  , 0  , 0, 0, 0, 0);
            }
        };
        if (grp == NG - 1)
#pragma nounroll
        for (int i = 0; i < 4; ++i) {
            if (i >= avx) wait_for(i, avx);
            u32x2 t0, t1, t2, t3;
            const u32 a = lbase + L_W + i * BLK_B + fw * 16 + (u32)(((lane >> 4) * 16 + ((lane & 15) >> 1)) * 64 + (lane & 1) * 8);
            asm volatile("ds_read_b64_tr_b8 %0, %4\n\tds_read_b64_tr_b8 %1, %4 offset:512\n\t"
                         "ds_read_b64_tr_b8 %2, %4 offset:4096\n\tds_read_b64_tr_b8 %3, %4 offset:4608\n\ts_waitcnt lgkmcnt(0)"
                         : "=&v"(t0), "=&v"(t1), "=&v"(t2), "=&v"(t3) : "v"(a) : "memory");
            u32x4 xa[16];
#pragma unroll
            for (int f = 0; f < 16; ++f) xa[f] = LDS_AT(u32x4, axx + i * XS_B + f * 1024);
            __builtin_amdgcn_sched_barrier(0);
            mma8(i < 3 ? i + 1 : 3 + NG, i32x8{(int)t0.x, (int)t0.y, (int)t1.x, (int)t1.y, (int)t2.x, (int)t2.y, (int)t3.x, (int)t3.y}, xa);
        }
        u32 xo = grp * XS_B, wo = (4 + grp) * BLK_B;
#pragma nounroll
        for (int i = 4 + grp; i < NSTEP; i += NG) {
            if (i >= avx) wait_for(i, avx);
            u32x4 xa[16];
#pragma unroll
            for (int f = 0; f < 16; ++f) xa[f] = LDS_AT(u32x4, axx + xo + f * 1024);
            if (i >= avw) { __builtin_amdgcn_s_setprio(0); wait_for(i, avw); __builtin_amdgcn_s_setprio(PRIO_C); }
            const u32 e0 = LDS_AT(unsigned short, aen + wo);
            const u32x4 q0 = LDS_AT(u32x4, aq0 + wo);
            const u32 q1 = LDS_AT(u32, aq1 + wo);
            const u32x4 wd = LDS_AT(u32x4, awd + wo);
            __builtin_amdgcn_sched_barrier(0);
            const u32 hdr = rfl(e0), pool = lbase + L_W + POOL_B + wo + (rdl(e0, 1) & 1023u) * 4;
            const u32 ea = e0 >> 11 ? amb + (e0 & 2047u) : scr;
            LDS_AT(unsigned char, ea) = (unsigned char)(e0 >> 11 << 2);
            const bool w3[2] = {__builtin_amdgcn_inverse_ballot_w64(((u64)rfl(wd.y) << 32) | rfl(wd.x)),
                                __builtin_amdgcn_inverse_ballot_w64(((u64)rfl(wd.w) << 32) | rfl(wd.z))};
            const u32 cw[2] = {w3[0] ? 3u : 2u, w3[1] ? 3u : 2u};
            const u32 n0 = cw[0] * __builtin_popcount(~(q0.w | (q0.w >> 1)) & 0x55555555u), nn = n0 + cw[1] * __builtin_popcount(~(q1 | (q1 >> 1)) & 0x55555555u);
            const u32 incl = wave_scan_incl(nn), bp = incl - nn;
            const u32 sa = pool + (bp >> 5) * 4;
            const u32 d0 = LDS_AT(u32, sa), d1 = LDS_AT(u32, sa + 4), d2 = LDS_AT(u32, sa + 8);
            if (__builtin_expect((hdr & 96u) == 32u, 0)) {
                const u32 ov = pool + ((rdl(incl, 63) + 31) >> 5) * 4;
                const u32 n = rfl(LDS_AT(unsigned short, ov)) & 511u;
                for (u32 k = 0; k < n; k += 64) {
                    const u32 e = LDS_AT(unsigned short, ov + 2 + (k + lane) * 2);
                    if (k + lane < n) LDS_AT(unsigned char, amb + (e & 2047u)) = (unsigned char)(e >> 11 << 2);
                }
                __builtin_amdgcn_s_waitcnt(0xC07F);
            }
            const u32x4 m0 = LDS_AT(u32x4, amb + lane * 16);
            const u32x4 m1 = LDS_AT(u32x4, amb + 1024 + lane * 16);
            LDS_AT(unsigned char, ea) = 0;
            if (__builtin_expect(hdr & 32u, 0)) { LDS_AT(u32x4, amb + lane * 16) = u32x4{0, 0, 0, 0}; LDS_AT(u32x4, amb + 1024 + lane * 16) = u32x4{0, 0, 0, 0}; }
            __builtin_amdgcn_sched_barrier(0);
            const u32 T4 = (hdr & 31u) * 0x04040404u;
            const u32 LC = (T4 + 0x24201C00u) | 0xFFu, LKlo = T4 + 0x10141828u, LKhi = T4 + 0x0004080Cu;
            u32 pc[8], sel[8];
#pragma unroll
            for (int h = 0; h < 2; ++h) {
                const u32 C = h ? q1 : q0.w;
                u32 Cb = 0x07070707u;
#pragma unroll
                for (int i = 0; i < 4; ++i) {
                    const int j = 4 * h + i;
                    pc[j] = __builtin_amdgcn_perm(0u, LC, (C >> (2 * i)) & 0x03030303u);
                    const u32 e = pc[j] & 0x01010101u, t = e + (e << 8), u = (t << 16) + (t + Cb);
                    sel[j] = __builtin_amdgcn_bitop3_b32(u, pc[j], 0x07070707u, 0xB1);
                    Cb = __builtin_amdgcn_perm(u, u, 0x03030303u);
                }
            }
            const u32 w0 = __builtin_amdgcn_alignbit(d1, d0, bp), w1 = __builtin_amdgcn_alignbit(d2, d1, bp);
            i32x8 wf;
#pragma unroll
            for (int h = 0; h < 2; ++h) {
                const u32 w = h ? __builtin_amdgcn_alignbit(w1, w0, n0) : w0;
                const u32 c4 = 4 * cw[h], s1 = w3[h] ? 10u : 12u, s2 = 8 - cw[h];
                const u32 k1 = w3[h] ? 0x003F003Fu : 0x000F000Fu, k2 = w3[h] ? 0x07070707u : 0x03030303u;
                const u32 wa = __builtin_amdgcn_ubfe(w, 0u, c4), wb = __builtin_amdgcn_ubfe(w, c4, c4);
                const u32 ya = (wa | (wa << s1)) & k1, yb = (wb | (wb << s1)) & k1;
                const u32 Ka = __builtin_amdgcn_perm(LKhi, LKlo, (ya | (ya << s2)) & k2);
                const u32 Kb = __builtin_amdgcn_perm(LKhi, LKlo, (yb | (yb << s2)) & k2);
                const u32x4 m = h ? m1 : m0;
#pragma unroll
                for (int i = 0; i < 4; ++i) wf[4 * h + i] = (__builtin_amdgcn_perm(Kb, Ka, sel[4 * h + i]) & pc[4 * h + i]) ^ m[i];
            }
            wf[0] |= q0.x & 0x83838383u; wf[1] |= __builtin_amdgcn_alignbit(q0.x, q0.x, 3) & 0x83838383u;
            wf[2] |= q0.y & 0x83838383u; wf[3] |= __builtin_amdgcn_alignbit(q0.y, q0.y, 3) & 0x83838383u;
            wf[4] |= q0.z & 0x83838383u; wf[5] |= __builtin_amdgcn_alignbit(q0.z, q0.z, 3) & 0x83838383u;
            wf[6] |= ((q0.x >> 5) & 0x03030303u) | ((q0.y << 1) & 0x80808080u);
            wf[7] |= ((q0.z >> 5) & 0x03030303u) | ((q0.y << 2) & 0x80808080u);
            if (__builtin_expect(hdr & 64u, 0)) {
                wf[0] = q0.x; wf[1] = q0.y; wf[2] = q0.z; wf[3] = q0.w; wf[4] = q1;
                wf[5] = LDS_AT(u32, pool + lane * 4); wf[6] = LDS_AT(u32, pool + 256 + lane * 4); wf[7] = LDS_AT(u32, pool + 512 + lane * 4);
                __builtin_amdgcn_s_waitcnt(0xC07F);
            }
            __builtin_amdgcn_sched_barrier(0);
            if (i >= NSTEP - NG)
                for (u32 it = 0; min_free() < NSTEP - 2 && it < (1u << 26); ++it) __builtin_amdgcn_s_sleep(0);
            mma8(i + NG, wf, xa);
            xo = (xo + NG * XS_B) & (NSX * XS_B - 1);
            wo = wo + NG * BLK_B >= NSW * BLK_B ? wo + NG * BLK_B - NSW * BLK_B : wo + NG * BLK_B;
        }
        constexpr u32 T_0 = L_X, T_1 = L_W + 5 * BLK_B;
        const u32 tw = lbase + (grp ? T_1 : T_0) + (lane & 15) * 256 + (((fw * 4 + (lane >> 4)) ^ (lane & 15)) * 16);
#pragma unroll
        for (int f = 0; f < 8; ++f) LDS_AT(f32x4, tw + f * 4096) = acc[f];
        WAIT_LGKM0();
        ctl[C_FREE + wave] = 2 * NSTEP;
        __builtin_amdgcn_s_setprio(0);
        for (u32 it = 0; min_free() < 2 * NSTEP && it < (1u << 24); ++it) __builtin_amdgcn_s_sleep(ESLEEP);
        const float sc = ctl[C_FAIL] != 0 ? __builtin_nanf("") : scale;
        constexpr int RW = 32 / NG;
        float* o = p.out + (size_t)(rb * 64 + wave * RW + (lane >> 4)) * LDO + cb * 64 + (lane & 15) * 4;
        f32x4 y[RW / 4];
#pragma unroll
        for (int f = 0; f < RW / 4; ++f) {
            const u32 tr = lbase + (wave * RW + f * 4 + (lane >> 4)) * 256 + (((lane & 15) ^ ((f * 4 + (lane >> 4)) & 15)) * 16);
            y[f] = LDS_AT(f32x4, tr + T_0);
            if (NG == 2) y[f] += LDS_AT(f32x4, tr + T_1);
        }
#pragma unroll
        for (int f = 0; f < RW / 4; ++f)
            st16_sys(o + (size_t)f * 4 * LDO, __builtin_bit_cast(u32x4, __builtin_elementwise_maximum(y[f] * sc, f32x4{0, 0, 0, 0})));
    } else if (wave == 8) {
        const u64 tstart = rt();
        WG_BARRIER(call);
        call = call + 1;
        par = call & 1;
        const int R = lane < NREM ? lane : 0, src = src_of_rem(R, rank, cb), ks = R / 7;
        const u64 baddr = (((u64)(u32)__shfl((int)((u64)fpeer >> 32), src) << 32) | (u32)__shfl((int)(u32)(u64)fpeer, src)) + WST_OFF + par * WST_PAR + (u64)(cb * 4 + ks) * BLK_G;
        const u32 blo = (u32)baddr, bhi = (u32)(baddr >> 32);
        GAS u64* fl = (GAS u64*)(p.ws + WFL_OFF + par * WFL_PAR) + ((cb * 8 + src) * 4 + ks);
        const u32 voff = lane * 16;
        u32 n16v = MIN16;
        bool dead = false;
        const u32 xcc = __builtin_amdgcn_s_getreg((3 << 11) | 20) & 15;
        GAS u64* inv = (GAS u64*)(p.ws + INV_OFF + xcc * 256);
        u32 prev = 0;
        if (lane == 0) prev = __hip_atomic_fetch_max((GAS u32*)(inv + 16), call, __ATOMIC_RELAXED, __HIP_MEMORY_SCOPE_AGENT);
        if (rfl(prev) < call) {
            __builtin_amdgcn_fence(__ATOMIC_ACQUIRE, "");
            WAIT_VM(0);
            if (lane == 0) __hip_atomic_store(inv, (u64)call << 32, __ATOMIC_RELAXED, __HIP_MEMORY_SCOPE_AGENT);
        }
        if (lane >= 28 && lane < 56) fl = (GAS u64*)(p.ws + XFL_OFF + par * XFL_PAR) + ((rb * 8 + src_of_rem(lane, rank, 0)) * 4 + (lane - 28) / 7);
        if (lane == 62) fl = (GAS u64*)(p.ws + TMO_OFF);
        if (lane == 63) fl = inv;
        for (u32 it = 0; rfl(ctl[C_PUB]) == 0 && it < (1u << 22); ++it) __builtin_amdgcn_s_sleep(4);
        while (rt() - tstart < PTIME) __builtin_amdgcn_s_sleep(4);
        {
            const u64 t0 = rt();
            for (u32 it = 1;; ++it) {
                const u64 v = __hip_atomic_load(fl, __ATOMIC_RELAXED, __HIP_MEMORY_SCOPE_SYSTEM);
                n16v = (u32)v;
                const u64 up = __ballot((u32)(v >> 32) >= call);
                if (((up >> 62) & 1) && rdl((u32)(v >> 32), 62) == call) { dead = true; break; }
                if ((up | 0x7FFFFFFFF0000000ull) == ~0ull && (__builtin_popcount((u32)(up >> 28) & 0xFFFFFFFu) >= XTH || rt() - t0 > 2000)) break;
                if ((it & 15) == 0 && expired(t0)) { dead = true; break; }
                __builtin_amdgcn_s_sleep(PSLEEP);
            }
            n16v = n16v < MIN16 ? MIN16 : (n16v > MAX16 ? MAX16 : n16v);
        }
        __builtin_amdgcn_s_setprio(PRIO_W);
        int pub = 0, freed = 0;
        u32 lo = lbase + L_W + 4 * BLK_B;
        auto publish = [&](int upto) { if (upto > pub) { pub = upto; ctl[C_FULLW] = upto; } };
        u64 g[LAST + 1]; u32 l[LAST + 1];
#pragma unroll
        for (int j = 0; j <= LAST; ++j) { g[j] = 0; l[j] = 0; }
#define W_LOAD(OFF, J, K, NOP) do { dma16<OFF, 0, NOP>(g[J], voff, l[J]); WAIT_VM(WIN); after(std::integral_constant<int, K>{}); } while (0)
        auto rest_of_slot = [&](int n, bool mid, auto&& after) {
            if (mid) { W_LOAD(2048, LAG, 3, 0); W_LOAD(3072, LAG, 4, 0); }
            if (n >= LAST) {
                g[LAST] += 4096; l[LAST] += 4096;
                W_LOAD(0, LAST, 5, 0);
                const u32 n16 = rdl(n16v, n - LAST);
                if (__builtin_expect(n16 > 64, 0))
                    for (u32 i = 64; i < n16; i += 64) { if (lane + i < n16) dma16<1024, 0>(g[LAST] + i * 16, voff, l[LAST] + i * 16); WAIT_VM(WIN); }
                if ((u32)lane < n16) dma16<1024, 0>(g[LAST], voff, l[LAST]);
                WAIT_VM(WIN); after(std::integral_constant<int, SL>{});
            }
#pragma unroll
            for (int j = LAST; j > 0; --j) { g[j] = g[j - 1]; l[j] = l[j - 1]; }
        };
        auto open = [&](int n, int done) {
            if (freed <= n + 4 - NSW) {
                const int need = n + 4 - NSW;
                const u64 t0 = rt();
                bool drained = false;
                for (;;) {
                    freed = min_free();
                    if (freed > need) break;
                    if (!drained) { WAIT_VM(0); publish(done); drained = true; continue; }
                    if (rt() - t0 > 2 * DEADLINE) { dead = true; break; }
                    __builtin_amdgcn_s_sleep(1);
                }
                if (dead) return;
            }
            g[0] = ((u64)rdl(bhi, n) << 32) | rdl(blo, n);
            l[0] = lo;
            lo = lo + BLK_B == lbase + L_W + NSW * BLK_B ? lbase + L_W : lo + BLK_B;
        };
        for (int n = 0; n < NB; ++n) {
            constexpr int PK = (WIN - 1) % SL + 1, PL = LAST + 1 + (WIN - PK) / SL;
            auto after = [&](auto k) { if constexpr (decltype(k)::value == PK) publish(n - PL + 1); };
            if (!dead) open(n, n - LAST);
            if (dead) break;
            W_LOAD(0, 0, 1, 4); W_LOAD(1024, 0, 2, 0);
            rest_of_slot(n, n >= LAG, after);
        }
        if (!dead) {
            static_for<LAST>([&](auto dd) {
                constexpr int d = decltype(dd)::value;
                rest_of_slot(NB + d, d < LAG, [&](auto k) { publish(std::integral_constant<int, w_landed(w_cum(d) - (SL - decltype(k)::value), WIN)>::value); });
            });
            for (int n = NB; n < NREM; ++n) {
                auto after = [&](auto k) {
                    constexpr int first = w_landed(w_cum(LAST) - (SL - decltype(k)::value), WIN);
                    if constexpr (decltype(k)::value <= WIN) publish(decltype(k)::value == WIN ? n : first);
                };
                open(n, n);
                if (dead) break;
                W_LOAD(0, 0, 1, 4); W_LOAD(1024, 0, 2, 0);
                g[LAG] = g[LAST] = g[0]; l[LAG] = l[LAST] = l[0];
                rest_of_slot(n + LAST, true, after);
            }
        }
        if (dead) { ctl[C_FAIL] = 1; WAIT_VM(0); ctl[C_FULLW] = NREM; }
        else { WAIT_VM(0); publish(NREM); }
    } else if (wave == 9) {
        WG_BARRIER(call);
        call = call + 1;
        par = call & 1;
        const int R = max((lane & 31) - 4, 0), src = src_of_rem(R, rank, cb), ks = R / 7;
        const u32 toff = (u32)((src * 4 + ks) * 8 + rb) * XT_B;
        const u64 xbase = (u64)p.ws + XRV_OFF + par * XRV_PAR;
        GAS u64* fl = (GAS u64*)(p.ws + XFL_OFF + par * XFL_PAR) + (((rb + (lane >> 5)) * 8 + src) * 4 + ks);
        constexpr u32 OWN = 0xFu;
        const u32 voff = lane * 16;
        const u32 xoff = (u32)(lane & 15) * LDX + (lane >> 4) * 16;
        u32 ready = OWN;
        u32 it = 0, la = lbase + L_X;
        bool dead = false;
        const u64 t0 = rt();
        int pub = 0, freed = 0;
        for (u32 k = 0; rfl(ctl[C_PUB]) == 0 && k < (1u << 22); ++k) __builtin_amdgcn_s_sleep(4);
        for (int s = 0; s < NSTEP && !dead; ++s) {
            if (!((ready >> s) & 1)) {
                WAIT_VM(0);
                if (pub < s) { pub = s; ctl[C_FULLX] = s; }
                for (;;) {
                    const u64 v = __hip_atomic_load(fl, __ATOMIC_RELAXED, __HIP_MEMORY_SCOPE_SYSTEM);
                    const u64 ok = __ballot((u32)(v >> 32) >= call);
                    ready = ((u32)ok & (u32)(ok >> 32)) | OWN;
                    if (ready == ~0u) { dead = rfl(__hip_atomic_load(tmo, __ATOMIC_RELAXED, __HIP_MEMORY_SCOPE_AGENT)) == call; break; }
                    if ((ready >> s) & 1) break;
                    if ((++it & 15) == 0 && expired(t0)) { dead = true; break; }
                    __builtin_amdgcn_s_sleep(XSLEEP);
                }
            }
            while (!dead && freed <= s - NSX) {
                freed = min_free();
                if (freed > s - NSX) break;
                if ((++it & 1023) == 0 && rt() - t0 > 2 * DEADLINE) dead = true;
                __builtin_amdgcn_s_sleep(1);
            }
            if (dead) break;
            if (s < 4) {
                const u64 b = (u64)p.x + (u64)(rank * 512 + rb * 64) * LDX + s * 128;
#pragma unroll
                for (int i = 0; i < 16; ++i) dma16<0, 0>(b + (i >> 1) * 16 * LDX + (i & 1) * 64, xoff, la + i * 1024);
#pragma unroll
                for (int i = 0; i < 8; ++i)
                    dma16<0, 0>((u64)p.w + (u64)(s * 128 + i * 16) * LDW + cb * 64, (u32)(lane >> 2) * LDW + (lane & 3) * 16, lbase + L_W + s * BLK_B + i * 1024);
            } else {
                const u64 b = xbase + rdl(toff, s);
#pragma unroll
                for (int i = 0; i < 4; ++i) {
                    dma16<0, 1, 4>(b + i * 4096, voff, la + i * 4096); dma16<1024, 1>(b + i * 4096, voff, la + i * 4096);
                    dma16<2048, 1>(b + i * 4096, voff, la + i * 4096); dma16<3072, 1>(b + i * 4096, voff, la + i * 4096);
                }
            }
            la = la + XS_B == lbase + L_X + NSX * XS_B ? lbase + L_X : la + XS_B;
            WAIT_VM(16 * (XD - 1));
            if (pub < s - XD + 2) { pub = s - XD + 2; ctl[C_FULLX] = pub; }
        }
        if (dead) { ctl[C_FAIL] = 1; WAIT_VM(0); }
        else {
            WAIT_VM(32); ctl[C_FULLX] = max(pub, NSTEP - 2);
            WAIT_VM(16); ctl[C_FULLX] = max(pub, NSTEP - 1);
            WAIT_VM(0);
        }
        ctl[C_FULLX] = NSTEP;
    } else {
        const int id = bid / 7, d = (rank + 1 + bid % 7) & 7;
        u32x4 v[8];
        if (id < 32) {
            const unsigned char* g = p.x + (size_t)(d * 512 + (id & 7) * 64 + (lane & 15)) * LDX + (id >> 3) * 128 + (lane >> 4) * 16;
#pragma unroll
            for (int i = 0; i < 8; ++i) v[i] = *(const GAS u32x4*)(g + (size_t)(i >> 1) * 16 * LDX + (i & 1) * 64);
        }
        WG_BARRIER(call);
        call = call + 1;
        par = call & 1;
        if (id < 32) {
            asm volatile("" : "+v"(v[0]), "+v"(v[1]), "+v"(v[2]), "+v"(v[3]), "+v"(v[4]), "+v"(v[5]), "+v"(v[6]), "+v"(v[7]), "+v"(fpeer));
            char* peer = (char*)(((u64)rdl((u32)((u64)fpeer >> 32), d) << 32) | rdl((u32)(u64)fpeer, d));
            char* dst = peer + XRV_OFF + par * XRV_PAR + (size_t)(rank * 32 + id) * XT_B + lane * 16;
#pragma unroll
            for (int i = 0; i < 8; ++i) { st16_sys(dst + i * 1024, v[i]); WAIT_VM(PUSH_W - 1); }
            WAIT_VM(0);
            if (lane == 0)
                __hip_atomic_store((GAS u64*)(peer + XFL_OFF + par * XFL_PAR) + (((id & 7) * 8 + rank) * 4 + (id >> 3)), (u64)call << 32,
                                   __ATOMIC_RELAXED, __HIP_MEMORY_SCOPE_SYSTEM);
        }
    }
    if (lane == 0) __hip_atomic_store(cntp, call, __ATOMIC_RELAXED, __HIP_MEMORY_SCOPE_AGENT);
}

extern "C" void kernel_launch(int rank, int nranks, ncclComm_t comm,
                              hipStream_t stream,
                              void* const* d_in, const int* in_sizes, int n_in,
                              void* d_out, int out_size,
                              const int* in_dtypes, int out_dtype,
                              void* const* d_out_all,
                              void* const* d_ws_all, size_t ws_bytes) {
    (void)comm; (void)in_dtypes; (void)out_dtype; (void)d_out_all;
    if (nranks != NR || (unsigned)rank >= (unsigned)NR || n_in != 4 || ws_bytes < WS_NEED ||
        in_sizes[0] != 4096 * 512 || in_sizes[1] != 512 * 4096 || out_size != 512 * 4096) {
        fprintf(stderr, "unexpected problem shape\n");
        return;
    }
    HIP_CHECK(hipFuncSetAttribute((const void*)gemm_xchg, hipFuncAttributeMaxDynamicSharedMemorySize, LDS_BYTES));
    Params p;
    p.x = (const unsigned char*)d_in[0];
    p.w = (const unsigned char*)d_in[1];
    p.sx = (const float*)d_in[2];
    p.sw = (const float*)d_in[3];
    p.out = (float*)d_out;
    p.ws_all = d_ws_all;
    p.ws = (char*)d_ws_all[rank];
    p.rank = rank;
    p.pad = 0;
    hipLaunchKernelGGL(gemm_xchg, dim3(NWG), dim3(NTHR), LDS_BYTES, stream, p);
    HIP_CHECK(hipGetLastError());
}
```
